```python
import numpy as np

kernel_name = "gpu_a2a_gemm_m8192_k8192_n8192_bf16_relu"


ABSMAX_RTOL = 0.008

COMPUTE_DTYPE = "bfloat16"
EPILOGUE = "relu"
IN_QUANT = None
OUT_QUANT = None
IN_DTYPES = {"x": "bfloat16", "w_mat": "bfloat16"}
WORLD_SIZE = 8

def _to_bf16(x):
    u = np.ascontiguousarray(x, dtype=np.float32).view(np.uint32)
    return ((u + 0x7FFF + ((u >> 16) & 1)) & 0xFFFF0000).view(np.float32)

def setup_inputs():

    x = _to_bf16(np.random.randn(8192, 8192).astype(np.float32))
    w_mat = _to_bf16((np.random.randn(8192, 8192) / np.sqrt(8192)).astype(np.float32))
    return {"x": x, "w_mat": w_mat}

def _epilogue(y):
    return np.maximum(y, 0.0).astype(np.float32)

def reference(x, w_mat):
    y = (x @ w_mat).astype(np.float32)
    return _epilogue(y)
```

```cpp
#include <hip/hip_runtime.h>
#include <rccl/rccl.h>
#include <cstdio>

#define HIP_CHECK(expr) do { hipError_t _e = (expr); if (_e != hipSuccess) { \
    fprintf(stderr, "HIP error %s at %s:%d\n", hipGetErrorString(_e), __FILE__, __LINE__); \
    return; } } while (0)

#define LAS __attribute__((address_space(3)))
typedef short bf16x8 __attribute__((ext_vector_type(8)));
typedef short bf16x4 __attribute__((ext_vector_type(4)));
typedef float f32x4 __attribute__((ext_vector_type(4)));
typedef unsigned u32x4 __attribute__((ext_vector_type(4)));

#define STAMP 0
#define PACE 13

constexpr int NRANK = 8;
constexpr int NCOL = 8192;
constexpr int HTB = 16384;
constexpr int LDS_BYTES = 8 * HTB;
constexpr size_t BLK_B = (size_t)1024 * 1024 * 2;
constexpr size_t WS_CTR = 0;
constexpr size_t WS_TMO = 32768;
constexpr size_t WS_XFLAG = 65536;
constexpr size_t WS_AFLAG = 131072;
constexpr size_t WS_A = (size_t)1 << 20;
constexpr size_t WS_XCH = (size_t)33 << 20;
constexpr size_t WS_DBG = (size_t)50 << 20;
constexpr size_t WS_NEED = (size_t)52 << 20;
constexpr unsigned long long DEADLINE = 50000000ull;

struct Params {
  const char* x;
  const char* w;
  char* out;
  char* ws;
  char* peer[7];
  int rank;
  int pad;
};

__device__ __forceinline__ unsigned cvt_pk_bf16(float lo, float hi) { unsigned r; asm("v_cvt_pk_bf16_f32 %0, %1, %2" : "=v"(r) : "v"(lo), "v"(hi)); return r; }
__device__ __forceinline__ void glds_sys(unsigned ldsaddr, unsigned voff, const char* base) {
  asm volatile("s_mov_b32 m0, %0\n\ts_nop 0\n\tglobal_load_lds_dwordx4 %1, %2 sc0 sc1" :: "s"(ldsaddr), "v"(voff), "s"(base) : "memory");
}
__device__ __forceinline__ void glds(unsigned ldsaddr, unsigned voff, const char* base) {
  asm volatile("s_mov_b32 m0, %0\n\ts_nop 0\n\tglobal_load_lds_dwordx4 %1, %2" :: "s"(ldsaddr), "v"(voff), "s"(base) : "memory");
}

__device__ __forceinline__ void slow_wait(const char* base, unsigned voff, unsigned call, unsigned* tmo) {
  const unsigned long long t0 = __builtin_amdgcn_s_memrealtime();
  for (;;) {
    unsigned f;
    asm volatile("global_load_dword %0, %1, %2 sc0 sc1\n\ts_waitcnt vmcnt(0)" : "=&v"(f) : "v"(voff), "s"(base) : "memory");
    if (__builtin_amdgcn_ballot_w64(f < call) == 0ull) return;
    if (__builtin_amdgcn_readfirstlane(__hip_atomic_load(tmo, __ATOMIC_RELAXED, __HIP_MEMORY_SCOPE_AGENT)) == call
        || __builtin_amdgcn_s_memrealtime() - t0 > DEADLINE) {
      __hip_atomic_store(tmo, call, __ATOMIC_RELAXED, __HIP_MEMORY_SCOPE_AGENT);
      return;
    }
    __builtin_amdgcn_s_sleep(4);
  }
}

#define ST(i) do { if (STAMP && threadIdx.x == 0) { st_r[i] = __builtin_amdgcn_s_memrealtime(); } } while (0)

__global__ __launch_bounds__(512, 2) void a2a_gemm(Params p) {
  extern __shared__ __attribute__((aligned(16))) char smem[];
  LAS char* lds = (LAS char*)smem;
  const int tid = threadIdx.x, lane = tid & 63;
  const int wid = __builtin_amdgcn_readfirstlane(tid >> 6);
  const int wr = wid >> 2, wc = wid & 3, fr = lane & 15, fq = lane >> 4;
  const int bid = blockIdx.x;
  const int xcd = bid & 7, idx = bid >> 3;
  const int h = xcd & 1, pm = idx & 3, pn = (xcd >> 1) * 8 + (idx >> 2);
  const int rank = p.rank;
  unsigned long long st_r[8]; unsigned nslow = 0;
  ST(0);

  unsigned* ctr = (unsigned*)(p.ws + WS_CTR) + bid * 32 + wid;
  unsigned* tmo = (unsigned*)(p.ws + WS_TMO);
  unsigned cv; const unsigned vzero = 0;
  asm volatile("global_load_dword %0, %1, %2 sc1" : "=v"(cv) : "v"(vzero), "s"(ctr) : "memory");

  unsigned voffA, voffB;
  { const int b = tid * 16; const int row = b >> 7, ch = ((b >> 4) & 7) ^ ((row >> 1) & 7); voffA = row * 2048 + ch * 16; }
  { const int b = tid * 16; const int row = b >> 8, c16 = (b >> 4) & 15; const int f = (row & 3) | (((row >> 3) & 1) << 2);
    voffB = row * 16384 + (((c16 >> 1) ^ f) * 32) + (c16 & 1) * 16; }
  const unsigned voffF = (lane & 3) * 4;
  const unsigned ldsw = (unsigned)(size_t)lds + wid * 1024;
  const int aoff0 = (wr * 64 + fr) * 128 + ((fq ^ (fr >> 1)) * 16), aoff1 = aoff0 ^ 64;
  const int hh = lane >> 5, g1 = (lane >> 4) & 1, q = (lane >> 2) & 3, pp = lane & 3, ff = q | (g1 << 2);
  int boff0 = 4 * HTB + (16 * hh + 8 * g1 + q) * 256 + (((2 * wc) ^ ff) * 32) + pp * 8; int boff1 = boff0 ^ 32;
  asm volatile("" : "+v"(boff0), "+v"(boff1));

#define SA(b, hf) (((b) * 2 + (hf)) * HTB)
#define SB(b, hf) (((b) * 2 + (hf)) * HTB)
#define STAGE_A(b, hf, g) do { glds_sys(ldsw + SA(b, hf), voffA, (g) + (hf) * (128 * 2048)); glds_sys(ldsw + SA(b, hf) + 8192, voffA, (g) + (hf) * (128 * 2048) + 64 * 2048); } while (0)
#define STAGE_B(b, hf, g) do { glds(ldsw + 4 * HTB + SB(b, hf), voffB, (g) + (hf) * 256); glds(ldsw + 4 * HTB + SB(b, hf) + 8192, voffB, (g) + (hf) * 256 + 32 * 16384); } while (0)
#define LDA(dst, b, hf) do { _Pragma("unroll") for (int m = 0; m < 4; ++m) { dst[m][0] = *(const LAS bf16x8*)(lds + SA(b, hf) + aoff0 + m * 2048); dst[m][1] = *(const LAS bf16x8*)(lds + SA(b, hf) + aoff1 + m * 2048); } } while (0)
#define TRR(off) __builtin_amdgcn_ds_read_tr16_b64_v4i16((LAS bf16x4*)(lds + (off)))
#define LDB(dst, b, hf) do { _Pragma("unroll") for (int k = 0; k < 2; ++k) { \
    { bf16x4 lo = TRR(SB(b, hf) + boff0 + k * 8192), hi = TRR(SB(b, hf) + boff0 + k * 8192 + 1024); dst[0][k] = __builtin_shufflevector(lo, hi, 0, 1, 2, 3, 4, 5, 6, 7); } \
    { bf16x4 lo = TRR(SB(b, hf) + boff1 + k * 8192), hi = TRR(SB(b, hf) + boff1 + k * 8192 + 1024); dst[1][k] = __builtin_shufflevector(lo, hi, 0, 1, 2, 3, 4, 5, 6, 7); } } } while (0)
#define SCHED __builtin_amdgcn_sched_barrier(0)
#define WAIT_V(n) asm volatile("s_waitcnt vmcnt(%0)" :: "n"(n) : "memory")
#define WAIT_L0 __builtin_amdgcn_s_waitcnt(0xC07F)
#define BAR __builtin_amdgcn_s_barrier()
#define MF1(ai, bj, Bt, m, n, k) acc[ai][bj][m][n] = __builtin_amdgcn_mfma_f32_16x16x32_bf16(Bt[n][k], At[m][k], acc[ai][bj][m][n], 0, 0, 0)
#define MMA32(ai) do { __builtin_amdgcn_s_setprio(1); \
    _Pragma("unroll") for (int i_ = 0; i_ < 32; ++i_) { const int k = i_ & 1, m = i_ >> 3, n = (i_ >> 1) & 1; \
      if ((i_ >> 2) & 1) MF1(ai, 1, B1, m, n, k); else MF1(ai, 0, B0, m, n, k); SCHED; } \
    __builtin_amdgcn_s_setprio(0); BAR; SCHED; } while (0)

  const char* aOwn = p.x + (size_t)rank * BLK_B + (size_t)pm * (256 * 2048);
  const char* wB = p.w + pn * 512;
  const char* fB = p.ws + WS_AFLAG + pm * 128;

  f32x4 acc[2][2][4][2];
#pragma unroll
  for (int a = 0; a < 2; ++a)
#pragma unroll
    for (int b = 0; b < 2; ++b)
#pragma unroll
      for (int m = 0; m < 4; ++m)
#pragma unroll
        for (int n = 0; n < 2; ++n) acc[a][b][m][n] = (f32x4){0.f, 0.f, 0.f, 0.f};
  bf16x8 At[4][2], B0[2][2], B1[2][2];

  const char* a1 = aOwn + (h * 8 + 1) * 128;
  {
    const char* a0 = aOwn + (h * 8) * 128;
    const char* b0 = wB + (size_t)(rank * 1024 + h * 512) * 16384; const char* b1 = b0 + 64 * 16384;
    STAGE_B(0, 0, b0); STAGE_B(0, 1, b0); STAGE_A(0, 0, a0); STAGE_A(0, 1, a0);
    unsigned dummy; asm volatile("global_load_dword %0, %1, %2 sc0 sc1" : "=v"(dummy) : "v"(voffF), "s"(fB) : "memory");
    STAGE_B(1, 0, b1); STAGE_B(1, 1, b1); STAGE_A(1, 0, a1);
    asm volatile("s_waitcnt vmcnt(6)" : "+v"(cv), "+v"(dummy) :: "memory");
  }
  const unsigned call = __builtin_amdgcn_readfirstlane(cv) + 1u;
  const char* aRem = p.ws + WS_A + (size_t)(call & 1u) * (8 * BLK_B) + (size_t)pm * (256 * 2048);
  auto tile_ptrs = [&](int t, const char*& a, const char*& b, const char*& f) {
    if (t > 63) t = 63;
    int src, j;
    if (t < 8) { src = rank; j = h * 8 + t; a = aOwn + j * 128; }
    else { const int v = 2 * (t - 8) + h; j = (v * 147) >> 10; src = (rank + 1 + v - j * 7) & 7; a = aRem + (size_t)src * BLK_B + j * 128; }
    b = wB + (size_t)(src * 1024 + j * 64) * 16384;
    f = fB + (src * 16 + j) * 512;
  };
  BAR;
  if (wr == 1) BAR;
  ST(1);
  WAIT_L0;

  unsigned F0 = 0xffffffffu, F1 = 0xffffffffu;
  const char *a2, *b2, *f2;
  tile_ptrs(2, a2, b2, f2);
  u32x4 P[7];
  unsigned voffP;

#define KTILE(b, EV, t, aN1, aN2, bN2, fN2, Fchk, fN3, Fld) do { \
    LDB(B0, b, 0); LDA(At, b, 0); LDB(B1, b, 1); SCHED; \
    if (EV == 4 && wid == 7) { \
      char* fp = lane == 0 ? p.peer[0] : lane == 1 ? p.peer[1] : lane == 2 ? p.peer[2] : lane == 3 ? p.peer[3] : lane == 4 ? p.peer[4] : lane == 5 ? p.peer[5] : p.peer[6]; \
      fp += WS_AFLAG + ((rank * 16 + (bid >> 4)) * 4 + ((bid >> 2) & 3)) * 128 + (bid & 3) * 4; \
      if (lane < 7) asm volatile("global_store_dword %0, %1, off sc0 sc1" :: "v"(fp), "v"(call) : "memory"); } \
    STAGE_A((b) ^ 1, 1, aN1); \
    asm volatile("global_load_dword %0, %1, %2 sc0 sc1" : "=v"(Fld) : "v"(voffF), "s"(fN3) : "memory"); \
    if (EV == 1) { \
      voffP = ((bid & 15) * 64 + wid * 8 + (lane >> 3)) * 2048 + (bid >> 4) * 128 + (lane & 7) * 16; \
      _Pragma("unroll") for (int j = 0; j < 7; ++j) \
        asm volatile("global_load_dwordx4 %0, %1, %2" : "=v"(P[j]) : "v"(voffP), "s"(p.x + (size_t)((rank + 1 + j) & 7) * BLK_B) : "memory"); } \
    if (EV == 2) asm volatile("s_waitcnt vmcnt(9)" : "+v"(Fchk), "+v"(P[0]), "+v"(P[1]), "+v"(P[2]), "+v"(P[3]), "+v"(P[4]), "+v"(P[5]), "+v"(P[6]) :: "memory"); \
    else asm volatile("s_waitcnt vmcnt(%1)" : "+v"(Fchk) : "n"((EV == 1 || EV == 3) ? 16 : 9) : "memory"); \
    WAIT_L0; SCHED; BAR; SCHED; MMA32(0); \
    if ((t) + 2 >= 8 && __builtin_amdgcn_ballot_w64(Fchk < call) != 0ull) { slow_wait(fN2, voffF, call, tmo); if (STAMP) ++nslow; } \
    if (EV == 2) { \
      _Pragma("unroll") for (int j = 0; j < 7; ++j) \
        asm volatile("global_store_dwordx4 %0, %1, %2 sc0 sc1\n\ts_nop 1" :: "v"(voffP), "v"(P[j]), "s"(p.peer[j] + WS_A + (size_t)(call & 1u) * (8 * BLK_B) + (size_t)rank * BLK_B) : "memory"); } \
    LDA(At, b, 1); SCHED; \
    STAGE_B(b, 0, bN2); STAGE_B(b, 1, bN2); STAGE_A(b, 0, aN2); \
    WAIT_V((EV == 1 || EV == 2) ? 16 : 9); \
    WAIT_L0; SCHED; BAR; SCHED; MMA32(1); } while (0)
#define ITER(E0, E1, t) do { \
    const char *a3, *b3, *f3, *a4, *b4, *f4; \
    tile_ptrs((t) + 3, a3, b3, f3); \
    KTILE(0, E0, (t), a1, a2, b2, f2, F1, f3, F0); \
    tile_ptrs((t) + 4, a4, b4, f4); \
    KTILE(1, E1, (t) + 1, a2, a3, b3, f3, F0, f4, F1); \
    a1 = a3; a2 = a4; b2 = b4; f2 = f4; } while (0)

  const int tA = 2 * (bid < 32 ? (bid >> 4) : 2 + (bid - 32) / PACE);
  int t = 0;
  for (; t < tA; t += 2) ITER(0, 0, t);
  ST(2);
  ITER(1, 2, t); t += 2;
  ITER(3, 4, t); t += 2;
  ST(3);
  for (; t < 64; t += 2) ITER(0, 0, t);
  asm volatile("s_waitcnt vmcnt(0)" : "+v"(F0), "+v"(F1) :: "memory");
  if (wr == 0) BAR;
  ST(4);

  int tz = threadIdx.x; asm volatile("" : "+v"(tz));
  const int ln = tz & 63;
  const int partner = bid ^ 1;
  {
    char* dst = p.ws + WS_XCH + (size_t)partner * 65536 + wid * 8192 + ln * 16;
#pragma unroll
    for (int bj = 0; bj < 2; ++bj)
#pragma unroll
      for (int m = 0; m < 4; ++m) {
        const f32x4 v0 = h ? acc[0][bj][m][0] : acc[1][bj][m][0], v1 = h ? acc[0][bj][m][1] : acc[1][bj][m][1];
        u32x4 w; w.x = cvt_pk_bf16(v0[0], v0[1]); w.y = cvt_pk_bf16(v0[2], v0[3]); w.z = cvt_pk_bf16(v1[0], v1[1]); w.w = cvt_pk_bf16(v1[2], v1[3]);
        asm volatile("global_store_dwordx4 %0, %1, off sc0 sc1\n\ts_nop 1" :: "v"(dst + (bj * 4 + m) * 1024), "v"(w) : "memory");
      }
  }
  if (ln == 0) __hip_atomic_store(ctr, call, __ATOMIC_RELAXED, __HIP_MEMORY_SCOPE_AGENT);
  WAIT_V(0);
  __syncthreads();
  ST(5);
  if (wid == 0) {
    if (ln == 0) __hip_atomic_store((unsigned*)(p.ws + WS_XFLAG) + partner * 32, call, __ATOMIC_RELAXED, __HIP_MEMORY_SCOPE_SYSTEM);
    slow_wait(p.ws + WS_XFLAG + bid * 128, 0u, call, tmo);
  }
  __syncthreads();
  ST(6);
  {
    const char* src = p.ws + WS_XCH + (size_t)bid * 65536 + wid * 8192 + ln * 16;
    const int fr2 = ln & 15, fq2 = ln >> 4;
    char* orow = p.out + (size_t)(pm * 256 + h * 128 + wr * 64 + fr2) * (NCOL * 2) + (pn * 256 + wc * 32) * 2 + ((fq2 & 1) * 32 + (fq2 >> 1) * 16);
    u32x4 r[8]; unsigned tv;
#pragma unroll
    for (int i = 0; i < 8; ++i) asm volatile("global_load_dwordx4 %0, %1, off sc0 sc1" : "=v"(r[i]) : "v"(src + i * 1024) : "memory");
    asm volatile("global_load_dword %0, %1, %2 sc1" : "=v"(tv) : "v"(vzero), "s"(tmo) : "memory");
    asm volatile("s_waitcnt vmcnt(0)" : "+v"(r[0]), "+v"(r[1]), "+v"(r[2]), "+v"(r[3]), "+v"(r[4]), "+v"(r[5]), "+v"(r[6]), "+v"(r[7]), "+v"(tv) :: "memory");
    const bool bad = tv == call;
#pragma unroll
    for (int bj = 0; bj < 2; ++bj)
#pragma unroll
      for (int m = 0; m < 4; ++m) {
        f32x4 v0 = h ? acc[1][bj][m][0] : acc[0][bj][m][0], v1 = h ? acc[1][bj][m][1] : acc[0][bj][m][1];
        const u32x4 w = r[bj * 4 + m];
        v0[0] += __uint_as_float(w.x << 16); v0[1] += __uint_as_float(w.x & 0xffff0000u); v0[2] += __uint_as_float(w.y << 16); v0[3] += __uint_as_float(w.y & 0xffff0000u);
        v1[0] += __uint_as_float(w.z << 16); v1[1] += __uint_as_float(w.z & 0xffff0000u); v1[2] += __uint_as_float(w.w << 16); v1[3] += __uint_as_float(w.w & 0xffff0000u);
#pragma unroll
        for (int j = 0; j < 4; ++j) { v0[j] = v0[j] < 0.f ? 0.f : v0[j]; v1[j] = v1[j] < 0.f ? 0.f : v1[j]; }
        const unsigned x0 = cvt_pk_bf16(v0[0], v0[1]), y0 = cvt_pk_bf16(v0[2], v0[3]), x1 = cvt_pk_bf16(v1[0], v1[1]), y1 = cvt_pk_bf16(v1[2], v1[3]);
        const auto sx = __builtin_amdgcn_permlane16_swap(x0, x1, false, false);
        const auto sy = __builtin_amdgcn_permlane16_swap(y0, y1, false, false);
        u32x4 o; o.x = sx[0]; o.y = sy[0]; o.z = sx[1]; o.w = sy[1];
        if (bad) o = (u32x4){0x7fc07fc0u, 0x7fc07fc0u, 0x7fc07fc0u, 0x7fc07fc0u};
        *(u32x4*)(orow + (size_t)m * 16 * (NCOL * 2) + bj * 256) = o;
      }
  }
  if (STAMP) { WAIT_V(0); ST(7); if (threadIdx.x == 0) { unsigned long long* d = (unsigned long long*)(p.ws + WS_DBG) + bid * 16; for (int i = 0; i < 8; ++i) d[i] = st_r[i]; d[8] = nslow; d[9] = call; } }
}

extern "C" void kernel_launch(int rank, int nranks, ncclComm_t comm,
                              hipStream_t stream,
                              void* const* d_in, const int* in_sizes, int n_in,
                              void* d_out, int out_size,
                              const int* in_dtypes, int out_dtype,
                              void* const* d_out_all,
                              void* const* d_ws_all, size_t ws_bytes) {
  if (nranks != NRANK || (unsigned)rank >= (unsigned)NRANK || n_in < 2 || ws_bytes < WS_NEED ||
      in_sizes[0] != 8192 * 1024 || in_sizes[1] != 8192 * 8192 || out_size != 1024 * 8192) {
    fprintf(stderr, "a2a_gemm: unexpected problem shape\n");
    return;
  }
  HIP_CHECK(hipFuncSetAttribute((const void*)a2a_gemm, hipFuncAttributeMaxDynamicSharedMemorySize, LDS_BYTES));
  Params p;
  p.x = (const char*)d_in[0];
  p.w = (const char*)d_in[1];
  p.out = (char*)d_out;
  p.ws = (char*)d_ws_all[rank];
  for (int j = 0; j < 7; ++j) p.peer[j] = (char*)d_ws_all[(rank + 1 + j) & 7];
  p.rank = rank;
  p.pad = 0;
  a2a_gemm<<<256, 512, LDS_BYTES, stream>>>(p);
  HIP_CHECK(hipGetLastError());
}
```

```cpp
#include <hip/hip_runtime.h>
#include <rccl/rccl.h>
#include <cstdio>

#define HIP_CHECK(expr) do { hipError_t _e = (expr); if (_e != hipSuccess) { \
    fprintf(stderr, "HIP error %s at %s:%d\n", hipGetErrorString(_e), __FILE__, __LINE__); \
    return; } } while (0)

#define LAS __attribute__((address_space(3)))
typedef short bf16x8 __attribute__((ext_vector_type(8)));
typedef short bf16x4 __attribute__((ext_vector_type(4)));
typedef float f32x4 __attribute__((ext_vector_type(4)));
typedef float f32x2 __attribute__((ext_vector_type(2)));
typedef __bf16 bf16x2_t __attribute__((ext_vector_type(2)));
typedef unsigned u32x4 __attribute__((ext_vector_type(4)));
typedef unsigned u32x8 __attribute__((ext_vector_type(8)));
typedef unsigned long long u64;

#define STAMP 0
#define NOCOMM 0
#define PER 160
#define SHIFT 40
#define XS 0x44
#define SMB 4
#define PFR 41
#define PFX 0
#define PFS 2
#define PFD 1
#define PFP 1
#define XE 0
#define NPK 4
#define LAG 1
#define LQ 0
#define OAUX 16
#define OMASK 0xff
#define KHM 1
#define TH 4

#define PHI 2
#define TV 5
#define TP 1
#define PL 3
#define NPR 1
#define FAKEX 0
#define PHIX 2
#define FLIP 0
#define RL 7

constexpr int NRANK = 8;
constexpr int NCOL = 8192;
constexpr int HTB = 16384;
constexpr int LDS_BYTES = 10 * HTB;
constexpr int LDS_CTRL = 9 * HTB + 8192;
constexpr size_t BLK_B = (size_t)1024 * 1024 * 2;
constexpr int N_POOL = 7;
constexpr int LDS_STG = 8 * HTB;
static_assert(PER <= 200 && PER / 2 - SHIFT <= 48 && PER - SHIFT <= 144 && 8 * PER - SHIFT <= 16 * 93, "a K-tile must be published before anyone can be waiting for it");
constexpr int LDS_ALT = 9 * HTB;
constexpr unsigned RROW = 128, RCOL = 32768;
constexpr size_t WS_CTR = 0;
constexpr size_t WS_TMO = 32768;
constexpr int TMO_VOTE = 256;
constexpr size_t WS_CLAIM = 49152;
constexpr size_t WS_XFLAG = 65536;
constexpr size_t WS_AFLAG = 131072;
constexpr size_t WS_ROLE = 196608;
constexpr size_t WS_A = (size_t)1 << 20;
constexpr size_t WS_XCH = (size_t)34 << 20;
constexpr size_t WS_DBG = (size_t)66 << 20;
constexpr size_t WS_NEED = (size_t)68 << 20;
constexpr unsigned DEADLINE = 50000000u;

template <int V> struct IC { static constexpr int value = V; };
struct Params {
  const char* x;
  const char* w;
  char* out;
  char* ws;
  const char* xo;
  const char* wo;
  unsigned rank;
  unsigned duty;
  char* peer[8];
};

__device__ __forceinline__ unsigned cvt_pk_bf16(float lo, float hi) { return __builtin_bit_cast(unsigned, __builtin_convertvector((f32x2){lo, hi}, bf16x2_t)); }
#define HV4(n) "v[" #n ":" #n "+3]"
#define HSET(n, x) asm volatile("v_mov_b32 v" #n ", %0" :: "v"(x) : "memory")
#define HLOAD4(n, vr, base, bits) asm volatile("global_load_dwordx4 " HV4(n) ", v" #vr ", %0 " bits :: "s"(base) : "memory")
#define HLOAD1(n, base, bits) asm volatile("global_load_dword v" #n ", v237, %0 " bits :: "s"(base) : "memory")
#define HSTORE4(n, vr, base) asm volatile("global_store_dwordx4 v" #vr ", " HV4(n) ", %0 sc0 sc1" :: "s"(base) : "memory")
#define HFIRST(n) ({ unsigned r_; asm volatile("v_readfirstlane_b32 %0, v" #n "\n\ts_nop 1" : "=s"(r_) :: "memory"); r_; })
#define NOW() ((unsigned)__builtin_amdgcn_s_memrealtime())
#define LANE() ({ int l_; asm volatile("v_mbcnt_lo_u32_b32 %0, -1, 0\n\tv_mbcnt_hi_u32_b32 %0, -1, %0" : "=v"(l_)); l_; })
__device__ __forceinline__ bool expired(unsigned t0, unsigned call, unsigned* tmo) {
  HLOAD1(234, tmo, "sc1"); asm volatile("s_waitcnt vmcnt(0)" ::: "memory");
  if (HFIRST(234) == call || NOW() - t0 > DEADLINE) {
    asm volatile("v_mov_b32 v234, %0\n\tglobal_store_dword v237, v234, %1 sc1" :: "s"(call), "s"(tmo) : "memory");
    return true;
  }
  __builtin_amdgcn_s_sleep(2);
  return false;
}
#define GLDS(l, LI, vr, base, GI, bits) asm volatile("s_add_u32 m0, %0, %1\n\ts_nop 0\n\tglobal_load_lds_dwordx4 v" #vr ", %2 offset:%3 " bits \
    :: "s"(l), "n"((LI) - (GI)), "s"(base), "n"(GI) : "memory", "scc")

#define DST(i, val) asm volatile("v_mov_b64 v[244:245], %0\n\tglobal_store_dwordx2 v237, v[244:245], %1 offset:%2" :: "s"((u64)(val)), "s"(p.ws + WS_DBG + (bid * 8 + wid) * 128), "n"((i) * 8) : "memory")
#define ST(i) do { if (STAMP) { const u64 r_ = __builtin_amdgcn_s_memrealtime(), c_ = __builtin_amdgcn_s_memtime(); DST(i, r_); DST(5 + (i), c_); } } while (0)

__global__ __launch_bounds__(512, 2) __attribute__((amdgpu_num_vgpr(116))) void a2a_gemm(Params) {
  asm volatile("v_mov_b32 v237, 0" ::: "v255", "memory");
  u64 st0 = STAMP ? __builtin_amdgcn_s_memrealtime() : 0;
  unsigned wid = __builtin_amdgcn_readfirstlane(threadIdx.x >> 6), bid = blockIdx.x;
  typedef const __attribute__((address_space(4))) Params* KP;
  KP kp = (KP)__builtin_amdgcn_kernarg_segment_ptr();
#define XCC() ({ unsigned x_; asm volatile("s_getreg_b32 %0, hwreg(HW_REG_XCC_ID, 0, 3)" : "=s"(x_)); x_; })
  unsigned vb = (bid & ~7u) | (FAKEX ? XCC() & (bid & 64 ? 5 : 7) : XCC());
  for (bool first = true;; first = false) {
  asm volatile("" : "+s"(kp), "+s"(wid), "+s"(bid));
  const unsigned pe = (vb & 4) + (vb >> 6), pg = ((vb >> 1) & 28) + (vb & 3);
  u32x8 ka; u32x4 kb; u64 kc, kd;
  asm volatile("s_load_dwordx4 %1, %4, 0x20\n\ts_load_dwordx8 %0, %4, 0\n\ts_load_dwordx2 %3, %4, 0x30\n\ts_load_dwordx2 %2, %4, %5"
               : "=&s"(ka), "=&s"(kb), "=&s"(kc), "=&s"(kd) : "s"(kp), "s"(0x38 + pe * 8));
  const unsigned wr = wid >> 2, wc = wid & 3;
#define TAKE(v, n) asm volatile("v_mov_b32 v254, %1\n\ts_mov_b64 exec, 1\n\tglobal_atomic_add v253, v237, v254, %0 sc0\n\ts_mov_b64 exec, -1" :: "s"(p.ws + WS_ROLE + (v) * 128), "n"(n) : "memory")
  const int lane = LANE();
  const unsigned h = (vb >> 2) & 1, cg = (vb & 3) ^ h, pm = (vb >> 3) & 3, pn = cg * 8 + (vb >> 5);
  const unsigned kh = KHM == 0 ? h : KHM == 1 ? h ^ (pm & 1) : KHM == 2 ? h ^ (pn & 1) : KHM == 3 ? h ^ ((pn >> 1) & 1) : h ^ (cg >> 1) ^ (KHM & 1);

  auto voA = [](int w, int ln) { const int row = w * 8 + (ln >> 3), ch = (ln & 7) ^ ((row >> 1) & 7); return (unsigned)(row * 2048 + ch * 16); };
  auto voB = [](int w, int ln) { const int row = w * 4 + (ln >> 4), c16 = ln & 15, f = (row & 3) | (((row >> 3) & 1) << 2);
                                 return (unsigned)(row * 16384 + (((c16 >> 1) ^ f) * 32) + (c16 & 1) * 16); };
#define SET_VOFF(ln) do { const unsigned a_ = voA(wid, ln), b_ = voB(wid, ln); HSET(238, a_); HSET(239, a_ + 64 * 2048); HSET(240, b_); HSET(241, b_ + 32 * 16384); } while (0)
  const unsigned ldsw = wid * 1024;
  const unsigned ldsX = (wr * 4 + 3) * 1024 + (wc == 1 ? 8192 : 0);

#define LP(off) ((LAS char*)(size_t)(unsigned)(off))
#define STAGE_A(l, LI, g) do { const unsigned l_ = ldsw + (l); const char* g_ = (g); GLDS(l_, (LI), 238, g_, 0, "sc0 sc1"); GLDS(l_, (LI) + 8192, 239, g_, 0, "sc0 sc1"); } while (0)
#define STAGE_B(l, LI, g, GI) do { const unsigned l_ = ldsw + (l); const char* g_ = (g); GLDS(l_, (LI), 240, g_, GI, ""); GLDS(l_, (LI) + 8192, 241, g_, GI, ""); } while (0)
#define STAGE_H(NX, l, hf, ga, gb) do { STAGE_B(l, 4 * HTB + (hf) * HTB, gb, (hf) * 256); \
    if (NX == 2) { const unsigned x_ = ldsX + (l); GLDS(x_, 4 * HTB + (hf) * HTB, 242, gb, (hf) * 256, ""); GLDS(x_, 4 * HTB + (hf) * HTB + 8192, 243, gb, (hf) * 256, ""); } \
    STAGE_A(l, (hf) * HTB, ga); \
    if (NX == 1) { const unsigned x_ = ldsX + (l); GLDS(x_, (hf) * HTB, 242, ga, 0, "sc0 sc1"); } } while (0)
#define LDA(dst, o0, o1, imm) do { _Pragma("unroll") for (int m = 0; m < 4; ++m) { dst[m][0] = *(const LAS bf16x8*)(LP(o0) + (imm) + m * 2048); dst[m][1] = *(const LAS bf16x8*)(LP(o1) + (imm) + m * 2048); } } while (0)
#define TRR(o, imm) __builtin_amdgcn_ds_read_tr16_b64_v4i16((LAS bf16x4*)(LP(o) + (imm)))
#define LDB(dst, o0, o1, imm) do { _Pragma("unroll") for (int k = 0; k < 2; ++k) { \
    { bf16x4 lo = TRR(o0, (imm) + k * 8192), hi = TRR(o0, (imm) + k * 8192 + 1024); dst[0][k] = __builtin_shufflevector(lo, hi, 0, 1, 2, 3, 4, 5, 6, 7); } \
    { bf16x4 lo = TRR(o1, (imm) + k * 8192), hi = TRR(o1, (imm) + k * 8192 + 1024); dst[1][k] = __builtin_shufflevector(lo, hi, 0, 1, 2, 3, 4, 5, 6, 7); } } } while (0)
#define SCHED __builtin_amdgcn_sched_barrier(0)
#define WAIT_V(n) asm volatile("s_waitcnt vmcnt(%0)" :: "n"(n) : "memory")
#define WAIT_L0 __builtin_amdgcn_s_waitcnt(0xC07F)
#define FENCE asm volatile("" ::: "memory")
#define BAR do { FENCE; __builtin_amdgcn_s_barrier(); FENCE; } while (0)
#define MM16(ai, bj, Ax, Bx) do { _Pragma("unroll") for (int i_ = 0; i_ < 16; ++i_) { const int m = i_ >> 2, n = (i_ >> 1) & 1, k = i_ & 1; \
    acc[ai][bj][m][n] = __builtin_amdgcn_mfma_f32_16x16x32_bf16(Bx[n][k], Ax[m][k], acc[ai][bj][m][n], 0, 0, 0); SCHED; } } while (0)
#define MM16H(ai, bj, Ax, Bx, H) do { _Pragma("unroll") for (int i_ = 0; i_ < 16; ++i_) { const int m = i_ >> 2, n = (i_ >> 1) & 1, k = i_ & 1; \
    acc[ai][bj][m][n] = __builtin_amdgcn_mfma_f32_16x16x32_bf16(Bx[n][k], Ax[m][k], acc[ai][bj][m][n], 0, 0, 0); SCHED; H; SCHED; } } while (0)
#define PRIO(n) __builtin_amdgcn_s_setprio(n)
#define XCV(n, v) asm volatile("v_cvt_pk_bf16_f32 v[%4], %0, %1\n\tv_cvt_pk_bf16_f32 v[%4+1], %2, %3" :: "v"(v[0]), "v"(v[1]), "v"(v[2]), "v"(v[3]), "n"(n) : "memory")
#define XST(i) do { XCV(244 + ((i) & 1) * 8, acc[0][(i) >> 2][(i) & 3][0]); XCV(246 + ((i) & 1) * 8, acc[0][(i) >> 2][(i) & 3][1]); \
    asm volatile("global_store_dwordx4 v248, v[%1:%1+3], %0 offset:%2 sc0 sc1" :: "s"(xdst), "n"(244 + ((i) & 1) * 8), "n"((i) * 1024 - 4096) : "memory"); } while (0)

  const unsigned sendoff = (1 - kh) * (128 * 2048), keepoff = kh * (128 * 2048);
  unsigned oa = pm * (256 * 2048) + h * 1024 + sendoff, ob = pn * 512 + h * (512 * 16384), oc = bid * 128 + wid * 4;
  if (wc != 3) {
    SET_VOFF(lane);
    if (wc == 2) { const unsigned b_ = voB(wr * 4 + 3, lane); HSET(242, b_); HSET(243, b_ + 32 * 16384); }
    else HSET(242, voA(wr * 4 + 3, lane) + wc * (64 * 2048));
  } else {
    const unsigned a_ = voA(wid, lane), o_ = ((pg & 15) * 64 + (lane >> 3)) * 2048 + (pg >> 4) * 256 + (lane & 7) * 16;
    HSET(238, (a_ >> 11) * RROW + (a_ & 127)); HSET(239, voB(wid, lane)); HSET(234, o_); HSET(249, o_ + 16384);
  }
  asm volatile("s_waitcnt lgkmcnt(0)" : "+s"(ka), "+s"(kb), "+s"(kc), "+s"(kd), "+s"(oa), "+s"(ob), "+s"(oc));
  const unsigned rank = (unsigned)kd, pj = ((unsigned)(kd >> 32) >> (pe * 4)) & 7;
#define PTR(lo, hi) ((char*)(((u64)(hi) << 32) | (lo)))
  const struct { const char *x, *w; char *out, *ws, *peer; } p = {PTR(ka[0], ka[1]), PTR(ka[2], ka[3]), PTR(ka[4], ka[5]), PTR(ka[6], ka[7]), (char*)kc};
  unsigned* ctr = (unsigned*)(p.ws + WS_CTR + oc);
  u64 sta = STAMP == 3 ? __builtin_amdgcn_s_memrealtime() : 0;
  const char* a0 = PTR(kb[0], kb[1]) + oa;
  const char* b0 = PTR(kb[2], kb[3]) + ob;
  const char* xs = p.x + ((rank + 1 + pj) & 7) * (unsigned)BLK_B;
  HLOAD1(232, ctr, "sc1");
  if (wc < 2) STAGE_H(1, 0, 0, a0, b0); else if (wc == 2) STAGE_H(2, 0, 0, a0, b0); else if (wr == 0) { if (first) TAKE(vb, 1); }
  else {
#pragma unroll
    for (int i = 0; i < 4; ++i) { const char* g_ = xs + i * 32768; GLDS(LDS_STG, i * 2048, 234, g_, 0, ""); GLDS(LDS_STG, i * 2048 + 1024, 249, g_, 0, ""); }
  }
  SCHED;
  unsigned* tmo = (unsigned*)(p.ws + WS_TMO);
  const char* aOwn = PTR(kb[0], kb[1]) + pm * (256 * 2048);
  const char* wB = p.w + pn * 512;
  const char* a1 = aOwn + (h * 8 + 1) * 128 + keepoff;
  const char* b1 = b0 + 64 * 16384;
  const bool pusher = pj < 7 && !NOCOMM;
  char* pd = p.peer + WS_A + (6 - (int)pj) * (long)BLK_B;
  char* pf = p.peer + WS_AFLAG + (((pg >> 2) & 3) * 8 + 6 - (int)pj) * 128 + (pg >> 4) * 16 + (pg & 3) * 4;
  u64 stb = STAMP == 3 ? __builtin_amdgcn_s_memrealtime() : 0;
#define RWO(q) ((((rank + 1 + (q)) & 7) * 1024 + (15 - 2 * ((q) & 1)) * 64) * 16384u)
#define RAO(q) ((q) * (unsigned)BLK_B + (15 - ((q) & 1)) * RCOL + kh * (128 * RROW))
  unsigned rwo = RWO(2);
  const char* rw0 = wB + RWO(1); const char* rw1 = wB + rwo;
  char* xdst = p.ws + WS_XCH + (size_t)(vb ^ 5) * 131072 + wid * 16384 + (PHIX == 2 ? wid & 1 : (h ^ PHIX ^ (vb >> 4)) & 1) * 8192 + 4096;
  const char* rab = p.ws + WS_A + (size_t)pm * (256 * 2048) + RAO(3);
  const char* xflag = p.ws + WS_XFLAG + vb * 128 + wid * 4;
  char* xpub = p.ws + WS_XFLAG + (vb ^ 5) * 128 + wid * 4;
  char* xsb = p.ws + WS_XCH + (size_t)vb * 131072 + wid * 16384 + (PHIX == 2 ? wid & 1 : (h ^ 1 ^ PHIX ^ (vb >> 4)) & 1) * 8192;
  asm volatile("" : "+s"(rw0), "+s"(rw1), "+s"(rwo), "+s"(xdst), "+s"(rab), "+s"(xflag), "+s"(xpub), "+s"(xsb));
  SCHED;
  f32x4 acc[2][2][4][2];
  bf16x8 A0[4][2], A1[4][2], B0[2][2], B1[2][2];
  unsigned aoff0, aoff1, boff0, boff1, call;
  const char* aRem;
  int t = 0; unsigned par = 0, more1 = 1;
  auto sloop = [&](auto role_) __attribute__((always_inline)) {
    constexpr int ROLE = decltype(role_)::value;
    const int lane = LANE();
#pragma unroll
    for (int i = 0; i < 32; ++i) { acc[i >> 4][(i >> 3) & 1][(i >> 1) & 3][i & 1] = (f32x4){0.f, 0.f, 0.f, 0.f}; asm volatile("" : "+v"(acc[i >> 4][(i >> 3) & 1][(i >> 1) & 3][i & 1])); }
#pragma unroll
    for (int i = 0; i < 8; ++i) { A1[i >> 1][i & 1] = (bf16x8){0, 0, 0, 0, 0, 0, 0, 0}; asm volatile("" : "+v"(A1[i >> 1][i & 1])); }
#pragma unroll
    for (int i = 0; i < 4; ++i) { B1[i >> 1][i & 1] = (bf16x8){0, 0, 0, 0, 0, 0, 0, 0}; asm volatile("" : "+v"(B1[i >> 1][i & 1])); }
    { const int fr = lane & 15, fq = lane >> 4;
      aoff0 = (wr * 64 + fr) * 128 + ((fq ^ (fr >> 1)) * 16); aoff1 = aoff0 ^ 64;
      const int hh = lane >> 5, g1 = (lane >> 4) & 1, q = (lane >> 2) & 3, pp = lane & 3, ff = q | (g1 << 2);
      boff0 = 4 * HTB + (16 * hh + 8 * g1 + q) * 256 + (((2 * wc) ^ ff) * 32) + pp * 8; boff1 = boff0 ^ 32;
      asm volatile("" : "+v"(aoff0), "+v"(aoff1), "+v"(boff0), "+v"(boff1)); }
    asm volatile("v_mov_b32 v236, %0" :: "n"(LDS_CTRL) : "memory");
    unsigned thr = 0;
    if (ROLE == 3) {
      const unsigned k = (lane * 37) >> 8, o = lane - k * 7;
      const bool r0 = lane == 6, r1 = lane == 63;
      const unsigned q = h ^ (cg >> 1) ^ (PHI == 2 ? 0 : PHI), i = q ? 6 - o : o, s = 2 * i - (i < 4 ? 0 : 7), k1 = k - 1;
      const bool own = k == 0, ded = k1 < 7;
      const unsigned pi = ded ? q : (s & 1) ^ (own ? 1 : 0), j = own ? h * 8 + 2 + o : (k1 >> 1) * 4 + pi * 2 + (k1 & 1), sl = 2 * k1 + pi;
      const unsigned e = own ? j * 128 : s * (unsigned)BLK_B + sl * RCOL, hb = own ? 128 * 2048 : 128 * RROW;
      unsigned e0 = e + (kh ? 0 : hb), e1 = e + (kh ? hb : 0), e2 = (((rank + (own ? 0 : 1 + s)) & 7) * 1024 + j * 64) * 16384u, e3 = s * 128 + pi * 16;
      asm volatile("" : "+v"(e0), "+v"(e1), "+v"(e2));
      e0 = r0 ? RAO(0) : e0; e0 = r1 ? RAO(2) : e0; e1 = r0 ? RAO(1) : e1; e2 = r0 ? RWO(0) : e2; e2 = r1 ? RWO(1) : e2;
      if (PHI == 2) {
        const unsigned x = ded ? ~0u : 0u;
        *(LAS u32x4*)LP(LDS_ALT + (k1 < 8 ? lane + 6 - 2 * o : lane) * 16) = (u32x4){e0 ^ (x & RCOL), e1 ^ (x & RCOL), e2 ^ (x & 0x200000u), e3 ^ (x & 16)};
        HSET(246, LDS_ALT + lane * 16);
      }
      HSET(248, e0); HSET(249, e1); HSET(250, e2); HSET(251, (r0 || r1) ? 0u : 1u); HSET(252, e3);
      HSET(247, (lane & 31) * 4);
      thr = (k == 0 || k == 8) ? 8 : k;
      const unsigned f = NOCOMM ? 0x7fffffffu : 0u;
      HSET(240, f); HSET(241, f); HSET(242, f); HSET(243, f);
    }
    if (ROLE <= 2) { STAGE_H(ROLE, 0, 1, a1 - 128, b0); if (wr == 0) STAGE_H(ROLE, 32768, 0, a0 + 128, b1); }
    if (ROLE == 4) {
      HSET(250, ((pg >> 2) & 3) * (256 * 2048) + (pg >> 4) * RCOL + ((pg & 3) * 64 + (lane >> 3)) * RROW + (lane & 7) * 16);
      if (PHI == 2 && pj == 7) {
        const unsigned g_ = (((rank + 4) & 7) * 1024 + 512) * 16384 - pn * 512;
        HSET(250, g_ + pg * 256 + (lane & PL) * 16384); HSET(251, g_ + pg * 256 + (lane & PL) * 16384 + 8192); HSET(241, g_ + (pg ^ 1) * 256 + (lane & 31) * 16384 + (lane >> 5) * 8192);
        HSET(234, pg * 4);
        asm volatile("global_load_dword v240, v241, %0" :: "s"(wB) : "memory");
      }
      if (PFR) HSET(252, PFP ? (64 + pm * 16 + (lane >> 2) + PFX * 128) * 16384 + (lane & 3) * 128 : (64 + lane + PFX * 128) * 16384 + pm * 128);
    }
    int pstart = (int)((((((pg >> 4) ^ pj) & 1) * 112 + (pg & 15) * 7 + pj) * PER) / 224) - SHIFT, pnext = pusher ? pstart : 0x7fffffff, pforce = max(pstart, 0) + 48, pleft = 8, pth = TH;
    unsigned pls = LDS_STG, pcs = 0xfffe0180u;
    int tp = pj == 7 ? TP : -1, tf = PFR;
    unsigned fo = (((rank + 1) & 7) * 1024 + 12 * 64) * 16384u;
    u64 fm = ~0ull >> (64 - 64 / PFD);
    const char* xn = xs + 128;
    if (ROLE == 4) { asm volatile("" : "+s"(pstart), "+s"(pnext), "+s"(pforce), "+s"(xn), "+s"(tp)); HSET(248, lane * 16); BAR; }
    const u64 stc = STAMP == 3 ? __builtin_amdgcn_s_memrealtime() : 0;
    if (ROLE > 2) WAIT_V(0); else if (wr == 0) WAIT_V(8 + 2 * ROLE); else WAIT_V(4 + ROLE);
    const u64 std_ = STAMP == 3 ? __builtin_amdgcn_s_memrealtime() : 0;
    call = HFIRST(232) + 1u;
    aRem = p.ws + WS_A + (size_t)(call & 1u) * (8 * BLK_B) + (size_t)pm * (256 * 2048);

    if (ROLE == 3 && first && __builtin_expect(HFIRST(253) != call - 1u, 0)) {
      unsigned v = vb;
      do { TAKE(v, -1); v = (v + 1) & 255; TAKE(v, 1); WAIT_V(0); } while (HFIRST(253) != call - 1u);
      HSET(251, 2 * v + 2);
    }
    char* pd0 = pd + (size_t)(call & 1u) * (8 * BLK_B);
    if (ROLE == 4) { HSET(233, (call - 1u) * 8 + 1); HSET(232, LDS_STG + lane * 16); asm volatile("" : "+s"(pd0)); }
    if (ROLE == 3) HSET(244, lane == 63 ? 0u : (call - 1u) * 8 + thr);
    u64 T = 0x00ffffffffffffbfull;
    unsigned cl = 56;
    const unsigned tk0 = (call - 1u) * (N_POOL + 2);
    const char* fB = p.ws + WS_AFLAG + pm * 1024;
    const char* claim = p.ws + WS_CLAIM + ((vb >> 3) * 4 + cg) * 128;
    if (ROLE <= 2) asm volatile("" : "+s"(aRem));
    u64 tslow = 0, pubs = 0; unsigned nslow = 0;
#define PUSH_STEP(now) do { unsigned tmp_; const int l_ = pleft; asm volatile( \
      "s_cmp_lt_i32 %[pt], %[nx]\n\ts_cbranch_scc1 9f\n\t" \
      "s_cmp_ge_i32 %[pt], %[fo]\n\ts_cbranch_scc1 2f\n\t" \
      "s_getreg_b32 %[tm], hwreg(HW_REG_IB_STS, 0, 4)\n\ts_cmp_gt_u32 %[tm], %[th]\n\ts_cbranch_scc1 9f\n" \
      "2:\n\ts_bitcmp1_b32 %[ls], 13\n\ts_cbranch_scc1 5f\n\t" \
      "s_mov_b32 m0, %[ls]\n\ts_add_i32 %[fo], %[fo], 16\n\tglobal_load_lds_dwordx4 v234, %[xn]\n\t" \
      "s_add_u32 m0, %[ls], 1024\n\ts_add_u32 %[ls], %[ls], 2048\n\tglobal_load_lds_dwordx4 v249, %[xn]\n\t" \
      "global_store_dwordx4 v250, v[240:243], %[d0] sc0 sc1\n\tglobal_store_dwordx4 v250, v[244:247], %[d0] offset:1024 sc0 sc1\n\t" \
      "v_add_u32 v234, 0x8000, v234\n\tv_add_u32 v249, 0x8000, v249\n\tv_add_u32 v250, %[ds], v250\n\tv_add_u32 v232, %[ls], v248\n\t" \
      "s_bitcmp1_b32 %[ls], 13\n\ts_cbranch_scc0 9f\n\ts_add_i32 %[fo], %[fo], 16\n\ts_mov_b32 %[th], 0\n\ts_branch 9f\n" \
      "5:\n\ts_waitcnt vmcnt(0)\n\ts_mov_b64 exec, 1\n\tglobal_store_dword v237, v233, %[pf] sc0 sc1\n\ts_mov_b64 exec, -1\n\t" \
      "s_mov_b32 %[ls], %[stg]\n\tv_add_u32 v234, %[cs], v234\n\tv_add_u32 v249, %[cs], v249\n\tv_add_u32 v250, %[de], v250\n\ts_xor_b32 %[cs], %[cs], 0x100\n\t" \
      "v_add_u32 v233, 1, v233\n\tv_add_u32 v232, %[ls], v248\n\ts_mov_b32 %[th], %[th0]\n\t" \
      "s_add_i32 %[ps], %[ps], %[per]\n\ts_add_i32 %[fo], %[ps], 48\n\ts_sub_u32 %[lf], %[lf], 1\n\t" \
      "s_cmp_lg_u32 %[lf], 0\n\ts_cselect_b32 %[nx], %[ps], 0x7fffffff\n\ts_cmp_lg_u32 %[lf], 1\n\ts_cselect_b64 %[xn], %[xn], %[xs]\n" \
      "9:" \
      : [nx] "+s"(pnext), [fo] "+s"(pforce), [lf] "+s"(pleft), [ps] "+s"(pstart), [ls] "+s"(pls), [xn] "+s"(xn), [th] "+s"(pth), [cs] "+s"(pcs), [tm] "=&s"(tmp_) \
      : [pt] "s"(now), [d0] "s"(pd0), [pf] "s"(pf), [xs] "s"(xs - 384), [per] "n"(PER), [stg] "n"(LDS_STG), [th0] "n"(TH), [ds] "n"(16 * RROW), [de] "n"((int)(2 * RCOL - 64 * RROW)) : "memory", "scc"); \
      if (STAMP == 1 && l_ != pleft) pubs |= (u64)((((now) - pstart + PER) >> 4) & 255) << ((7 - pleft) * 8); } while (0)
#define PUSH_READ asm volatile("ds_read_b128 v[240:243], v232\n\tds_read_b128 v[244:247], v232 offset:1024" ::: "memory")
#define CHOOSE() ({ u64 c_, K_; unsigned b_; asm volatile( \
      "s_waitcnt vmcnt(%[pf])\n\tv_min3_u32 v254, v240, v241, v242\n\tv_min_u32 v254, v254, v243\n\tv_cmp_le_u32 %[K], v244, v254\n\t" \
      "s_cmp_eq_u64 %[T], 0\n\ts_cbranch_scc1 7f\n\t" \
      "s_or_b64 %[c], %[K], 63\n\ts_and_b64 %[c], %[c], %[T]\n\ts_cbranch_scc0 8f\n\t" \
      "s_ff1_i32_b64 %[b], %[c]\n\ts_lshl_b64 exec, 1, %[b]\n\tds_write_b128 v236, v[248:251]\n\ts_bitset0_b64 %[T], %[b]\n\t" \
      "s_cmp_eq_u64 %[T], 0\n\ts_cbranch_scc0 8f\n\ts_mov_b64 exec, 1\n\ts_branch 6f\n" \
      "7:\n\tv_readfirstlane_b32 %[b], v253\n\ts_or_b64 %[c], %[K], 0x7f\n\ts_sub_u32 %[b], %[b], %[tk]\n\ts_cmp_lt_u32 %[b], %[np]\n\ts_cselect_b32 %[b], %[cl], 6\n\ts_cselect_b64 %[c], %[c], %[K]\n\t" \
      "s_cmp_eq_u64 %[c], -1\n\ts_cselect_b64 %[c], 1, 0\n\ts_cbranch_scc0 8f\n\t" \
      "s_cmp_eq_u32 %[cl], 0\n\ts_cselect_b32 %[b], 63, %[b]\n\ts_lshl_b64 exec, 1, %[b]\n\tds_write_b128 v236, v[248:251]\n\t" \
      "s_cmp_eq_u32 %[b], %[cl]\n\ts_cselect_b64 exec, 1, 0\n\ts_cselect_b32 %[cl], %[cl], -1\n\ts_add_u32 %[cl], %[cl], 1\n" \
      "6:\n\tglobal_atomic_add v253, v237, v251, %[cm] sc0\n" \
      "8:\n\ts_not_b64 exec, %[K]\n\tglobal_load_dwordx4 v[240:243], v252, %[fB] sc0 sc1\n\ts_mov_b64 exec, -1" \
      : [K] "=&s"(K_), [c] "=&s"(c_), [b] "=&s"(b_), [T] "+s"(T), [cl] "+s"(cl) \
      : [fB] "s"(fB), [cm] "s"(claim), [tk] "s"(tk0), [np] "n"(N_POOL), [pf] "n"(STAMP == 2 ? 1 : 0) : "memory", "scc"); c_; })
#define LB4(dst, o0, o1, imm, k) do { \
    { bf16x4 lo = TRR(o0, (imm) + k * 8192), hi = TRR(o0, (imm) + k * 8192 + 1024); dst[0][k] = __builtin_shufflevector(lo, hi, 0, 1, 2, 3, 4, 5, 6, 7); } \
    { bf16x4 lo = TRR(o1, (imm) + k * 8192), hi = TRR(o1, (imm) + k * 8192 + 1024); dst[1][k] = __builtin_shufflevector(lo, hi, 0, 1, 2, 3, 4, 5, 6, 7); } } while (0)
#define LA4(dst, imm, m0_) do { _Pragma("unroll") for (int m = m0_; m < m0_ + 2; ++m) { dst[m][0] = *(const LAS bf16x8*)(LP(aoff0) + (imm) + m * 2048); dst[m][1] = *(const LAS bf16x8*)(LP(aoff1) + (imm) + m * 2048); } } while (0)
#define READ_STAGE(Ax, Bx, rh, l, hf, ga, gb) do { const unsigned l_ = ldsw + (l), x_ = ldsX + (l); const char* ga_ = (ga); \
      LB4(Bx, boff0, boff1, (rh) * HTB, 0); SCHED; GLDS(l_, (4 + (hf)) * HTB, 240, gb, (hf) * 256, ""); SCHED; \
      LB4(Bx, boff0, boff1, (rh) * HTB, 1); SCHED; GLDS(l_, (4 + (hf)) * HTB + 8192, 241, gb, (hf) * 256, ""); SCHED; \
      LA4(Ax, (rh) * HTB, 0); SCHED; GLDS(l_, (hf) * HTB, 238, ga_, 0, "sc0 sc1"); SCHED; \
      LA4(Ax, (rh) * HTB, 2); SCHED; GLDS(l_, (hf) * HTB + 8192, 239, ga_, 0, "sc0 sc1"); \
      if (ROLE == 2) { GLDS(x_, (4 + (hf)) * HTB, 242, gb, (hf) * 256, ""); GLDS(x_, (4 + (hf)) * HTB + 8192, 243, gb, (hf) * 256, ""); } \
      else GLDS(x_, (hf) * HTB, 242, ga_, 0, "sc0 sc1"); \
      WAIT_V(8 + 2 * ROLE); } while (0)

    if (ROLE == 4) PUSH_READ;
    BAR;
    if (ROLE <= 2 && wr == 1) { STAGE_H(ROLE, 32768, 0, a0 + 128, b1); BAR; }
    const u64 st1r = STAMP ? __builtin_amdgcn_s_memrealtime() : 0, st1c = STAMP ? __builtin_amdgcn_s_memtime() : 0;

    for (;;) {
      if (ROLE == 4) { WAIT_L0; FENCE; SCHED; }
      if (ROLE <= 2) READ_STAGE(A0, B0, 0, par ^ 32768u, 1, a1, b1);
      else {
        LDB(B0, boff0, boff1, 0); SCHED;
        if (ROLE == 4) PUSH_STEP(32 * t);
        else if (__builtin_expect(CHOOSE() == 0ull, 0)) {
          const unsigned t0 = NOW();
          do {
            if (expired(t0, call, tmo)) HSET(244, 0u);
          } while (CHOOSE() == 0ull);
          if (STAMP) { nslow += T ? 1 : 256; tslow += NOW() - t0; }
        }
        SCHED; LDA(A0, aoff0, aoff1, 0);
      }
      FENCE; WAIT_L0; SCHED; BAR;
      SCHED;
#define PUSH_READ1 do { if (ROLE == 4) { if (i_ == 2) asm volatile("ds_read_b128 v[240:243], v232" ::: "memory"); if (i_ == 4) asm volatile("ds_read_b128 v[244:247], v232 offset:1024" ::: "memory"); } } while (0)
#define ENTRY_READ do { if (i_ == 0) { if (ROLE <= 2) asm volatile("ds_read_b128 v[232:235], v236" ::: "memory"); else asm volatile("ds_read_b32 v235, v236 offset:12" ::: "memory"); } PUSH_READ1; } while (0)
      PRIO(1); MM16(1, 1, A1, B1); MM16H(0, 0, A0, B0, ENTRY_READ); PRIO(0); BAR; SCHED;
      unsigned e0, e1, e2, more2;
      if (ROLE <= 2) asm volatile("s_waitcnt lgkmcnt(0)\n\tv_readfirstlane_b32 %0, v232\n\tv_readfirstlane_b32 %1, v233\n\tv_readfirstlane_b32 %2, v234\n\tv_readfirstlane_b32 %3, v235"
                                  : "=s"(e0), "=s"(e1), "=s"(e2), "=s"(more2) :: "memory");
      else asm volatile("s_waitcnt lgkmcnt(0)\n\tv_readfirstlane_b32 %0, v235" : "=s"(more2) :: "memory");
      SCHED;
      if (ROLE <= 2) {
        if (__builtin_expect(t == 6, 0)) {
          asm volatile("v_lshrrev_b32 v244, 4, v238\n\tv_bfi_b32 v238, %0, v238, v244\n\tv_add_u32 v239, 0x2000, v238" :: "s"(127) : "memory");
          if (ROLE == 1) asm volatile("v_lshrrev_b32 v244, 4, v242\n\tv_bfi_b32 v242, %0, v242, v244" :: "s"(127) : "memory");
        }
        const char* ab = t < 6 ? aOwn : aRem;
        b1 = wB + e2; a1 = ab + e1;
        READ_STAGE(A1, B1, 1, par, 0, ab + e0, b1);
      } else {
        LDB(B1, boff0, boff1, HTB); SCHED;
        if (ROLE == 4) {
          PUSH_STEP(32 * t + 16);
          if (PHI == 2 && __builtin_expect(t == tp, 0) && more2 == 1) {
            int d_ = 0, n_ = 2 * NPR; u64 m_ = 0;
            asm volatile("" : "+s"(n_));
#pragma clang loop unroll(disable)
            for (; n_ >= 0; --n_) {
              u64 t0_, t1_;
              asm volatile("s_mov_b64 exec, %3\n\tglobal_load_dword v240, v250, %2\n\tglobal_load_dword v241, v251, %2\n\ts_waitcnt vmcnt(1)\n\ts_memtime %0\n\ts_waitcnt vmcnt(0)\n\ts_memtime %1\n\t"
                           "v_swap_b32 v250, v251\n\tv_add_u32 v250, %4, v250\n\tv_add_u32 v251, %4, v251\n\ts_mov_b64 exec, -1\n\ts_waitcnt lgkmcnt(0)"
                           : "=&s"(t0_), "=&s"(t1_) : "s"(wB), "s"(m_), "n"((PL + 1) * 16384) : "memory");
              d_ = (min((int)((unsigned)t1_ - (unsigned)t0_), 400) & (int)m_) - d_; m_ = ~0ull;
            }
            const unsigned v_ = 2 * call + ((XCC() >> 2) ^ (d_ < 0 ? 0 : 1));
            asm volatile("v_mov_b32 v233, %0\n\tglobal_store_dword v234, v233, %1 offset:%2 sc1" :: "s"(v_), "s"(tmo), "n"(TMO_VOTE) : "memory");
          }
          if (PFR && __builtin_expect(t == tf, 0)) {
            if (t == PFR) asm volatile("ds_read_b32 v255, v236 offset:8\n\ts_waitcnt lgkmcnt(0)\n\tv_and_b32 v255, 0x200000, v255\n\tv_xor_b32 v252, v252, v255" ::: "memory");
            asm volatile("s_mov_b64 exec, %1\n\tglobal_load_dword v255, v252, %0\n\ts_mov_b64 exec, -1" :: "s"(wB + fo), "s"(fm) : "memory");
            tf += PFS;
            if (PFD > 1) { fm <<= 64 / PFD; if (fm == 0) { fm = ~0ull >> (64 - 64 / PFD); fo = (fo + 0x1000000u) & 0x7ffffffu; } }
            else fo = (fo + 0x1000000u) & 0x7ffffffu;
            if (tf == PFR + 7 * PFD * PFS) tf = -1;
          }
        }
        if (ROLE == 3 && PHI == 2 && __builtin_expect((unsigned)(t - (TV - 1)) < 2u, 0)) {
#define VOTES asm volatile("global_load_dword v245, v247, %0 offset:%1 sc1" :: "s"(tmo), "n"(TMO_VOTE) : "memory")
          if (t == TV) {
            u64 m_; unsigned n_;
#define COUNT asm volatile("s_waitcnt vmcnt(0)\n\tv_cmp_le_u32 %0, %1, v245" : "=s"(m_) : "s"(2 * call) : "memory")
            COUNT;
            if (m_ != ~0ull) { const unsigned t0 = NOW(); do { if (expired(t0, call, tmo)) break; VOTES; COUNT; } while (m_ != ~0ull); if (STAMP) nslow += 65536; }
            asm volatile("v_cmp_lt_u32 %0, %2, v245\n\ts_bcnt1_i32_b64 %1, %0" : "=&s"(m_), "=s"(n_) : "s"(2 * call) : "memory", "scc");
            if (STAMP) pubs = n_;
            if (n_ > 32) { asm volatile("ds_read_b32 v252, v246 offset:12\n\tds_read_b96 v[248:250], v246\n\ts_waitcnt lgkmcnt(1)" ::: "memory");
              if (!NOCOMM) asm volatile("global_load_dwordx4 v[240:243], v252, %0 sc0 sc1" :: "s"(fB) : "memory"); }
          } else VOTES;
        }
        if (ROLE == 3 && STAMP == 2) asm volatile("v_mov_b32 v255, %0\n\tglobal_store_dword v237, v255, %1" :: "s"((unsigned)__builtin_amdgcn_s_memtime()), "s"(p.ws + WS_DBG + (1 << 20) + bid * 512 + t * 4) : "memory");
        SCHED; LDA(A1, aoff0, aoff1, HTB);
      }
      aoff0 ^= 32768; aoff1 ^= 32768; boff0 ^= 32768; boff1 ^= 32768; par ^= 32768u;
      asm volatile("" : "+v"(aoff0), "+v"(aoff1), "+v"(boff0), "+v"(boff1));
      SCHED;
      if (XE && __builtin_expect(more1 == 0, 0)) {
        if (ROLE != 4) asm volatile("v_mbcnt_lo_u32_b32 v248, -1, 0\n\tv_mbcnt_hi_u32_b32 v248, -1, v248\n\tv_lshlrev_b32 v248, 4, v248" ::: "memory");
        if (ROLE >= 3) WAIT_V(0);
        if (XE > 0) XST(0); if (XE > 1) XST(1); if (XE > 2) XST(2); if (XE > 3) XST(3);
      }
      FENCE; WAIT_L0; SCHED; BAR; SCHED;
      PRIO(1); MM16(0, 1, A0, B1); MM16H(1, 0, A1, B0, PUSH_READ1); PRIO(0); BAR; SCHED;
      ++t;
      if (more1 != 1) break;
      more1 = more2;
    }
    if (!XE && ROLE >= 3) WAIT_V(0);
    if (ROLE >= 3) asm volatile("s_waitcnt lgkmcnt(0)\n\tv_mov_b32 v240, v239\n\tv_add_u32 v241, 0x80000, v239\n\tv_add_u32 v239, %0, v238" :: "n"(64 * RROW) : "memory");
    if (STAMP == 2) asm volatile("v_writelane_b32 v235, %0, 40" :: "s"(t));
    if (STAMP && STAMP != 2) { ST(2); DST(1, st1r); DST(6, st1c); DST(10, nslow); DST(11, tslow); DST(12, t); DST(13, pubs); if (STAMP == 3) { const u64 q0_ = ((sta - st0) & 0xffff) | (((stb - st0) & 0xffff) << 16) | (((stc - st0) & 0xffff) << 32) | (((std_ - st0) & 0xffff) << 48), q1_ = st1r - st0; DST(10, q0_); DST(11, q1_); } }
  };
  if (wc < 2) sloop(IC<1>{}); else if (wc == 2) sloop(IC<2>{}); else if (wr == 0) sloop(IC<3>{}); else sloop(IC<4>{});
  par = __builtin_amdgcn_readfirstlane(par); call = __builtin_amdgcn_readfirstlane(call); more1 = __builtin_amdgcn_readfirstlane(more1);
  if (__builtin_expect(more1 != 0, 0)) { vb = (more1 >> 1) - 1; if (wr == 0) BAR; WAIT_V(0); WAIT_L0; BAR; continue; }

  bool bad = false, got = true;
  u64 rtp = 0;
  u32x4 pr[8];
  if (!XE) asm volatile("v_mbcnt_lo_u32_b32 v248, -1, 0\n\tv_mbcnt_hi_u32_b32 v248, -1, v248\n\tv_lshlrev_b32 v248, 4, v248" ::: "memory");
  const __amdgpu_buffer_rsrc_t xsrc = __builtin_amdgcn_make_buffer_rsrc(xsb, 0, 8192, 0x00020000);
  unsigned vl16 = 0, orow = 0, tw0 = 0, tw1 = 0, trd = 0;
#define PRLD(i) pr[i] = __builtin_amdgcn_raw_buffer_load_b128(xsrc, vl16 + ((i) & 3) * 1024, ((i) >> 2) * 4096, 17)
#define PRL(i0) do { asm volatile("" : "+v"(vl16)); PRLD(i0); SCHED; PRLD(i0 + 1); SCHED; PRLD(i0 + 2); SCHED; PRLD(i0 + 3); SCHED; } while (0)
  {
#define NS(r) ((r) < 0 ? XE : (XS >> (4 * ((r) & 7))) & 15)
#define X0(r) (XE + ((r) > 0 ? NS(0) : 0) + ((r) > 1 ? NS(1) : 0) + ((r) > 2 ? NS(2) : 0) + ((r) > 3 ? NS(3) : 0))
#define PUBB(r) ((r) > 1 && X0((r) - 1) >= 8 && X0((r) - 2) < 8)
#define NG(r) (((r) + 1 < RL ? 2 : 0) + ((r) + 2 < RL ? ((r) ? 4 : 2) : 0))
#define NL(r) ((r) == SMB + 1 || (r) == SMB + 2 ? 4 : 0)
    static_assert(X0(4) == 8 && SMB + 2 == RL - 1 && NS(SMB) == 0 && !PUBB(SMB), "");
    const unsigned lX = ldsw + par, lY = lX ^ 32768u;
    const unsigned dYZ = (4 * HTB + (par ^ 32768u)) ^ (8 * HTB);
    const char* ra = nullptr;
    unsigned bn0 = boff0, bn1 = boff1;
#define XL(r, q) do { if (q < NS(r)) XST((X0(r) + q) & 7); } while (0)
#define G0(r, q) do { SCHED; if (r + 1 < RL) GLDS(r % 3 == 0 ? lY : r % 3 == 1 ? ldsw : lX, (r % 3 == 1 ? 9 : 5) * HTB + (q) * 8192, 24##q, rw0, 256, ""); SCHED; } while (0)
#define GA(r, q, v) do { SCHED; if (r + 2 < RL) GLDS(r % 3 == 2 ? lX : lY, (r % 3 == 0 ? 0 : HTB) + (q) * 8192, v, ra, 0, "sc0 sc1"); SCHED; } while (0)
#define G1(r, q) do { SCHED; if (r + 2 < RL) GLDS(r % 3 == 0 ? ldsw : r % 3 == 1 ? lX : lY, (r % 3 == 0 ? 8 : 4) * HTB + (q) * 8192, 24##q, rw1, 0, ""); SCHED; } while (0)
#define RNEXT(r) do { asm volatile("" : "+s"(rwo), "+v"(aoff0), "+v"(aoff1), "+v"(boff0), "+v"(boff1)); \
      rw0 = rw1; \
      if (r + 3 < RL) { rwo = (rwo + (r & 1 ? 0x1200000u : 0xe00000u)) & 0x7ffffffu; rw1 = wB + rwo; \
        if (r == 0) { asm volatile("" : "+s"(call)); ra = rab + (size_t)(call & 1u) * (8 * BLK_B); } \
        else { asm volatile("" : "+s"(ra)); ra += r & 1 ? BLK_B + RCOL : BLK_B - RCOL; } \
        asm volatile("" : "+s"(ra), "+s"(rw1)); } \
      if (r % 3 == 1 || (r % 3 == 0 && r > 0)) { aoff0 ^= 32768; aoff1 ^= 32768; } \
      const unsigned d_ = r % 3 == 0 ? 32768u : r % 3 == 1 ? dYZ : dYZ ^ 32768u; \
      bn0 = boff0 ^ d_; bn1 = boff1 ^ d_; \
      asm volatile("" : "+v"(aoff0), "+v"(aoff1), "+v"(bn0), "+v"(bn1)); } while (0)
#define PINB(B) asm volatile("" : "+v"(B[0][0]), "+v"(B[0][1]), "+v"(B[1][0]), "+v"(B[1][1]))
#define PINC(j) asm volatile("" : "+v"(acc[1][j][0][0]), "+v"(acc[1][j][0][1]), "+v"(acc[1][j][1][0]), "+v"(acc[1][j][1][1]), "+v"(acc[1][j][2][0]), "+v"(acc[1][j][2][1]), "+v"(acc[1][j][3][0]), "+v"(acc[1][j][3][1]))
#define RT(i) do { if (STAMP == 2) { if (i) asm volatile("v_writelane_b32 v235, %0, %1" :: "s"((unsigned)rtp), "n"((i) ? (i) - 1 : 0)); rtp = __builtin_amdgcn_s_memtime(); } } while (0)
#define PR1(r, q) do { if (r == SMB + 1 || r == SMB + 2) { constexpr int i = (r == SMB + 1 ? 0 : 4) + (q); PRLD(i); SCHED; } } while (0)
#define RSTEP(r, Ar, Ap) { constexpr int ai = (r == 0 || r % 3 == 2) ? 0 : HTB; \
      RT(3 * r); \
      if (r == SMB) HLOAD1(232, xflag, "sc0 sc1"); \
      if (r == SMB + 1) { HLOAD1(233, xflag, "sc0 sc1"); HLOAD1(244, tmo, "sc1"); } \
      if (r > 0) { LB4(B1, boff0, boff1, HTB, 0); G0(r, 0); LB4(B1, boff0, boff1, HTB, 1); G0(r, 1); } \
      if (r == SMB + 1) { WAIT_V(NG(r - 1) + 4); got = HFIRST(232) >= call; } \
      PR1(r, 0); \
      LA4(Ar, ai, 0); if (r) GA(r, 0, 238); else G0(r, 0); \
      PR1(r, 1); \
      LA4(Ar, ai, 2); if (r) GA(r, 1, 239); else G0(r, 1); \
      PR1(r, 2); \
      LB4(B0, bn0, bn1, 0, 0); G1(r, 0); \
      PR1(r, 3); \
      LB4(B0, bn0, bn1, 0, 1); G1(r, 1); \
      boff0 = bn0; boff1 = bn1; \
      SCHED; \
      XL(r, 0); XL(r, 1); XL(r, 2); XL(r, 3); \
      if (r + 1 < RL) RNEXT(r); \
      if (r == SMB) { vl16 = LANE() * 16; asm volatile("" : "+v"(vl16)); } \
      if (r == SMB + 2) { const unsigned l_ = vl16 >> 4, g_ = l_ >> 4; \
        orow = (pm * 256 + kh * 128 + wr * 64 + (l_ >> 2)) * (NCOL * 2) + (pn * 256 + wc * 32) * 2 + (l_ & 3) * 16; \
        tw0 = wid * 8192 + (l_ & 15) * 64 + (((g_ >> 1) ^ ((l_ >> 2) & 3)) * 16) + (g_ & 1) * 8; tw1 = tw0 ^ 32; trd = wid * 8192 + (vl16 ^ (g_ * 16)); \
        asm volatile("" : "+v"(orow), "+v"(tw0), "+v"(tw1), "+v"(trd)); } \
      WAIT_V(NS(r - 1) + (PUBB(r - 1) ? 1 : 0) + NL(r - 1) + (r == SMB ? 1 : r == SMB + 1 ? 2 : 0) + NG(r) + NL(r) + NS(r)); \
      if (PUBB(r)) asm volatile("v_mov_b32 v234, %0\n\tglobal_store_dword v237, v234, %1 sc0 sc1" :: "s"(call), "s"(xpub) : "memory"); \
      if (r == SMB + 2) { bad = HFIRST(244) == call; if (__builtin_expect(!got, 0)) { got = HFIRST(233) >= call; PRL(0); SCHED; PRL(4); } } \
      FENCE; WAIT_L0; SCHED; \
      RT(3 * r + 1); \
      BAR; \
      SCHED; \
      PRIO(1); PINB(B1); MM16(1, 1, Ap, B1); PINC(1); PINB(B0); MM16(1, 0, Ar, B0); PINC(0); PRIO(0); \
      RT(3 * r + 2); \
      if (r + 1 < RL || wr == 0) BAR; \
      SCHED; }
    RSTEP(0, A0, A1) RSTEP(1, A1, A0) RSTEP(2, A0, A1) RSTEP(3, A1, A0) RSTEP(4, A0, A1) RSTEP(5, A1, A0) RSTEP(6, A0, A1)
  }
  RT(3 * RL);
  LB4(B1, boff0, boff1, HTB, 0); LB4(B1, boff0, boff1, HTB, 1);
  SCHED;
  const bool got0 = got;
  if (__builtin_expect(!got, 0)) {
    const unsigned t0 = NOW();
    for (;;) {
      HLOAD1(232, xflag, "sc0 sc1"); WAIT_V(0);
      if (HFIRST(232) >= call) break;
      if (expired(t0, call, tmo)) { bad = true; break; }
    }
    PRL(0); PRL(4);
    __builtin_amdgcn_s_waitcnt(0x0F70);
  }
  {
    const __amdgpu_buffer_rsrc_t odst = __builtin_amdgcn_make_buffer_rsrc(p.out, 0, 1024 * NCOL * 2, 0x00020000);
    typedef short s16x2 __attribute__((ext_vector_type(2)));
    typedef unsigned u32x2 __attribute__((ext_vector_type(2)));
#define RELU(x) __builtin_bit_cast(unsigned, __builtin_elementwise_max(__builtin_bit_cast(s16x2, x), (s16x2){0, 0}))
#define OST(i, o) do { if ((OMASK >> (i)) & 1) __builtin_amdgcn_raw_buffer_store_b128(o, odst, orow + ((i) >> 2) * 256, ((i) & 3) * 16 * (NCOL * 2), OAUX); \
      else __builtin_amdgcn_raw_buffer_store_b128(o, odst, orow + ((i) >> 2) * 256, ((i) & 3) * 16 * (NCOL * 2), 0); asm volatile("s_nop 1" :: "v"(o) : "memory"); } while (0)
    u32x4 o[8];
    auto fin = [&](int i, int q) __attribute__((always_inline)) {
      f32x4 v = acc[1][i >> 2][i & 3][q];
      const unsigned a = pr[i][2 * q], b = pr[i][2 * q + 1];
      v[0] += __uint_as_float(a << 16); v[1] += __uint_as_float(a & 0xffff0000u); v[2] += __uint_as_float(b << 16); v[3] += __uint_as_float(b & 0xffff0000u);
      if (i < NPK) asm volatile("" : "+v"(v[0]), "+v"(v[2]));
      *(LAS u32x2*)(LP(q ? tw1 : tw0) + i * 1024) = (u32x2){RELU(cvt_pk_bf16(v[0], v[1])), RELU(cvt_pk_bf16(v[2], v[3]))};
      if (q) { FENCE; o[i] = *(const LAS u32x4*)(LP(trd) + i * 1024); FENCE; }
      if (q == LQ && i >= LAG) OST(i - LAG, o[i - LAG]);
    };
    fin(0, 0); fin(0, 1);
    SCHED;
    RT(3 * RL + 1);
    PINB(B1); MM16H(1, 1, A0, B1, if (i_ % 5 % 2 == 0 && i_ % 5 < 4 && i_ < 15) fin(1 + i_ / 5, i_ % 5 / 2)); PINC(1);
    RT(3 * RL + 2);
    if (STAMP != 2) ST(3);
#pragma unroll
    for (int i = 4; i < 8; ++i) { fin(i, 0); SCHED; fin(i, 1); SCHED; }
#pragma unroll
    for (int i = 8 - LAG; i < 8; ++i) OST(i, o[i]);
    if (__builtin_expect(bad, 0)) {
      unsigned n_; asm volatile("s_waitcnt vmcnt(0)\n\tv_mov_b32 %0, 0x7fc07fc0" : "=v"(n_) :: "memory");
      const u32x4 o_ = {n_, n_, n_, n_};
#pragma unroll
      for (int i = 0; i < 8; ++i) OST(i, o_);
    }
    asm volatile("v_mov_b32 v234, %0\n\ts_mov_b64 exec, 1\n\tglobal_store_dword v237, v234, %1 sc1\n\ts_mov_b64 exec, -1" :: "s"(call), "s"(ctr) : "memory");
  }
  RT(3 * RL + 3);
  if (STAMP == 2) WAIT_V(0);
  RT(3 * RL + 4);
  if (STAMP == 2) { unsigned t_; asm volatile("v_readlane_b32 %0, v235, 40" : "=s"(t_)); DST(12, t_); }
  if (STAMP) { DST(14, got0 ? 0 : 1); DST(0, st0); unsigned xc_; asm volatile("s_getreg_b32 %0, hwreg(HW_REG_XCC_ID, 0, 4)" : "=s"(xc_)); DST(15, xc_); }
  if (STAMP == 2 && (wid & 3) == 0) { RT(3 * RL + 5); asm volatile("global_store_dword %0, v235, %1" :: "v"(LANE() * 4), "s"(p.ws + WS_DBG + (1 << 20) + 131072 + bid * 512 + wr * 256) : "memory"); }
  if (STAMP) { WAIT_V(0); ST(4); }
  break;
  }
}

extern "C" void kernel_launch(int rank, int nranks, ncclComm_t comm,
                              hipStream_t stream,
                              void* const* d_in, const int* in_sizes, int n_in,
                              void* d_out, int out_size,
                              const int* in_dtypes, int out_dtype,
                              void* const* d_out_all,
                              void* const* d_ws_all, size_t ws_bytes) {
  if (nranks != NRANK || (unsigned)rank >= (unsigned)NRANK || n_in < 2 || ws_bytes < WS_NEED ||
      in_sizes[0] != 8192 * 1024 || in_sizes[1] != 8192 * 8192 || out_size != 1024 * 8192) {
    fprintf(stderr, "a2a_gemm: unexpected problem shape\n");
    return;
  }
  HIP_CHECK(hipFuncSetAttribute((const void*)a2a_gemm, hipFuncAttributeMaxDynamicSharedMemorySize, LDS_BYTES));
  Params p;
  p.x = (const char*)d_in[0];
  p.w = (const char*)d_in[1];
  p.out = (char*)d_out;
  p.ws = (char*)d_ws_all[rank];
  p.xo = p.x + rank * BLK_B;
  p.wo = p.w + (size_t)rank * 1024 * 16384;
  p.rank = rank;
  p.duty = 7u << (FLIP ? 28 : 12);
  for (int e = 0; e < 8; ++e) p.peer[e] = p.ws;
  for (int j = 0, n[2] = {0, 4}; j < 7; ++j) {
    const int q = (rank + 1 + j) & 7, e = n[((0xb2 >> rank) ^ (0xb2 >> q) ^ FLIP) & 1]++;
    p.duty |= (unsigned)j << (4 * e);
    p.peer[e] = (char*)d_ws_all[q];
  }
  a2a_gemm<<<256, 512, LDS_BYTES, stream>>>(p);
  HIP_CHECK(hipGetLastError());
}
```

```cpp
#include <hip/hip_runtime.h>
#include <rccl/rccl.h>
#include <cstdio>

#define HIP_CHECK(expr) do { hipError_t _e = (expr); if (_e != hipSuccess) { \
    fprintf(stderr, "HIP error %s at %s:%d\n", hipGetErrorString(_e), __FILE__, __LINE__); \
    return; } } while (0)

typedef unsigned long long u64;

#define W4 1
#define STAMP 0
#define NOCOMM 0
#define FAKEX 0
#define PER 80
#define SHIFT 28
#define FPQ 5
#define TV 5
#define TP 1
#define RL 11
#define AB 3
#define FG 291
#define LAGF 72
#define LAGP 80
#define PFR 17
#define PFS 1
#define PFN 4
#define DR0 46
#define DRA 0x1b
#define DRB 0x16
#define EM 0x4a52
#define SWP 0xc4
#define VTH 40
#define XL 1
#define CE 1
#define EVF 1
#define TCH 0
#define EB 2
#define PFX 1
#define OST " sc1"
#define XST " sc0 sc1"
#define XQ 16

constexpr int NRANK = 8;
constexpr int NCOL = 8192;
constexpr int U = 16384;
constexpr int LDS_BYTES = 10 * U;
constexpr int LDS_CTRL = 9 * U + 1024;
constexpr size_t BLK_B = (size_t)1024 * 1024 * 2;
constexpr int N_POOL = 7;
constexpr int N_X = (RL - 7) / 2;
constexpr unsigned XSRC = N_X == 1 ? 6u : N_X == 4 ? 0x6614u : 0x14614u;
static_assert(PFR >= TV && PFR + ((7 + N_X) * PFN - 1) * PFS <= 54 - N_X, "");
static_assert(__builtin_popcount(DRA) + __builtin_popcount(DRB) == N_POOL && ((u64)(DRA | DRB) << DR0) >> (56 - 3 - N_X) == 0, "the tickets come in time");
static_assert(RL >= 7 && RL <= 17 && (RL & 1), "");
constexpr unsigned RROW = 128, RCOL = 32768;
constexpr int WS_TMO = 0;
constexpr int WS_VOTE = 256;
constexpr size_t WS_CTR = 16384;
constexpr size_t WS_CLAIM = 49152;
constexpr size_t WS_XFLAG = 65536;
constexpr size_t WS_AFLAG = 131072;
constexpr size_t WS_ROLE = 196608;
constexpr size_t WS_A = (size_t)1 << 20;
constexpr size_t WS_XCH = (size_t)34 << 20;
constexpr unsigned AWAY = (50u << 20) + 128;
constexpr size_t WS_DBG = (size_t)101 << 20;
constexpr size_t WS_NEED = (size_t)103 << 20;
constexpr unsigned DEADLINE = 50000000u;

constexpr int event(int s, int k, int p) { return 2 * k + (p ^ (s & 1)); }
constexpr unsigned sender(int e) { return (EM >> e & 1 ? EM : ~EM) & 0xffff; }
constexpr unsigned offA(int s, int k, int p) { return s * BLK_B + event(s, k, p) * RCOL; }
constexpr unsigned offW(int s, int k, int p) { return ((1 + s) * 16 + k / 2 * 4 + p * 2 + k % 2) << 20; }
constexpr int rsrc(int r) { return r < 7 ? r : (XSRC >> (r - 7) / 2 * 4) & 7; }
constexpr int rcol(int r) { return r < 7 ? 7 : (0x55666 >> (r - 7) / 2 * 4) & 7; }
constexpr unsigned table(int q, int l, int i) {
  const int k = l / 7, o = q ? 6 - l % 7 : l % 7, w = l < 7 || l == 63, s = w ? (l + 6) % 7 : (o * 2) % 7, p = w ? ~s & 1 : k < 8 ? q : s & 1, own = l < 5 ? l + 3 : 3;
  const int z = (l - PFR) / PFS, y = z / PFN, r = y < 7 ? y : 2 * y - 7;
  const int e = event(s, w ? 7 : k - 1, p);
  return i == 0 ? (w ? own * 128 : offA(s, k - 1, p)) : i == 1 ? (w ? own << 20 : offW(s, k - 1, p)) : i == 2 ? (l ? s * 128 + (sender(e) & 1) * 16 : 1u << 31)
       : (l >= PFR && z * PFS + PFR == l && r < RL ? offW(rsrc(r), rcol(r), q) + z % PFN * (0x40000 / PFN) : 1u << 31) | __builtin_popcount(sender(e) & ((2 << e) - 1));
}
constexpr u64 TODO = 0x00ffffffffffffbeull & ~((N_X & 1 ? 1ull << 52 : 0) | (N_X >= 2 ? 5ull << 51 : 0) | (N_X == 4 ? 0x81ull << 45 : 0) | (N_X == 5 ? 5ull << 44 : 0));
constexpr int PHASE = PER * 65536 / 112 + 1;
constexpr bool live() {
  for (int q = 0; q < 2; ++q) for (int m = 0; m < 4; ++m) {
    int n[99] = {};
    for (int l = 1; l < 64; ++l) { const int s = table(q, l, 2) / 128, c = table(q, l, 3) & 15, r = table(q, l, 2) / 16 & 1; int t = -2;
      for (int e = 0, i = 0; i < c; ++e) if ((~EM >> e & 1) == r) { const int u = ((((m * 4 + 3) * 7 + 6 - s) * PHASE >> 16) - SHIFT + e * PER) >> 5 & -2; t = u > t + 2 ? u : t + 2; ++i; }
      if (t + 3 > 54 - N_X) return false;
      n[t + 3] += l > 6 && (TODO >> l & 1); }
    for (int t = 1; t < 54 - N_X; ++t) if ((n[t] += n[t - 1]) < t - 4) return false; }
  return (EM & 3) == 2 && (EM >> 14 == 1 || EM >> 14 == 2);
}
static_assert(live(), "");

struct Srd { const void* base; unsigned bytes, mode; };
struct Duty { const char* x; char* to; char* flag; unsigned who, mode; };
struct Params {
  Srd ws, w, x, out;
  Duty duty[8];
};

constexpr int FA0 = 128, FA1 = 160, FB0 = 192, FB1 = 224, FA2 = 0, FA3 = 32, FO = 64, VX = 32, VY = 38, VZ = 44, VRA = 50, VRB = 52, VDA = 72, VDB = 80;
constexpr int SWS = 52, SW = 56, SA = 60, SRA = 44, SLX = 74, SLW = 77, SA0 = 78, SA1 = 79, SB1 = 80, SB0 = 81;
#define sT "s48"
#define sCALL "s49"
#define sMORE "s50"
#define sEV "s51"
#define sWS "s[52:55]"
#define sOUT "s[64:67]"
#define sXD "s[68:69]"
#define sXS "s70"
#define sXF "s71"
#define sXP "s72"
#define sCTR "s73"
#define sAH "s82"
#define sOF "s83"
#define cT "s[88:89]"
#define cK "s[90:91]"
#define cC "s[92:93]"
#define cB "s94"
#define cCL "s95"
#define cTK "s96"
#define cFB "s97"
#define cCM "s98"
#define sT0 "s99"
#define pXN "s[88:89]"
#define pD "s[90:91]"
#define pF "s[92:95]"
#define pB "s96"
#define pE "s97"
#define pS "s98"
#define pM "s99"

#define DI __device__ __forceinline__
#define X(s, ...) asm volatile(s :: __VA_ARGS__ : "memory")
#define Y(s) asm volatile(s ::: "memory")
#define N "\n\t"
template <int V> struct IC { static constexpr int value = V; };
template <int I, int E, class F> DI void rep(F f) { if constexpr (I < E) { f(IC<I>{}); rep<I + 1, E>(f); } }
#define REP(i, a, b, ...) rep<a, b>([](auto i##_) __attribute__((always_inline)) { constexpr int i = decltype(i##_)::value; __VA_ARGS__; })

template <int KEEP, int T, int FB, int FA> DI void mfma() {
  if constexpr (KEEP) X("v_mfma_f32_16x16x32_bf16 v[%0:%0+3], a[%1:%1+3], a[%2:%2+3], v[%0:%0+3]", "n"(128 + T * 4), "n"(FB), "n"(FA));
  else X("v_mfma_f32_16x16x32_bf16 a[%0:%0+3], a[%1:%1+3], a[%2:%2+3], a[%0:%0+3]", "n"(T * 4), "n"(FB), "n"(FA));
}
template <int F, int i, int S, int O> DI void rdA() { X("ds_read_b128 a[%0:%0+3], v[%1] offset:%2", "n"(F + i * 4), "n"(S + (i & 1)), "n"(O + (i >> 1) * 2048)); }
template <int F, int i, int S, int O> DI void rdB() { X("ds_read_b64_tr_b16 a[%0:%0+1], v[%1] offset:%2", "n"(F + i * 2), "n"(S + 2 + (i >> 2)), "n"(O + ((i >> 1) & 1) * 8192 + (i & 1) * 1024)); }
template <int S, int O> DI void m0() { X("s_add_u32 m0, s[%0], %1", "n"(S), "n"(O)); }
template <int V, int R, int SO, int i> DI void dmaA() { X("buffer_load_dwordx4 v[%0], s[%1:%1+3], s[%2] offen offset:%3 sc0 sc1 lds", "n"(V), "n"(R), "n"(SO), "n"((i & 3) * 1024)); }
template <int V, int SO, int i> DI void dmaB() { X("buffer_load_dwordx4 v[%0], s[56:59], s[%1] offen offset:%2 lds", "n"(V), "n"(SO), "n"((i & 3) * 1024)); }
template <int V, int L> DI void wait() {
  if constexpr (V < 0) X("s_waitcnt lgkmcnt(%0)", "n"(L)); else if constexpr (L < 0) X("s_waitcnt vmcnt(%0)", "n"(V)); else X("s_waitcnt vmcnt(%0) lgkmcnt(%1)", "n"(V), "n"(L));
}
template <int P, int Q, int i> DI void fm0() { if constexpr (Q == 0 || Q == 3) m0<SLW, (P * 3 + Q / 3) * U + (i >> 2) * 4096>(); else m0<87 + P * 2 + Q, (i >> 2) * 4096>(); }
template <int Q, int i, int SO = Q == 3 ? SA1 : Q ? SA1 + Q : SA0> DI void fill() { if constexpr (Q == 0 || Q == 3) dmaA<VDA + i, SA, SO, i>(); else dmaB<(i < 4 ? VRB : VDB) + i, SO, i>(); }
constexpr int early(int P, int Q, int g) {
  int n = P * 44 + Q * 12;
  for (int x = 0; x < 32; ++x) { const bool y = x % 4 == 1 || (Q == 0 ? x >= 15 && x < 30 && x % 2 : Q < 3 && x % 8 == 3); if (x == g) return y ? n : -1; n += y; }
  return -1;
}
#define STP(i) do { if (STAMP) X("s_memrealtime s[84:85]" N "s_memtime s[100:101]" N "s_waitcnt lgkmcnt(0)" N "v_writelane_b32 v31, s84, %0" N "v_writelane_b32 v31, s85, %0+1" N \
    "v_writelane_b32 v31, s100, %0+10" N "v_writelane_b32 v31, s101, %0+11", "n"(2 * (i))); } while (0)
#define STQ(i) do { if (STAMP == 3) X("s_memrealtime s[84:85]" N "s_waitcnt lgkmcnt(0)" N "v_writelane_b32 v31, s84, %0", "n"(20 + (i))); } while (0)
#define CNT(lane, n) do { if (STAMP && STAMP != 3) X("v_readlane_b32 s84, v31, %0" N "s_add_u32 s84, s84, %1" N "v_writelane_b32 v31, s84, %0", "n"(2 * (lane)), "n"(n)); } while (0)
#define SAMPLE "buffer_load_dwordx4 v[72:75], v82, " sWS ", " cFB " offen sc0 sc1"
#define ARRIVED "v_min3_u32 v77, v72, v73, v74" N "v_min_u32 v77, v77, v75" N "v_cmp_le_u32 vcc, v76, v77" N "s_or_b64 " cK ", " cK ", vcc"
#define EXPIRED "buffer_load_dword v60, off, " sWS ", 0 sc1" N "s_memrealtime s[84:85]" N "s_sleep 1" N "s_waitcnt vmcnt(0) lgkmcnt(0)" N "v_readfirstlane_b32 s85, v60" N \
    "s_or_b32 s84, s84, 1" N "s_cmp_eq_u32 " sT0 ", 0" N "s_cselect_b32 " sT0 ", s84, " sT0 N "s_sub_u32 s84, s84, " sT0 N \
    "s_cmp_eq_u32 s85, " sCALL N "s_cselect_b32 s84, -1, s84" N "s_cmp_gt_u32 s84, %[dl]" N "s_cbranch_scc0 1f" N \
    "v_mov_b32 v60, " sCALL N "buffer_store_dword v60, off, " sWS ", 0 sc1\n1:" N
#define DL [dl] "n"(DEADLINE)
#define NEXT0 "s_ff1_i32_b32 " pE ", " pM N "s_bitset0_b32 " pM ", " pE
#define NEXT1 "s_lshl_b32 s84, " pE ", 2"
#define NEXT2 "s_lshr_b64 s[84:85], s[20:21], s84"
#define NEXT3 "s_and_b32 s84, s84, 15" N "s_lshl_b32 s84, s84, 7"
#define NEXT4 "s_add_u32 s88, s16, s84" N "s_addc_u32 s89, s17, 0"

constexpr int nth(int g) { return (g & 3) == 1 ? -99 : g - (g + 2) / 4; }
constexpr int sb(int g) { return ((g >> 1) & 3) ^ (g & 8 ? 3 : 0); }
constexpr int sk(int g) { return (g ^ (g >> 1)) & 1; }
template <int ROLE, int P> DI void stile() {
  REP(Q, 0, 4, REP(g, 0, 32,
    constexpr int cur = P ? VY : VX, nxt = P ? VX : VY, keep = Q == 0 || Q == 3, a = g >> 3, m = sb(g), k = g & 1, r = nth(g), y = nth(g & 7);
    mfma<keep, (Q >> 1) * 16 + m * 4 + a, (keep ? FB1 : FB0) + (a * 2 + sk(g)) * 4, (Q >> 1 ? FA1 : FA0) + (m * 2 + sk(g)) * 4>();
    if constexpr (Q == 0 && r >= 0 && r < 4) rdB<FB1, 12 + r, cur, 2 * U>();
    if constexpr (Q == 0 && r >= 4 && r < 11) rdB<FB0, 5 + r, VZ, P * U>();
    if constexpr (Q == 1 && g < 16 && !k) rdA<FA1, g / 2, cur, U>();
    if constexpr (Q == 2 && g < 16 && !k) rdA<FA0, g / 2, nxt, 0>();
    if constexpr (Q == 3 && a > 0 && y >= 0 && y < 4) rdB<FB1, (a - 1) * 4 + y, nxt, 2 * U>();
    if constexpr (Q == 3 && a < 3 && y >= (a ? 4 : 1)) rdB<FB0, a * 2 + y - 1, VZ, (P ^ 1) * U>();
    if constexpr (Q == 0 && g == 15) wait<-1, 7>();
    if constexpr (Q == 3 && g == 0) { if constexpr (ROLE % 4) Y("ds_read_b32 v66, v68 offset:8"); else Y("ds_read_b96 v[64:66], v68"); }
    if constexpr (Q == 3 && g == 8) wait<-1, 6>();
    if constexpr (Q == 3 && g == 10) Y("v_readfirstlane_b32 " sMORE ", v66");
    if constexpr (ROLE % 4 == 0) { constexpr int e = early(P, Q, g), f = early(P, Q, g + 1);
      if constexpr (!ROLE && g % 16 == 0) fm0<P, Q, g / 4>();
      if constexpr (!ROLE && g % 4 == 1) fill<Q, g / 4>();
      if constexpr (ROLE && f >= 0 && f % 4 == 0) fm0<(f / 8 + 5) / 4 % 2, (f / 8 + 1) % 4, f % 8>();
      if constexpr (ROLE && e >= 0) { if constexpr (e < 56) fill<(e / 8 + 1) % 4, e % 8, 93 + e / 8>(); else fill<(e / 8 + 1) % 4, e % 8>(); }
      if constexpr (ROLE && P && Q == 1 && g == 18) Y("v_readfirstlane_b32 s4, v63");
      if constexpr (ROLE && P && Q == 1 && g == 22) Y("s_add_u32 " sCALL ", s4, 1" N "s_lshl_b32 s5, " sCALL ", 24");
      if constexpr (ROLE && P && Q == 1 && g == 23) Y("s_and_b32 s5, s5, 0x1000000" N "s_add_u32 s44, s44, s5");
      if constexpr (ROLE && P && Q == 1 && g == 26) Y("s_xor_b32 s45, s44, 0x4000");
      if constexpr (Q == 0 && g == 22) Y("s_xor_b32 s79, s78, " sAH);
      if constexpr (Q == 3 && g == 11) Y("v_readfirstlane_b32 s78, v64");
      if constexpr (Q == 3 && g == 12) Y("v_readfirstlane_b32 s80, v65");
      if constexpr (Q == 3 && g == 14) Y("s_xor_b32 s80, s80, s86");
      if constexpr (Q == 3 && g == 15) Y("s_xor_b32 s81, s80, 0x100");
    }
    if constexpr (ROLE == 1 && Q == 1 && g == 17) Y("v_readlane_b32 s87, v83, " sT);
    if constexpr (ROLE == 1 && Q == 1 && g == 23) Y("buffer_load_dword v91, v92, s[56:59], s87 offen");
    if constexpr (ROLE == 1 && !(Q & 1) && g == 26) Y("s_lshl_b64 exec, 1, " sT);
    if constexpr (ROLE == 1 && !(Q & 1) && g == 27) X("buffer_atomic_add v[%0], v[%1], " sWS ", " cCM " offen sc0" N "s_mov_b64 exec, -1", "n"(Q ? 69 : 78), "n"(70 + Q / 2));
    if constexpr (ROLE == 1 && Q == 1) {
      if constexpr (g == 24) Y("v_cmp_le_u32 vcc, " cTK ", v78" N "v_cmp_le_u32 s[84:85], " cTK ", v69");
      if constexpr (g == 26) Y("s_bcnt1_i32_b64 s86, vcc" N "s_bcnt1_i32_b64 s84, s[84:85]");
      if constexpr (g == 27) Y("s_lshr_b32 s86, " cCL ", s86" N "s_lshr_b32 s86, s86, s84");
      if constexpr (g == 28) Y("s_and_b32 s93, s86, " cCL N "s_cselect_b32 s92, 0, 1");
    }
    if constexpr (ROLE == 1 && Q == 2) {
      if constexpr (g == 7) { wait<2, -1>(); Y("v_min3_u32 v77, v72, v73, v74"); }
      if constexpr (g == 8) Y("v_min_u32 v77, v77, v75");
      if constexpr (g == 9) Y("v_cmp_le_u32 vcc, v76, v77");
      if constexpr (g == 11) Y("s_or_b64 " cK ", " cK ", vcc" N "s_cmp_eq_u64 " cT ", 0");
      if constexpr (g == 13) Y("s_cselect_b64 " cT ", " cC ", " cT N "s_cselect_b32 " cCL ", 0, " cCL);
      if constexpr (g == 15) Y("s_cmp_eq_u64 " cK ", -2");
      if constexpr (g == 16) Y("s_cselect_b64 " cC ", 63, 62" N "s_or_b64 " cC ", " cC ", " cK);
      if constexpr (g == 17) X("s_and_b64 " cC ", " cC ", " cT N "s_cbranch_scc0 .Lc_slow\n.Lc_back%0:", "n"(P));
      if constexpr (g == 18) Y("s_ff1_i32_b64 " cB ", " cC N "s_lshl_b64 " cC ", 1, " cB);
      if constexpr (g == 19) Y("s_andn2_b64 " cT ", " cT ", " cC);
      if constexpr (g == 20) Y("v_cndmask_b32 v88, v89, v68, " cC N "v_cndmask_b32 v82, v82, v90, " cK);
      if constexpr (g >= 21 && g < 24) X("ds_write_b32 v88, v[%0] offset:%1", "n"(79 + (g - 20) % 3), "n"((g - 21) * 4));
      if constexpr (g == 25) Y(SAMPLE);
    }
    if constexpr (ROLE == 1 && STAMP == 2 && Q == 1 && g == 20) Y("s_memtime s[100:101]" N "s_mov_b32 m0, " sT);
    if constexpr (ROLE == 1 && STAMP == 2 && Q == 2 && g == 1) Y("v_writelane_b32 v59, s100, m0");
    if constexpr ((ROLE & 2) && P * 4 + Q <= (ROLE & 1 ? 4 : FPQ)) {
      constexpr int young = ROLE & 1 ? (P * 4 + Q) * 3 + 3 : P * 4 + Q + 1;
      if constexpr (g == 9) { wait<(P * 4 + Q < FPQ ? 15 : young), -1>(); Y("s_getreg_b32 s84, hwreg(HW_REG_IB_STS, 0, 4)"); }
      if constexpr (g == 13) X("s_or_b32 s84, s84, " pB N "s_cmp_le_u32 s84, %0", "n"(young));
      if constexpr (g == 17) Y("s_cselect_b32 s94, 4, 0" N "s_cselect_b32 " pB ", 64, " pB);
      if constexpr (g == 21) Y("buffer_store_dword v81, off, " pF ", 0 sc0 sc1");
    }
    if constexpr (ROLE == 3) { constexpr int u = P * 4 + Q;
      if constexpr (g == 1 && u % 7 == 0) wait<19, -1>();
      if constexpr (g == 1) X("global_store_dwordx4 v56, v[%0:%0+3], " pD " offset:%1 sc0 sc1", "n"(96 + u * 4), "n"(u * 1024 - 4096));
      if constexpr (g == 5) X("global_load_dwordx4 v[%0:%0+3], v[%1], " pXN, "n"(96 + u * 4), "n"(88 + u));
      if constexpr (u == 6 && g == 17) X("s_mul_i32 s84, " pE ", %0" N "s_add_i32 s84, s84, " pS, "n"(PER));
      if constexpr (u == 6 && g == 18) Y("s_ashr_i32 s84, s84, 5" N "s_and_b32 s84, s84, -2");
      if constexpr (u == 6 && g == 19) Y("s_add_u32 " sEV ", " sEV ", 2" N "s_max_i32 " sEV ", " sEV ", s84");
      if constexpr (u == 6 && g == 20) Y("s_cmp_ge_i32 " pE ", 0" N "s_cselect_b64 vcc, -1, 0");
      if constexpr (u == 6 && g == 21) Y("s_orn2_b32 " sEV ", " sEV ", vcc_lo" N "s_lshl_b32 s84, " pE ", 15");
      if constexpr (u == 6 && g == 22) Y("s_mov_b32 " pB ", 0" N "v_add_u32 v81, 1, v81");
      if constexpr (u == 7 && g == 1) Y("s_add_u32 s90, s22, s84" N "s_addc_u32 s91, s23, 0");
      if constexpr (u == 7 && g == 9) Y(NEXT0);
      if constexpr (u == 7 && g == 11) Y(NEXT1);
      if constexpr (u == 7 && g == 12) Y(NEXT2);
      if constexpr (u == 7 && g == 13) Y(NEXT3);
      if constexpr (u == 7 && g == 17) Y(NEXT4);
    }
    if constexpr (!(Q & 1) && g == 30) wait<ROLE % 4 ? -1 : !ROLE ? 32 : P * 12 + Q * 4 + 12, 0>();
    if constexpr (!(Q & 1) && g == 31) Y("s_barrier");
    if constexpr (Q == 1 && g == 31) wait<-1, 0>();
    if constexpr (Q == 3 && g == 18) Y("s_add_u32 " sT ", " sT ", 1");
    if constexpr (Q == 3 && g == 19) Y("s_cmp_eq_u32 " sT ", " sEV);
    if constexpr (Q == 3 && g == 20) Y("s_cselect_b32 s84, 0, " sMORE);
    if constexpr (Q == 3 && g == 22) Y("s_cmp_lg_u32 s84, 1");
    if constexpr (Q == 3 && g == 30) wait<-1, 6>();
  ));
}
#define SLOOP(role, l, ev) do { Y(l "0:"); stile<role, 0>(); Y("s_cbranch_scc1 " ev "\n" l "1:"); stile<role, 1>(); Y("s_cbranch_scc0 " l "0"); } while (0)
#define BACK(l) "s_bitcmp1_b32 " sT ", 0" N "s_cbranch_scc1 " l "1" N "s_branch " l "0"
#define LEAVE "s_cmp_lg_u32 " sMORE ", 1" N "s_cbranch_scc1 .Lexit" N

enum { NONE, RA, RB, M0, DA, DB, WT, BR, RN, ZB, XS, PB, FL, FC, PR, AD, OU, NP, TS, ID, OA };
struct Op { short k, a, b, c, d; };
constexpr int NQ = 8 + 2 * RL, NG = NQ * 32, L = RL - 1, NX = NG + 48;
struct Plan {
  Op mf[NG] = {}, op[NX][8 * AB + 6] = {};
  char n[NX] = {}, room[NX] = {};
  bool ok = true;
  short mark[160] = {};
  static constexpr bool st4(const Op& o) { return (o.k == XS && o.a % 13 == 12) || (o.k == OU && o.b == 5); }
  static constexpr bool vm(const Op& o) { return o.k == DA || o.k == DB || o.k == FL || o.k == PR || st4(o) || (o.k == PB && o.a); }
  static constexpr bool ds(const Op& o) { return o.k == RA || o.k == RB || (o.k == OU && (o.b == 2 || o.b == 3)); }
  static constexpr bool vs(const Op& o) { return o.k == FC && o.a < 2; }
  static constexpr bool va(const Op& o) { return vs(o) || o.k == ZB || o.k == ID || ((o.k == XS || o.k == PB || o.k == OU) && !vm(o) && !ds(o)); }
  static constexpr bool wr(const Op& o) { return (o.k == PB && o.a) || (o.k == OU && o.b == 2); }
  constexpr bool fits(int g, const Op& o) {
    if (n[g] >= 2 + room[g]) return false;
    if (n[g] && op[g][n[g] - 1].k == AD) return va(o) || (ds(o) && !wr(o));
    if (st4(o)) return !n[g] || (n[g] == 1 && op[g][0].k == WT);
    if (g < NG - 128 && (o.k == RA || o.k == RB) && g % 4 == 1) return false;
    for (int j = 0; j < n[g]; ++j) { const Op& p = op[g][j];
      if (wr(p) || ((vm(o) || ds(o)) && (va(p) || p.k == BR || ds(p) || (vm(p) && !st4(p)))) || (vs(p) && !va(o) && o.k != WT)) return false; }
    return true;
  }
  constexpr int put(int g, int k, int a = 0, int b = 0, int c = 0, int d = 0) { while (!fits(g, Op{(short)k, (short)a, (short)b, (short)c, (short)d})) ++g; return at(g, k, a, b, c, d); }
  constexpr int at(int g, int k, int a = 0, int b = 0, int c = 0, int d = 0) { const Op o = {(short)k, (short)a, (short)b, (short)c, (short)d}; if (st4(o)) room[g] = 2 + n[g]; if (k == AD) room[g] = n[g]; op[g][(int)n[g]++] = o; return g; }
  constexpr void mm(int q, int keep, int half, int fb, int fa) {
    for (int g = q * 32; g < q * 32 + 32; ++g) { const int nn = (g >> 3) & 3, m = sb(g), k = sk(g);
      mf[g] = Op{(short)keep, (short)(half * 16 + m * 4 + nn), (short)(fb + (nn * 2 + k) * 4), (short)(fa + (m * 2 + k) * 4), 0}; }
  }
  static constexpr int set(int r) { return r % 3 == 0 ? VX : r % 3 == 1 ? VY : VZ; }
  constexpr void ra(int g, int st, int f, int s, int u, int tag) { for (int i = 0; i < 8; ++i) put(g + i * st, RA, tag, f * 16 + i, s, u); }
  constexpr void rb(int g, int i0, int i1, int f, int s, int u, int tag) { for (int i = i0; i < i1; ++i) put(g + i - i0, RB, tag, f * 16 + i, s, u); }
  constexpr void jit(int q, int s) { for (int j = 0; j < 4; ++j) rb(q * 32 + 8 + j * 8, j * 4, j * 4 + 4, FB1, s, 2, 90 + (j >> 1)); }
  constexpr void sync(int q, int t0, int t1, int lt) { at(q * 32 + 31, WT, t0, t1, lt); at(q * 32 + 31, BR); }
  int nf = 0;
  constexpr void fill(int g, int skip, int u, int r) {
    if (r > L) return;
    const int so = SA0 + (nf++ & 1), tag = (u == 2 ? 20 : u * 40) + r;
    ok = ok && put(g - 7, RN, so, u, r) <= g - 4 && (u < 2 || put(g - 3, RN, so, 3) < g) && put(g - 2, M0, r % 3 < 2 ? SLX + r % 3 : SLW, u + r % 3 / 2 * 6) < g;
    for (int i = 0; i < 4; g += 4) if (!((skip >> (g / 4 % 4)) & 1)) ok = ok && put(g, u == 2 ? DB : DA, tag, i++, so) == g;
  }
  constexpr int count(bool fix, int tag) {
    int nv = 0, nl = 0;
    for (int i = 0; i < 160; ++i) mark[i] = 30000;
    mark[60] = -24; mark[61] = -8; mark[62] = 0;
    for (int g = 0; g < NX; ++g) for (int j = 0; j < n[g]; ++j) { Op& o = op[g][j];
      if (vm(o)) { ++nv; mark[o.k == XS ? (o.a == 207 ? 64 : 159) : o.k == FL ? 65 : o.k == PR ? 70 + o.a : (o.k == DA || o.k == DB) && o.b == 3 ? o.a : 159] = nv; }
      if (ds(o)) { ++nl; mark[o.k == OU ? 100 + o.a : o.a] = nl; }
      if (o.k == WT) {
        const int v = o.a < 0 ? -1 : nv - (o.b >= 0 && mark[o.b] > mark[o.a] ? mark[o.b] : mark[o.a]), l = o.c < 0 ? -1 : o.c == 0 ? 0 : nl - mark[o.c];
        if (!fix && o.a >= 0 && v <= nv - mark[tag]) return g;
        if (fix) { ok = ok && (o.a < 0 || (v >= 0 && v < 64)) && (o.c < 0 || l >= 0); o.a = v; o.c = l > 14 ? 14 : l; }
      } }
    return -1;
  }
  int st = 0, lw[32] = {}, fin[32] = {}, ad[32] = {}, io = 0, no = 0;
  Op ou[200] = {};
  constexpr void out(int g) {
    for (bool go = true; go && io < no; ) {
      go = false;
      for (int x = io; x < no && x < io + 12 && !go; ++x) { const Op o = ou[x]; const int T = o.a * 4 + o.c;
        if (o.k == NONE || (x > io && !va(o))) continue;
        if (o.k == OU && o.b == 0 && !(g >= fin[T] && st - lw[T] >= 8 && ad[T])) {
          if (x == io && g >= NG && ad[T]) { at(g, NP, 7 - (st - lw[T])); st += 8 - (st - lw[T]); go = true; }
          break; }
        if (g < NG ? fits(g, o) && (o.k != WT || !n[g]) : x == io && n[g] < 2) { at(g, o.k, o.a, o.b, o.c, o.d); ou[x].k = NONE; ++st; go = true; }
        else if (va(o)) break;
      }
      while (io < no && ou[io].k == NONE) ++io;
    }
  }
  constexpr Plan() {
    sync(0, 60, -1, 0); sync(2, 61, -1, 0); sync(4, 62, 0, 0); sync(6, 20, -1, 0); sync(7, 40, -1, 90);
    for (int r = 0; r < L - 1; ++r) { sync(8 + 2 * r, r + 1, 21 + r, 0); if (r < L - 2) sync(9 + 2 * r, 41 + r, -1, 90); else sync(9 + 2 * r, L, 20 + L, 90); }
    sync(NQ - 4, 40 + L, -1, 97);
    if (STAMP == 2) for (int q = 0; q < NQ; ++q) { at(q * 32 + 23, TS, q); at(q * 32 + 23, NONE); }
    rb(0, 12, 16, FB1, VX, 2, 91); rb(4, 9, 16, FB0, VZ, 0, 92);
    for (int t = 0; t < 2; ++t) { const int q = t * 4;
      mm(q, 1, 0, FB1, FA0); mm(q + 1, 0, 0, FB0, FA0); mm(q + 2, 0, 1, FB0, FA1); mm(q + 3, 1, 1, FB1, FA1);
      ra(q * 32 + 32, 2, FA1, set(t), 1, 92); ra(q * 32 + 64, 2, FA0, set(t ^ 1), 0, 92); jit(q + 3, set(t ^ 1));
      if (!t) rb(98, 0, 16, FB0, VZ, 0, 92);
      at(q * 32 + 63, WT, -1, -1, 0);
    }
    for (int r = 0; r < L - 1; ++r) { const int q = 8 + 2 * r;
      mm(q, 1, 0, FB1, FA0); mm(q + 1, 1, 1, FB1, FA1);
      jit(q + 1, set(r + 1)); ra(q * 32 + 6, 2, FA1, set(r), 1, 92); ra(q * 32 + 32, 1, FA0, set(r + 1), 0, 92);
    }
    for (int g = 0; g < 128; ++g) { const int k = g & 1, m = g < 24 ? g % 6 / 2 : g < 48 ? g / 8 - 3 : g / 16, nn = g < 24 ? g / 6 : g < 48 ? g / 2 % 4 : g / 4 % 4, s = g < 48 ? g >= 24 : g / 2 % 2;
      mf[NG - 128 + g] = Op{1, (short)(m * 4 + nn), (short)((s ? FB0 : FB1) + (nn * 2 + k) * 4), (short)((m < 4 ? s ? FA2 : FA0 : s ? FA3 : FA1) + (m % 4 * 2 + k) * 4), 0}; }
    for (int i = 0; i < 8; ++i) { put(NG - 124, RA, 97 + i / 6, FA2 * 16 + i, set(L), 0); if (i == 1) { rb(NG - 124, 0, 8, FB0, set(L), 2, 92); rb(NG - 124, 8, 16, FB0, set(L), 2, 89); } }
    for (int i = 0; i < 16; ++i) put(NG - 84 + (i >> 2) * 16 + (i & 3), RA, 93 + (i >> 2), (i & 2 ? FA3 : FA1) * 16 + (i >> 2) * 2 + (i & 1), set(L - 1 + ((i >> 1) & 1)), 1);
    at(15, WT, -1, -1, 91); at(127, WT, -1, -1, 90); at(143, WT, -1, -1, 91);
    for (int q = 8; q < NQ - 4; q += 2) at(q * 32 + 15, WT, -1, -1, 91);
    at(NG - 117, WT, -1, -1, 91); at(NG - 105, WT, -1, -1, 92); at(NG - 101, WT, -1, -1, 89); at(NG - 81, WT, -1, -1, 98);
    for (int i = 0; i < 4; ++i) at(NG - 65 + i * 16, WT, -1, -1, 93 + i);
    fill(9, 2, 0, 0); fill(33, 10, 2, 0); fill(65, 10, 2, 2); fill(97, 10, 1, 0); fill(129, 10, 0, 1); fill(161, 10, 2, 1); fill(193, 10, 0, 2);
    for (int r = -1; r < L; ++r) { const int g = (9 + 2 * r) * 32; if (r < 0) fill(g + 1, 8, 1, 1); else fill(g + 1, 8, 2, r + 3); fill(g + 21, 8, 1, r + 3); fill(g + 41, 8, 0, r + 4); }
    for (int t = 0; t < 2; ++t) for (int i = 0; i < 4; ++i) put(t * 128 + 50 + 2 * i, ZB, i, t);
    { int g = 6 * 32 + 2;
      for (int i = 0; i < 208; ++i) {
        if (i % 13 == 12) { g += 1; while (g % 4 != 1 || n[g] || g < 6 * 32 + 13 + i / 13 * XQ) ++g; }
        g = put(g, XS, i); ok = ok && (i < 104 || g >= 7 * 32 + 8);
      } }
    { int g = count(false, 64); ok = ok && g > 0; g = put(g - 3, PB, 0) + 4; while (n[g] > 1 || va(op[g][0])) ++g; at(g, PB, 1); ok = ok && g < NG - FG;
      for (int i = 0; i < 21; ++i) g = put(g + 1, OA, i);
      for (int i = 0; i < 4; ++i) g = put(g + 1, ID, i); }
    int gp[16] = {};
    { int g = put(NG - FG, FL, 1); ok = ok && put(g - 16, FL, 0) < g; g += LAGF;
      g = put(g, WT, 65, -1, -1); g = put(g, FC, 0); g = put(g + 1, FC, 1); ++g;
      do ++g; while (n[g] || g % 4 == 1);
      at(g, FC, 2); at(g, NONE);
      for (int i = 0; i < 16; ++i) { do ++g; while (g % 4 != 1 || n[g] > 1 || vm(op[g][0]) || va(op[g][0])); gp[i] = at(g, PR, i); } }
    for (int G = 0; G <= 8; ++G) for (int c = 0; c < 4; ++c) {
      if (G < 8) for (int e = 0; e < 4; ++e) ou[no++] = Op{OU, (short)G, (short)(e >> 1), (short)c, (short)(e & 1)};
      if (G > 0 && c == 2) ou[no++] = Op{WT, -1, -1, (short)(99 + G), 0};
      if (G > 0 && c >= 2) ou[no++] = Op{OU, (short)(G - 1), 5, (short)(c - 2), 0};
      if (G < 8) ou[no++] = Op{OU, (short)G, 2, (short)c, 0};
      if (G < 8 && c == 3) for (int e = 0; e < 2; ++e) ou[no++] = Op{OU, (short)G, 3, (short)e, 0};
    }
    for (int g = 0; g < NG; ++g) if (mf[g].k) fin[mf[g].a] = g;
    for (int g = 0; g < NX; ++g) {
      if (g < NG) { ++st; if (mf[g].k) lw[mf[g].a] = st; }
      st += n[g];
      out(g);
      if (g < NG - 4 && !room[g] && (!n[g] || (op[g][n[g] - 1].k != BR && op[g][0].k != BR))) {
        int c = 0, u[8] = {}, f = 0;
        while (f < 8 && ad[f * 4]) ++f;
        for (int G = f; G < 8 && c < AB; ++G) {
          bool y = !ad[G * 4] && g >= gp[G * 2 + 1] + LAGP;
          for (int i = 0; i < 4; ++i) y = y && st + 1 + c * 4 + i - lw[G * 4 + i] >= 6 && mf[g + 1 + i].a / 4 != G;
          if (y) u[c++] = G; }
        if (c && u[0] == f && (c > 1 || (g >= fin[f * 4 + 3] && io >= f * 25 - 14) || f == 7)) {
          at(g, WT, 71 + u[c - 1] * 2, -1, -1); ++st;
          for (int i = 0; i < c * 4; ++i) { const int T = u[i / 4] * 4 + i % 4; at(g, AD, T); lw[T] = ++st - 3; ad[T] = 1; out(g); } }
      }
    }
    ok = ok && io == no;
    count(true, 0);
  }
};
constexpr Plan PLAN;
static_assert(PLAN.ok, "");
template <int k, int a, int b, int c, int d> DI void emit() {
  if constexpr (k == RA) rdA<b / 16, b % 16, c, d * U>();
  if constexpr (k == RB) rdB<b / 16, b % 16, c, d * U>();
  if constexpr (k == M0) m0<a, b * U>();
  if constexpr (k == DA) dmaA<VRA + (b & 1), SWS, c, b>();
  if constexpr (k == DB) dmaB<VRB + b, c, b>();
  if constexpr (k == WT) wait<a, c>();
  if constexpr (k == BR) Y("s_barrier");
  if constexpr (k == RN && b < 3) X("s_add_u32 s[%0], s[%1], %2", "n"(a), "n"(SRA + b), "n"(b < 2 ? offA(rsrc(c), rcol(c), ~c & 1) : offW(rsrc(c), rcol(c), ~c & 1)));
  if constexpr (k == RN && b == 3) X("s_and_b32 s[%0], s[%0], 0x7ffffff", "n"(a));
  if constexpr (k == ZB) { if constexpr (b) X("v_and_b32 v[%0], 0xffffbfff, v[%0]", "n"(VZ + 2 + a)); else X("v_xor_b32 v[%0], 0x4000, v[%0]", "n"(VZ + 2 + a)); }
  if constexpr (k == XS) { constexpr int i = a / 13, e = a % 13, o = 64 + (i & 1) * 4;
    if constexpr (e == 12) X("global_store_dwordx4 v[%0], v[%1:%1+3], " sXD " offset:%2" XST, "n"(56 + (i >> 3)), "n"(o), "n"((i & 7) * 1024 - 4096));
    else if constexpr (e >= 8) X("v_cvt_pk_bf16_f32 v[%0], v[%1], v[%1+1]", "n"(o + e - 8), "n"((e - 8) * 2));
    else X("v_accvgpr_read_b32 v[%0], a[%1]", "n"(e), "n"(i * 8 + e)); }
  if constexpr (k == PB) { if constexpr (a) Y("buffer_store_dword v62, off, " sWS ", " sXP " sc0 sc1"); else Y("v_mov_b32 v62, " sCALL); }
  if constexpr (k == FL) { if constexpr (a) Y("buffer_load_dword v61, off, " sWS ", " sXF " sc0 sc1"); else Y("buffer_load_dword v60, off, " sWS ", 0 sc1"); }
  if constexpr (k == FC) {
    if constexpr (a == 0) Y("v_readfirstlane_b32 " sMORE ", v60");
    if constexpr (a == 1) Y("v_readfirstlane_b32 s84, v61");
    if constexpr (a == 2) Y("s_cmp_lt_u32 s84, " sCALL N "s_cbranch_scc1 .Lx_slow\n.Lx_back:"); }
  if constexpr (k == PR) X("buffer_load_dwordx4 v[%0:%0+3], v56, " sWS ", s[%2] offen offset:%1 sc0 sc1", "n"(64 + a * 4), "n"((a & 3) * 1024), "n"(24 + a / 4));
  if constexpr (k == AD) X("v_mfma_f32_4x4x4_16b_bf16 v[%0:%0+3], v[8:9], v[%1:%1+1], v[%0:%0+3]", "n"(128 + a * 4), "n"(64 + a * 2));
  if constexpr (k == ID) { if constexpr (a == 0) Y("v_and_b32 v10, 48, v56"); if constexpr (a == 1) Y("v_mov_b32 v8, 0x3f80"); if constexpr (a == 2) Y("v_mov_b32 v9, 0");
    if constexpr (a == 3) Y("v_lshlrev_b64 v[8:9], v10, v[8:9]"); }
  if constexpr (k == OA && a == 20) Y("s_mov_b32 " sT0 ", 0");
  if constexpr (k == OA && a >= 16 && a < 20) X("s_add_u32 s[%0], " sXS ", %1", "n"(8 + a), "n"((a - 16) * 4096));
  if constexpr (k == OA && a < 16) X("s_xor_b32 s[%0], " sOF ", %1", "n"(8 + a), "n"(((a >> 3) * 128 + (a & 7) * 8) * (NCOL * 2)));
  if constexpr (k == OU) { constexpr int t = c * 2 + d, r = 128 + (a * 4 + c) * 4 + d * 2, o = FO + (a * 2 + c) * 4;
    if constexpr (b == 0) X("v_cvt_pk_bf16_f32 v[%0], v[%1], v[%1+1]", "n"(t), "n"(r));
    if constexpr (b == 1) X("v_pk_max_i16 v[%0], v[%0], 0", "n"(t));
    if constexpr (b == 2) X("ds_write_b64 v[%0], v[%1:%1+1] offset:%2", "n"(24 + c), "n"(t), "n"((a & 1) * 2048));
    if constexpr (b == 3) X("ds_read_b128 a[%0:%0+3], v28 offset:%1", "n"(o), "n"((a & 1) * 2048 + c * 1024));
    if constexpr (b == 5) X("buffer_store_dwordx4 a[%0:%0+3], v29, " sOUT ", s[%1] offen" OST, "n"(d ? FO : o), "n"(8 + a * 2 + c)); }
  if constexpr (k == NP) X("s_nop %0", "n"(a));
  if constexpr (k == TS) { if constexpr (a) X("v_writelane_b32 v30, s100, %0", "n"(a - 1)); Y("s_memtime s[100:101]"); }
}

template <int i> DI void zero() {
  if constexpr (i & 1) X("v_mfma_f32_32x32x4_2b_bf16 v[%0:%0+31], v[60:61], v[60:61], 0", "n"(128 + i / 2 * 32));
  else X("v_mfma_f32_32x32x4_2b_bf16 a[%0:%0+31], v[60:61], v[60:61], 0", "n"(i / 2 * 32));
}
#define CALL(n) "s_waitcnt vmcnt(" #n ")" N "v_readfirstlane_b32 s4, v63" N "s_add_u32 " sCALL ", s4, 1" N "s_lshl_b32 s5, " sCALL ", 24" N "s_and_b32 s5, s5, 0x1000000" N \
    "s_add_u32 s44, s44, s5" N "s_xor_b32 s45, s44, 0x4000" N
template <int V> DI void go() {
  STQ(2); wait<V, -1>(); STQ(3); Y("s_barrier");
  REP(i, 0, 8, rdA<FA0, i, VX, 0>()); REP(i, 0, 12, rdB<FB1, i, VX, 2 * U>()); REP(i, 0, 9, rdB<FB0, i, VZ, 0>());
}
#define TAKE(n) "v_mov_b32 v78, " #n N "buffer_atomic_add v78, v93, " sWS ", s22 offen sc0" N

__global__ __launch_bounds__(256, 1) void a2a_gemm(Params) {
  if (STAMP) { Y("v_mov_b32 v31, 0"); STP(0); }
  asm volatile("s_load_dwordx16 s[52:67], s[0:1], 0" N "s_getreg_b32 s3, hwreg(HW_REG_XCC_ID, 0, 3)" N "s_and_b32 s4, s2, 0xf8" N "s_mov_b32 s15, 0"
    :: "{s[0:1]}"(__builtin_amdgcn_kernarg_segment_ptr()), "{s2}"(blockIdx.x), "{v0}"(threadIdx.x) : "v255", "a255", "s101", "vcc", "memory");
  if (FAKEX) Y("s_bitcmp1_b32 s2, 6" N "s_cselect_b32 s5, 5, 7" N "s_and_b32 s3, s3, s5");
  X("s_or_b32 s3, s3, s4\n.Lstart:" N "s_and_b32 s4, s3, 4" N "s_lshr_b32 s5, s3, 6" N "s_add_u32 s4, s4, s5" N "s_lshl_b32 s4, s4, 5" N
    "s_load_dwordx8 s[88:95], s[0:1], s4 offset:0x40" N "s_getpc_b64 s[16:17]\n.Lpc:" N "v_readfirstlane_b32 s14, v0" N "v_mov_b64 v[60:61], 0" N
    "v_lshrrev_b32 v9, 4, v0" N "s_and_b32 s4, s3, 0xf8" N "s_lshl_b32 s4, s4, 8" N "v_lshl_add_u32 v5, v9, 7, s4" N "v_min_u32 v5, .Lend - .Lpc - 4, v5" N
    "global_load_dword v65, v5, s[16:17]" N "s_bfe_u32 s9, s3, 0x10002" N "s_and_b32 s13, s3, 3" N "s_bfe_u32 s10, s3, 0x20003" N
    "s_xor_b32 s12, s13, s9" N "s_lshr_b32 s5, s3, 5" N "s_lshl3_add_u32 s12, s12, s5" N "s_lshl_b32 s12, s12, 9" N "s_and_b32 s11, s5, 1" N
    "s_lshr_b32 s8, s14, 6" N "s_lshl_b32 s4, s2, 5" N "s_add_u32 s4, s4, s8" N "s_lshl2_add_u32 " sCTR ", s4, %0" N
    "v_and_b32 v1, 63, v0" N "v_lshlrev_b32 v56, 4, v1" N "v_lshrrev_b32 v4, 3, v1" N
    "s_brev_b32 s14, s14" N "s_lshr_b32 s77, s14, 12" N "s_lshr_b32 s4, s14, 8" N
    "v_lshrrev_b32 v3, 4, v1" N "v_bitop3_b32 v2, v1, v3, 7 bitop3:0x6c" N "v_lshl_add_u32 v72, v4, 11, s4" N
    "v_lshl_add_u32 v72, v2, 4, v72" N "v_xor_b32 v5, 64, v72" N "v_add_u32 v73, 0x3c00, v5" N "v_add_u32 v74, 0x7800, v72" N "v_add_u32 v75, 0xb400, v5" N
    "s_lshr_b32 s4, s14, 6" N "s_add_u32 s4, s4, s12" N "v_bfe_u32 v6, v1, 1, 3" N
    "v_xor_b32 v6, v6, v3" N "v_lshl_add_u32 v52, v3, 14, s4" N "v_lshl_add_u32 v52, v6, 5, v52" N
    "v_and_or_b32 v52, v56, 16, v52" N "v_xor_b32 v5, 0x80, v52" N "v_add_u32 v53, 0xfc00, v52" N "v_add_u32 v54, 0x1f800, v5" N "v_add_u32 v55, 0x2f400, v5" N
    "s_lshl1_add_u32 s4, s10, s11" N "s_lshl_b32 s4, s4, 8" N "s_or_b32 s4, s4, s9" N "s_lshl_b32 s78, s4, 10" N "s_mov_b32 m0, s77" N
    "s_xor_b32 s4, s10, s8" N "s_and_b32 s4, s4, 1" N
    "s_lshl_b32 s18, s4, 16" N "s_or_b32 s18, s18, s77" N "s_bitset1_b32 s18, 15" N
    "s_xor_b32 s4, s4, s9" N "s_lshl_b32 s4, s4, 8" N "s_lshl_b32 s5, s9, 23" N "s_or_b32 s80, s4, s5", "n"(WS_CTR));
#define CTR "buffer_load_dword v63, off, " sWS ", " sCTR " sc1"
  X("s_cmp_lg_u32 s8, 2" N "s_cbranch_scc1 .Lall" N
    "s_lshr_b32 s4, s3, 1" N "s_xor_b32 s4, s4, s3" N "s_and_b32 s4, s4, 2" N "s_lshl_b32 s4, s4, 9" N "v_or_b32 v5, s4, v56" N "v_xor_b32 v7, 0x400, v5" N
    "v_add_u32 v5, .Ltab - .Lpc, v5" N "v_add_u32 v7, .Ltab - .Lpc, v7" N "global_load_dwordx4 v[80:83], v5, s[16:17]" N "global_load_dwordx4 v[84:87], v7, s[16:17]" N
    "v_min_u32 v79, 1, v1" N "v_lshlrev_b32 v93, 30, v79" N "s_lshl_b32 s22, s3, 7" N "s_or_b32 s22, s22, s15" N "s_add_u32 s22, s22, %0" N
    "v_add_u32 v89, %1 + 16, v56" N "s_waitcnt lgkmcnt(0)" N CTR N TAKE(1) ".Lall:" N "s_waitcnt lgkmcnt(0)", "n"(WS_ROLE), "n"(LDS_CTRL));
  STQ(0);
  REP(i, 0, 4, fill<0, i>());
  Y("s_mov_b32 m0, s18" N "s_andn2_b32 s47, s94, 7" N "s_or_b32 s80, s80, s47");
  REP(i, 0, 4, fill<1, i>());
  Y("s_xor_b32 m0, s18, 0x10000" N "s_xor_b32 s81, s80, 0x100");
  REP(i, 0, 4, fill<2, i>());
  if (CE) Y(CTR);
  REP(i, 0, 4, zero<i>());
  Y("v_and_b32 v7, 0x70, v56");
  Y("s_bfe_u32 s6, s8, 0x10001" N "s_and_b32 s7, s8, 1" N "s_lshl_b32 s7, s7, 7" N "s_lshl_b32 s4, s6, 13" N
    "v_and_b32 v12, 15, v1" N "v_lshl_add_u32 v32, v6, 4, s4" N "v_lshl_add_u32 v32, v12, 7, v32" N "v_xor_b32 v33, 64, v32" N
    "v_bfe_u32 v5, v1, 2, 3" N "v_lshlrev_b32 v34, 5, v5" N "v_xor_b32 v34, s7, v34" N "v_and_b32 v5, 3, v5" N "v_lshl_add_u32 v34, v5, 8, v34");
  zero<4>();
  Y("v_lshl_add_u32 v34, v3, 11, v34" N "v_and_b32 v14, 3, v1" N "v_lshl_add_u32 v34, v14, 3, v34" N "v_xor_b32 v35, 32, v34" N "v_xor_b32 v36, 64, v34" N "v_xor_b32 v37, 0x60, v34");
  REP(i, 0, 12, X("v_add_u32 v[%0], %1, v[%2]", "n"(VY + i), "n"((i / 6 + 1) * 3 * U), "n"(VX + i % 6)); if constexpr (i == 6) zero<5>());
  X("s_lshl_b32 s4, s8, 12" N "s_add_u32 s4, s4, %0" N "v_lshrrev_b32 v5, 5, v1" N "v_bitop3_b32 v5, v1, v5, 7 bitop3:0x6c" N "v_lshl_add_u32 v24, v12, 7, s4" N
    "v_lshl_add_u32 v24, v5, 4, v24" N "v_and_b32 v5, 1, v3" N "v_lshl_add_u32 v24, v5, 3, v24", "n"(9 * U));
  zero<6>();
  X("v_xor_b32 v25, 32, v24" N "v_xor_b32 v26, 64, v24" N "v_xor_b32 v27, 0x60, v24" N
    "v_bitop3_b32 v5, v1, v4, 7 bitop3:0x6c" N "v_lshl_add_u32 v28, v4, 7, s4" N "v_lshl_add_u32 v28, v5, 4, v28" N
    "s_lshl2_add_u32 s4, s10, s6" N "s_lshl_b32 s4, s4, 20" N "s_lshl_b32 s5, s9, 8" N "s_or_b32 s46, s47, s5" N "s_or_b32 s4, s4, s5" N "s_or_b32 s4, s4, s7" N "s_or_b32 s4, s4, s12" N
    "v_lshl_add_u32 v29, v4, 14, s4" N "v_add_u32 v29, v29, v7" N "v_mov_b32 v68, %0", "n"(LDS_CTRL));
  zero<7>();
  X("v_lshl_add_u32 v50, v4, 7, s77" N "v_lshl_add_u32 v50, v2, 4, v50" N "s_bfm_b64 exec, 8, 24" N "v_add_u32 v50, %0, v50" N "v_add_u32 v56, %0, v56" N "s_mov_b64 exec, -1" N
    "v_xor_b32 v51, 64, v50" N "v_add_u32 v57, 0x2000, v56" N
    "s_lshl2_add_u32 s4, s3, s8" N "s_lshl_b32 s4, s4, 14" N "s_add_u32 " sXS ", s4, %1" N "s_xor_b32 s4, " sXS ", 0x51000" N "s_add_u32 s68, s52, s4" N "s_addc_u32 s69, s53, 0" N
    "s_lshl_b32 s4, s3, 5" N "s_add_u32 s4, s4, s8" N "s_lshl2_add_u32 " sXF ", s4, %2" N "s_xor_b32 " sXP ", " sXF ", 0x280" N
    "s_lshl_b32 " sOF ", s11, 21" N "s_mov_b32 " sT ", 0" N "s_mov_b32 s74, s77" N "s_add_u32 s75, s77, 0xc000" N
    "s_lshl_b32 s21, s10, 19" N "s_lshl_b32 s4, s11, 14" N "s_or_b32 s4, s4, s21" N "s_add_u32 s44, s4, %3", "n"(AWAY), "n"(WS_XCH), "n"(WS_XFLAG), "n"(WS_A));
  STQ(1);
  Y("s_cmp_eq_u32 s8, 2" N "s_cbranch_scc1 .Lc_go" N "s_cmp_eq_u32 s8, 3" N "s_cbranch_scc1 .Lp_go" N "s_branch .Ld_go\n"
    ".p2align 6\n.fill ((.Ld_go - .Ld0) & 63) / 4, 4, 0xbf800000\n.Ld_go:");
  {
    REP(i, 0, 4, X("v_add_u32 v[%0+4], 0x10000, v[%0]" N "v_add_u32 v[%1], 0x40000, v[%2]", "n"(VDA + i), "n"(VDB + 4 + i), "n"(VRB + i)));
    Y("s_xor_b32 s79, s78, 0x40000" N "s_add_u32 s92, s78, 128" N "s_mov_b32 s88, s18" N "s_xor_b32 s89, s18, 0x10000" N
      "s_lshr_b32 s4, s18, 1" N "s_and_b32 s4, s4, 0x8000" N "s_or_b32 s4, s4, s77" N "s_or_b32 s90, s4, 0x14000" N "s_xor_b32 s91, s90, 0x8000" N
      "s_add_u32 s93, s80, 0x100000" N "s_add_u32 s94, s81, 0x100000" N "s_add_u32 s95, s79, 128" N "s_add_u32 s96, s78, 0x100" N
      "s_add_u32 s97, s80, 0x200000" N "s_add_u32 s98, s81, 0x200000");
    Y("s_add_u32 m0, s77, 0x4000" N "s_add_u32 s99, s79, 0x100");
    REP(i, 0, 4, fill<0, i, SA1>());
    Y("s_add_u32 m0, s77, 0x5000" N "s_lshl_b32 s86, s8, 8");
    REP(i, 4, 8, fill<0, i, SA1>());
    Y("s_add_u32 m0, s77, 0xc000" N "s_mov_b32 " sEV ", 6");
    REP(i, 0, 4, fill<0, i, 92>());
    Y("s_add_u32 m0, s77, 0xd000" N "s_mov_b32 " sAH ", 0x40000");
    REP(i, 4, 8, fill<0, i, 92>());
    if (!CE) Y(CTR);
    go<17>(); wait<-1, 13>();
    STP(1);
    stile<4, 0>(); Y("s_cbranch_scc1 .Lexit"); stile<4, 1>();
    SLOOP(0, ".Ld", ".Ld_ev");
    Y(".Ld_ev:" N LEAVE "s_mov_b64 s[60:61], s[52:53]" N "s_mov_b32 s62, s54" N "s_mov_b32 " sAH ", 0x4000");
    REP(i, 0, 8, X("v_add_u32 v[%0], %1, v[%2]", "n"(VDA + i), "n"((i >> 2) * 4096), "n"(VRA + (i & 1))));
    Y("s_branch .Ld0");

    X(".p2align 6\n.fill ((.Lc_go - .Lc0) & 63) / 4, 4, 0xbf800000\n.Lc_go:" N
      "s_lshl_b32 s4, s10, 18" N "s_xor_b32 s5, s12, %[px]" N "s_or_b32 s4, s4, s5" N "v_lshrrev_b32 v5, 2, v1" N "v_lshl_add_u32 v92, v5, 14, s4" N "v_lshl_add_u32 v92, v14, 7, v92" N
      "v_cmp_gt_u32 vcc, %[pn], v1" N "v_bfrev_b32 v90, 1" N "v_and_b32 v94, 15, v1" N "v_lshlrev_b32 v94, 2, v94" N "v_cndmask_b32 v92, v93, v92, vcc" N "v_mov_b32 v72, 0" N
      "s_mov_b32 s88, %[t0]" N "s_mov_b32 s89, %[t1]" N "s_mov_b64 " cK ", %[cm]" N "s_mov_b32 " cCL ", 0x7f000000" N "s_mov_b32 " sEV ", %[tv] - 1" N
      "s_lshl_b32 s4, s10, 10" N "s_add_u32 " cFB ", s4, %[af]" N "s_lshl_b32 s4, s10, 7" N "s_or_b32 s4, s4, s12" N "s_add_u32 " cCM ", s4, %[cl]" N
      "s_andn2_b32 s5, s80, 0x800000" N "v_mov_b32 v6, s5" N "s_waitcnt vmcnt(%[ce])" N CALL(13)
      "v_mov_b32 v5, s44" N "s_mov_b64 exec, 0x7f" N "v_mov_b32 v5, s78" N "v_mov_b32 v6, s80" N "s_mov_b64 exec, -1" N
      "v_and_b32 v76, 15, v83" N "v_lshl_add_u32 v76, s4, 4, v76" N "v_and_b32 v67, 15, v87" N "v_lshl_add_u32 v67, s4, 4, v67" N "s_mul_i32 " cTK ", s4, 2 * %[np]" N "s_add_u32 " cTK ", " cTK ", %[np]",
      [ce] "n"(12 + CE), [px] "n"(PFX * 512), [pn] "n"(64 / PFN), [t0] "n"((unsigned)TODO), [t1] "n"((unsigned)(TODO >> 32)), [cm] "n"(-NOCOMM), [tv] "n"(TV), [af] "n"(WS_AFLAG), [cl] "n"(WS_CLAIM), [np] "n"(N_POOL));
    REP(i, 0, 2, X("v_add_u32 v[%0], v[%0], v5" N "v_add_u32 v[%0+1], v[%0+1], v6" N "v_add_u32 v[%0+3], s47, v[%0+3]" N
      "v_and_b32 v[%0+1], 0x7ffffff, v[%0+1]" N "v_and_b32 v[%0+3], 0x87ff0000, v[%0+3]", "n"(80 + i * 4)));
    Y("v_readfirstlane_b32 s5, v78" N "s_cmp_eq_u32 s15, 0" N "s_cselect_b32 s5, s5, s4" N "s_cmp_lg_u32 s5, s4" N "s_cbranch_scc1 .Lc_taken\n.Lc_mine:");
    go<CE>();
    X("v_mov_b32 v78, 1" N "v_mov_b32 v69, 1" N "v_mov_b32 v70, v90" N "v_mov_b32 v71, v90" N "s_lshl_b64 exec, %0, %2" N "v_mov_b32 v70, 0" N "s_lshl_b64 exec, %1, %2" N "v_mov_b32 v71, 0" N "s_mov_b64 exec, -1",
      "n"(DRA), "n"(DRB), "n"(DR0));
    wait<-1, 13>();
    STP(1);
    SLOOP(1, ".Lc", ".Lc_ev");
#define VOTES "buffer_load_dword v95, v94, " sWS ", 0 offen offset:%[vo] sc1" N
    if (EVF) X(".Lc_ev:" N LEAVE "s_lshl_b32 " cB ", " sCALL ", 1" N "s_cmp_eq_u32 " sT ", %[tv]" N "s_cbranch_scc1 3f" N VOTES "s_mov_b32 " sEV ", %[tv]" N "s_mov_b32 " sT0 ", 0" N "s_branch .Lc0\n"
      "3:" N "v_cmp_le_u32 vcc, " cB ", v95" N "v_cmp_lt_u32 s[86:87], " cB ", v95" N "s_cmp_eq_u64 vcc, -1" N "s_cbranch_scc0 2f\n"
      "4:" N "s_bcnt1_i32_b64 " cB ", s[86:87]" N "s_cmp_gt_u32 " cB ", %[th]" N "s_cbranch_scc0 .Lc1" N
      "v_mov_b64 v[82:83], v[86:87]" N "v_mov_b64 v[80:81], v[84:85]" N "v_mov_b32 v76, v67" N "s_mov_b64 " cK ", %[cm]" N SAMPLE N "s_branch .Lc1\n"
      "2:" N VOTES EXPIRED "s_cbranch_scc0 3b" N "s_branch 4b",
      [tv] "n"(TV), [vo] "n"(WS_VOTE), [cm] "n"(-NOCOMM), [th] "n"(VTH), DL);
    else
    X(".Lc_ev:" N LEAVE "s_cmp_eq_u32 " sT ", %[tv]" N "s_cbranch_scc1 2f" N VOTES "s_mov_b32 " sEV ", %[tv]" N "s_branch .Lc0\n"
      "2:" N "s_mov_b32 " sT0 ", 0" N "s_lshl_b32 " cB ", " sCALL ", 1\n"
      "3:" N "v_cmp_le_u32 vcc, " cB ", v95" N "s_cmp_eq_u64 vcc, -1" N "s_cbranch_scc1 4f" N VOTES EXPIRED "s_cbranch_scc0 3b\n"
      "4:" N "v_cmp_lt_u32 vcc, " cB ", v95" N "s_bcnt1_i32_b64 " cB ", vcc" N "s_cmp_gt_u32 " cB ", %[th]" N "s_cbranch_scc0 .Lc1" N
      "v_mov_b64 v[82:83], v[86:87]" N "v_mov_b64 v[80:81], v[84:85]" N "v_mov_b32 v76, v67" N "s_mov_b64 " cK ", %[cm]" N SAMPLE N "s_branch .Lc1",
      [tv] "n"(TV), [vo] "n"(WS_VOTE), [cm] "n"(-NOCOMM), [th] "n"(VTH), DL);
    static_assert(TV == 5, "");
    Y(".Lc_slow:" N "s_mov_b32 " sT0 ", 0\n.Lc_again:");
    CNT(10, 1);
    X(SAMPLE N EXPIRED "s_cbranch_scc0 5f" N "v_mov_b32 v76, 0\n5:" N ARRIVED N "s_cmp_eq_u64 " cK ", -2" N "s_cselect_b64 " cC ", 63, 62" N "s_or_b64 " cC ", " cC ", " cK N
      "s_and_b64 " cC ", " cC ", " cT N "s_cbranch_scc0 .Lc_again" N BACK(".Lc_back"), DL);
    Y(".Lc_taken:" N TAKE(-1) "s_add_u32 s22, s22, 128" N "s_bitset0_b32 s22, 15" N "s_waitcnt vmcnt(0)" N TAKE(1) "s_waitcnt vmcnt(0)" N "v_readfirstlane_b32 s5, v78" N
      "s_cmp_lg_u32 s5, s4" N "s_cbranch_scc1 .Lc_taken" N "v_mov_b32 v79, s22" N "s_branch .Lc_mine");

    Y(".p2align 6\n.fill ((.Lp_go - .Lpi0) & 63) / 4, 4, 0xbf800000\n.Lp_go:");
    if (!CE) Y(CTR);
    X("s_lshl2_add_u32 s20, s10, s13" N "s_lshl_b32 s4, s20, 17" N "v_lshl_add_u32 v88, v4, 11, s4" N "v_add_u32 v88, v88, v7" N
      "s_and_b32 s4, s3, 0xc7" N "s_cmp_lg_u32 s4, %[sw]" N "s_addc_u32 s19, s11, 0" N "s_xor_b32 s4, s19, s94" N "s_bitcmp1_b32 s4, 0" N "s_cselect_b32 s4, 0x100, 0" N
      "s_mov_b64 s[16:17], s[88:89]" N "s_add_u32 s86, s88, s4" N "s_addc_u32 s87, s89, 0" N
      "s_and_b32 s19, s19, 1" N "s_cselect_b32 " pM ", 0, 0xfffc" N "s_xor_b32 " pM ", " pM ", %[em] - 2" N
      "s_lshl_b32 s4, s10, 6" N "s_or_b32 s4, s4, s19" N "s_xor_b32 s4, s4, 1" N "s_lshl2_add_u32 s4, s4, s13" N "s_lshl2_add_u32 s92, s4, s92" N "s_addc_u32 s93, s93, 0" N
      "s_lshl_b32 s4, s13, 13" N "s_or_b32 s7, s21, s4" N
      "s_and_b32 s6, s94, 7" N "s_mul_i32 s4, s20, 7" N "s_add_u32 s4, s4, s6" N "s_mul_i32 s4, s4, %[pr]" N "s_lshr_b32 s4, s4, 16" N "s_sub_i32 " pS ", s4, %[sh]" N
      "s_mul_i32 s4, s19, %[pe]" N "s_add_i32 s4, s4, " pS N "s_max_i32 s4, s4, 0" N "s_lshr_b32 s4, s4, 6" N "s_lshl_b32 " sEV ", s4, 1" N
      "s_lshl_b32 s14, s20, 2" N "s_bitcmp1_b32 s6, 0" N "s_cselect_b32 s4, 0x22222222, 0" N "s_xor_b32 s20, s4, 0x75643120" N "s_xor_b32 s21, s4, 0xfdecb9a8" N
      NEXT0 N NEXT1 N NEXT2 N NEXT3 N NEXT4 N
      "s_mov_b32 s94, 0" N "s_mov_b32 " pB ", 64" N "s_mov_b64 vcc, -1" N "s_lshl_b32 s19, s19, 15" N
      "s_cmp_lt_u32 s6, %[end]" N "s_cbranch_scc1 1f" N
      "v_mov_b32 v86, s14" N "s_lshl_b32 s4, s14, 7" N "s_or_b32 s4, s4, s47" N "s_xor_b32 s4, s4, 0x4800000" N
      "v_lshl_add_u32 v84, v14, 14, s4" N "v_add_u32 v85, 0x2000, v84" N "s_xor_b32 s4, s4, 0x200" N "v_and_b32 v5, 31, v1" N "v_lshl_add_u32 v83, v5, 14, s4" N
      "v_lshrrev_b32 v5, 5, v1" N "v_lshl_add_u32 v83, v5, 13, v83" N
      "s_add_u32 s6, s6, s11" N "s_cmp_eq_u32 s6, 8" N "s_cselect_b32 " sEV ", %[tp], -1" N "s_mov_b64 vcc, 0\n1:",
      [sw] "n"(SWP), [pr] "n"(PHASE), [pe] "n"(PER), [em] "n"(EM), [sh] "n"(SHIFT), [end] "n"(NOCOMM ? 0 : 7), [tp] "n"(TP));
    REP(i, 1, 8, X("v_add_u32 v[%0], %1, v88", "n"(88 + i), "n"(i * 16384)));
    if (XL == 2) go<1>();
    REP(i, 0, 8, X("global_load_dwordx4 v[%0:%0+3], v[%1], s[86:87]", "n"(96 + i * 4), "n"(88 + i)));
    if (XL == 1) go<9>();
    X(CALL(8) "s_add_u32 s5, s5, s7" N "s_add_u32 s22, s90, s5" N "s_addc_u32 s23, s91, 0" N "s_add_u32 s90, s22, s19" N "s_addc_u32 s91, s23, 0" N
      "s_lshl_b32 s4, s4, 4" N "v_mov_b32 v81, s4" N
      "s_cmp_eq_u32 " sEV ", %0" N "s_cselect_b64 exec, -1, 0" N "global_load_dword v82, v83, s[56:57]" N "s_mov_b64 exec, -1", "n"(TP));
    wait<-1, 13>();
    STP(1);
    Y("s_cmp_lg_u32 " sEV ", 0" N "s_cbranch_scc1 .Lpi0" N "s_waitcnt vmcnt(0)\n.Lpp0:");
    stile<3, 0>(); Y("s_cbranch_scc1 .Lp_ev"); stile<3, 1>(); Y("s_cbranch_scc1 .Lpp0");
    Y(".Lpi0:"); stile<2, 0>(); Y("s_cbranch_scc1 .Lp_ev\n.Lpi1:"); stile<2, 1>();
    X("s_cbranch_scc0 .Lpi0" N "s_cbranch_vccnz .Lpp0\n.Lp_ev:" N LEAVE
      "s_mov_b32 s98, 0" N "s_mov_b32 s97, 2\n"
      "4:" N "s_mov_b64 exec, vcc" N "global_load_dword v82, v84, s[56:57]" N "global_load_dword v83, v85, s[56:57]" N "s_waitcnt vmcnt(1)" N "s_memtime s[88:89]" N
      "s_waitcnt vmcnt(0)" N "s_memtime s[90:91]" N "v_swap_b32 v84, v85" N "v_add_u32 v84, %0, v84" N "v_add_u32 v85, %0, v85" N "s_mov_b64 exec, -1" N "s_waitcnt lgkmcnt(0)" N
      "s_sub_u32 s88, s90, s88" N "s_min_i32 s88, s88, 400" N "s_and_b32 s88, s88, vcc_lo" N "s_sub_i32 s98, s88, s98" N "s_mov_b64 vcc, -1" N
      "s_sub_u32 s97, s97, 1" N "s_cbranch_scc0 4b" N
      "s_getreg_b32 s88, hwreg(HW_REG_XCC_ID, 2, 1)" N "s_lshr_b32 s98, s98, 31" N "s_mov_b64 vcc, 0" N "s_xor_b32 s88, s88, s98" N "s_xor_b32 s88, s88, 1" N "s_lshl1_add_u32 s88, " sCALL ", s88" N
      "v_mov_b32 v82, s88" N "buffer_store_dword v82, v86, " sWS ", 0 offen offset:%1 sc1" N BACK(".Lpi"), "n"(4 * 16384), "n"(WS_VOTE));

    X(".Lx_slow:" N "buffer_load_dword v61, off, " sWS ", " sXF " sc0 sc1" N EXPIRED "s_cselect_b32 " sMORE ", " sCALL ", " sMORE N "s_cbranch_scc1 .Lx_back" N
      "v_readfirstlane_b32 s84, v61" N "s_cmp_lt_u32 s84, " sCALL N "s_cbranch_scc1 .Lx_slow", DL);
    CNT(14, 1);
    Y("s_branch .Lx_back");

    Y(".Lexit:" N "s_cmp_lg_u32 " sMORE ", 0" N "s_cbranch_scc1 .Lagain" N "s_bitcmp1_b32 " sT ", 0" N "s_cbranch_scc0 .Lt" N
      "s_mov_b32 s84, s74" N "s_mov_b32 s74, s75" N "s_mov_b32 s75, s84");
    REP(i, 0, 6, X("v_swap_b32 v[%0], v[%1]", "n"(VX + i), "n"(VY + i)));
    REP(i, 0, 4, emit<ZB, i, 0, 0, 0>());
    Y(".Lt:");
    STP(2);
    REP(q, 0, NX / 16, REP(j, 0, 16, constexpr int g = q * 16 + j;
      if constexpr (g < NG) { constexpr Op f = PLAN.mf[g < NG ? g : 0]; mfma<f.k, f.a, f.b, f.c>(); }
      if constexpr (g == NG) STP(3);
      REP(i, 0, PLAN.n[g], constexpr Op o = PLAN.op[g][i]; emit<o.k, o.a, o.b, o.c, o.d>())));
    Y("s_cmp_eq_u32 " sMORE ", " sCALL N "s_cbranch_scc0 .Lfin" N "s_waitcnt vmcnt(0)" N "v_mov_b32 v0, 0x7fc07fc0");
    REP(i, 0, 4, X("v_accvgpr_write_b32 a[%0], v0", "n"(FO + i)));
    REP(i, 0, 16, emit<OU, i / 2, 5, i % 2, 1>());
    Y(".Lfin:" N "v_mov_b32 v0, " sCALL N "buffer_store_dword v0, off, " sWS ", " sCTR " sc1");
    if (STAMP) {
      Y("s_waitcnt vmcnt(0)"); STP(4);
      Y("s_getreg_b32 s84, hwreg(HW_REG_XCC_ID, 0, 4)" N "v_writelane_b32 v31, s84, 30" N "v_writelane_b32 v31, " sT ", 24" N "v_mbcnt_lo_u32_b32 v0, -1, 0" N "v_mbcnt_hi_u32_b32 v0, -1, v0" N "v_lshlrev_b32 v0, 2, v0" N
        "s_and_b32 s84, " sCTR ", 15" N "s_lshr_b32 s84, 0x7310, s84" N "s_and_b32 s84, s84, 7" N "s_lshl_b32 s84, s84, 7" N "s_lshr_b32 s85, " sCTR ", 7" N "s_sub_u32 s85, s85, 128" N
        "s_lshl_b32 s86, s85, 10" N "s_add_u32 s84, s84, s86" N "s_add_u32 s84, s84, 0x6500000" N "s_mov_b32 exec_hi, 0" N "buffer_store_dword v31, v0, " sWS ", s84 offen" N "s_mov_b64 exec, -1");
      if (STAMP == 2) Y("s_and_b32 s84, " sCTR ", 15" N "s_cmp_eq_u32 s84, 8" N "s_cbranch_scc0 1f" N "s_lshl_b32 s85, s85, 9" N "s_add_u32 s85, s85, 0x6600000" N "v_writelane_b32 v30, s100, 63" N
        "buffer_store_dword v59, v0, " sWS ", s85 offen" N "buffer_store_dword v30, v0, " sWS ", s85 offen offset:256\n1:");
    }
    Y("s_endpgm\n.Lagain:" N "s_waitcnt vmcnt(0) lgkmcnt(0)" N "s_barrier" N "s_bfe_u32 s3, " sMORE ", 0x80007" N "s_brev_b32 s15, 4" N "s_branch .Lstart\n.p2align 7\n.Ltab:");
    REP(q, 0, 2, REP(l, 0, 64, X(".long %0, %1, %2, %3", "n"(table(q, l, 0)), "n"(table(q, l, 1)), "n"(table(q, l, 2)), "n"(table(q, l, 3)))));
    Y(".Lend:");
  }
}

extern "C" void kernel_launch(int rank, int nranks, ncclComm_t comm,
                              hipStream_t stream,
                              void* const* d_in, const int* in_sizes, int n_in,
                              void* d_out, int out_size,
                              const int* in_dtypes, int out_dtype,
                              void* const* d_out_all,
                              void* const* d_ws_all, size_t ws_bytes) {
  if (nranks != NRANK || (unsigned)rank >= (unsigned)NRANK || n_in < 2 || ws_bytes < WS_NEED ||
      in_sizes[0] != 8192 * 1024 || in_sizes[1] != 8192 * 8192 || out_size != 1024 * 8192) {
    fprintf(stderr, "a2a_gemm: unexpected problem shape\n");
    return;
  }
  HIP_CHECK(hipFuncSetAttribute((const void*)a2a_gemm, hipFuncAttributeMaxDynamicSharedMemorySize, LDS_BYTES));
  const char* x = (const char*)d_in[0];
  char* ws = (char*)d_ws_all[rank];
  Params p = {{ws, (unsigned)WS_NEED, 0x20000}, {d_in[1], 8192 * 16384, 0x20000}, {x + rank * BLK_B, (unsigned)BLK_B, 0x20000}, {d_out, 1024 * 16384, 0x20000}, {}};
  p.duty[3] = {x, ws, ws, (unsigned)rank << 24 | 7, 0x20000};
  for (int j = 0, n[2] = {0, 4}; j < 7; ++j) {
    const int q = (rank + 1 + j) & 7, e = n[((0xb2 >> rank) ^ (0xb2 >> q)) & 1]++;
    char* to = (char*)d_ws_all[q];
    p.duty[e] = {x + q * BLK_B, to + WS_A + (6 - j) * BLK_B + 4096, to + WS_AFLAG + (6 - j) * 128, (unsigned)rank << 24 | j, 0x20000};
  }
  a2a_gemm<<<256, 256, LDS_BYTES, stream>>>(p);
  HIP_CHECK(hipGetLastError());
}
```

```cpp
#include <hip/hip_runtime.h>
#include <rccl/rccl.h>
#include <cstdio>

#define HIP_CHECK(expr) do { hipError_t _e = (expr); if (_e != hipSuccess) { \
    fprintf(stderr, "HIP error %s at %s:%d\n", hipGetErrorString(_e), __FILE__, __LINE__); \
    return; } } while (0)

typedef unsigned long long u64;

#define W4 1
#define STAMP 0
#define NOCOMM 0
#define FAKEX 0
#define PER 80
#define SHIFT 28
#define FPQ 5
#define TV 5
#define TP 1
#define RL 11
#define AB 3
#define FG 291
#define LAGF 72
#define LAGP 80
#define PFR 44
#define PFS 1
#define PFN 1
#define DR0 46
#define DRA 0x1b
#define DRB 0x16
#define EM 0x4a52
#define SWP 0xc4
#define VTH 40
#define XL 1
#define CE 1
#define EVF 1
#define TCH 0
#define EB 2
#define PFX 1
#define OST " sc1"
#define XST " sc0 sc1"
#define XQ 16

constexpr int NRANK = 8;
constexpr int NCOL = 8192;
constexpr int U = 16384;
constexpr int LDS_BYTES = 10 * U;
constexpr int LDS_CTRL = 9 * U + 1024;
constexpr size_t BLK_B = (size_t)1024 * 1024 * 2;
constexpr int N_POOL = 7;
constexpr int N_X = (RL - 7) / 2;
constexpr unsigned XSRC = N_X == 1 ? 6u : N_X == 4 ? 0x6614u : 0x14614u;
static_assert(PFR >= TV && PFR + ((7 + N_X) * PFN - 1) * PFS <= 54 - N_X, "");
static_assert(__builtin_popcount(DRA) + __builtin_popcount(DRB) == N_POOL && ((u64)(DRA | DRB) << DR0) >> (56 - 3 - N_X) == 0, "the tickets come in time");
static_assert(RL >= 7 && RL <= 17 && (RL & 1), "");
constexpr unsigned RROW = 128, RCOL = 32768;
constexpr int WS_TMO = 0;
constexpr int WS_VOTE = 256;
constexpr size_t WS_CTR = 16384;
constexpr size_t WS_CLAIM = 49152;
constexpr size_t WS_XFLAG = 65536;
constexpr size_t WS_AFLAG = 131072;
constexpr size_t WS_ROLE = 196608;
constexpr size_t WS_A = (size_t)1 << 20;
constexpr size_t WS_XCH = (size_t)34 << 20;
constexpr unsigned AWAY = (50u << 20) + 128;
constexpr size_t WS_DBG = (size_t)101 << 20;
constexpr size_t WS_NEED = (size_t)103 << 20;
constexpr unsigned DEADLINE = 50000000u;

constexpr int event(int s, int k, int p) { return 2 * k + (p ^ (s & 1)); }
constexpr unsigned sender(int e) { return (EM >> e & 1 ? EM : ~EM) & 0xffff; }
constexpr unsigned offA(int s, int k, int p) { return s * BLK_B + event(s, k, p) * RCOL; }
constexpr unsigned offW(int s, int k, int p) { return ((1 + s) * 16 + k / 2 * 4 + p * 2 + k % 2) << 20; }
constexpr int rsrc(int r) { return r < 7 ? r : (XSRC >> (r - 7) / 2 * 4) & 7; }
constexpr int rcol(int r) { return r < 7 ? 7 : (0x55666 >> (r - 7) / 2 * 4) & 7; }
constexpr unsigned table(int q, int l, int i) {
  const int k = l / 7, o = q ? 6 - l % 7 : l % 7, w = l < 7 || l == 63, s = w ? (l + 6) % 7 : (o * 2) % 7, p = w ? ~s & 1 : k < 8 ? q : s & 1, own = l < 5 ? l + 3 : 3;
  const int z = (l - PFR) / PFS, y = z / PFN, r = y < 7 ? y : 2 * y - 7;
  const int e = event(s, w ? 7 : k - 1, p);
  return i == 0 ? (w ? own * 128 : offA(s, k - 1, p)) : i == 1 ? (w ? own << 20 : offW(s, k - 1, p)) : i == 2 ? (l ? s * 128 + (sender(e) & 1) * 16 : 1u << 31)
       : (l >= PFR && z * PFS + PFR == l && r < RL ? offW(rsrc(r), rcol(r), q) + z % PFN * (0x40000 / PFN) : 1u << 31) | __builtin_popcount(sender(e) & ((2 << e) - 1));
}
constexpr u64 TODO = 0x00ffffffffffffbeull & ~((N_X & 1 ? 1ull << 52 : 0) | (N_X >= 2 ? 5ull << 51 : 0) | (N_X == 4 ? 0x81ull << 45 : 0) | (N_X == 5 ? 5ull << 44 : 0));
constexpr int PHASE = PER * 65536 / 112 + 1;
constexpr bool live() {
  for (int q = 0; q < 2; ++q) for (int m = 0; m < 4; ++m) {
    int n[99] = {};
    for (int l = 1; l < 64; ++l) { const int s = table(q, l, 2) / 128, c = table(q, l, 3) & 15, r = table(q, l, 2) / 16 & 1; int t = -2;
      for (int e = 0, i = 0; i < c; ++e) if ((~EM >> e & 1) == r) { const int u = ((((m * 4 + 3) * 7 + 6 - s) * PHASE >> 16) - SHIFT + e * PER) >> 5 & -2; t = u > t + 2 ? u : t + 2; ++i; }
      if (t + 3 > 54 - N_X) return false;
      n[t + 3] += l > 6 && (TODO >> l & 1); }
    for (int t = 1; t < 54 - N_X; ++t) if ((n[t] += n[t - 1]) < t - 4) return false; }
  return (EM & 3) == 2 && (EM >> 14 == 1 || EM >> 14 == 2);
}
static_assert(live(), "");

struct Srd { const void* base; unsigned bytes, mode; };
struct Duty { const char* x; char* to; char* flag; unsigned who, mode; };
struct Params {
  Srd ws, w, x, out;
  Duty duty[8];
};

constexpr int FA0 = 128, FA1 = 160, FB0 = 192, FB1 = 224, FA2 = 0, FA3 = 32, FO = 64, VX = 32, VY = 38, VZ = 44, VRA = 50, VRB = 52, VDA = 72, VDB = 80;
constexpr int SWS = 52, SW = 56, SA = 60, SRA = 44, SLX = 74, SLW = 77, SA0 = 78, SA1 = 79, SB1 = 80, SB0 = 81;
#define sT "s48"
#define sCALL "s49"
#define sMORE "s50"
#define sEV "s51"
#define sWS "s[52:55]"
#define sOUT "s[64:67]"
#define sXD "s[68:69]"
#define sXS "s70"
#define sXF "s71"
#define sXP "s72"
#define sCTR "s73"
#define sAH "s82"
#define sOF "s83"
#define cT "s[88:89]"
#define cK "s[90:91]"
#define cC "s[92:93]"
#define cB "s94"
#define cCL "s95"
#define cTK "s96"
#define cFB "s97"
#define cCM "s98"
#define sT0 "s99"
#define pXN "s[88:89]"
#define pD "s[90:91]"
#define pF "s[92:95]"
#define pB "s96"
#define pE "s97"
#define pS "s98"
#define pM "s99"

#define DI __device__ __forceinline__
#define X(s, ...) asm volatile(s :: __VA_ARGS__ : "memory")
#define Y(s) asm volatile(s ::: "memory")
#define N "\n\t"
template <int V> struct IC { static constexpr int value = V; };
template <int I, int E, class F> DI void rep(F f) { if constexpr (I < E) { f(IC<I>{}); rep<I + 1, E>(f); } }
#define REP(i, a, b, ...) rep<a, b>([](auto i##_) __attribute__((always_inline)) { constexpr int i = decltype(i##_)::value; __VA_ARGS__; })

template <int KEEP, int T, int FB, int FA> DI void mfma() {
  if constexpr (KEEP) X("v_mfma_f32_16x16x32_bf16 v[%0:%0+3], a[%1:%1+3], a[%2:%2+3], v[%0:%0+3]", "n"(128 + T * 4), "n"(FB), "n"(FA));
  else X("v_mfma_f32_16x16x32_bf16 a[%0:%0+3], a[%1:%1+3], a[%2:%2+3], a[%0:%0+3]", "n"(T * 4), "n"(FB), "n"(FA));
}
template <int F, int i, int S, int O> DI void rdA() { X("ds_read_b128 a[%0:%0+3], v[%1] offset:%2", "n"(F + i * 4), "n"(S + (i & 1)), "n"(O + (i >> 1) * 2048)); }
template <int F, int i, int S, int O> DI void rdB() { X("ds_read_b64_tr_b16 a[%0:%0+1], v[%1] offset:%2", "n"(F + i * 2), "n"(S + 2 + (i >> 2)), "n"(O + ((i >> 1) & 1) * 8192 + (i & 1) * 1024)); }
template <int S, int O> DI void m0() { X("s_add_u32 m0, s[%0], %1", "n"(S), "n"(O)); }
template <int V, int R, int SO, int i> DI void dmaA() { X("buffer_load_dwordx4 v[%0], s[%1:%1+3], s[%2] offen offset:%3 sc0 sc1 lds", "n"(V), "n"(R), "n"(SO), "n"((i & 3) * 1024)); }
template <int V, int SO, int i> DI void dmaB() { X("buffer_load_dwordx4 v[%0], s[56:59], s[%1] offen offset:%2 lds", "n"(V), "n"(SO), "n"((i & 3) * 1024)); }
template <int V, int L> DI void wait() {
  if constexpr (V < 0) X("s_waitcnt lgkmcnt(%0)", "n"(L)); else if constexpr (L < 0) X("s_waitcnt vmcnt(%0)", "n"(V)); else X("s_waitcnt vmcnt(%0) lgkmcnt(%1)", "n"(V), "n"(L));
}
template <int P, int Q, int i> DI void fm0() { if constexpr (Q == 0 || Q == 3) m0<SLW, (P * 3 + Q / 3) * U + (i >> 2) * 4096>(); else m0<87 + P * 2 + Q, (i >> 2) * 4096>(); }
template <int Q, int i, int SO = Q == 3 ? SA1 : Q ? SA1 + Q : SA0> DI void fill() { if constexpr (Q == 0 || Q == 3) dmaA<VDA + i, SA, SO, i>(); else dmaB<(i < 4 ? VRB : VDB) + i, SO, i>(); }
constexpr int early(int P, int Q, int g) {
  int n = P * 44 + Q * 12;
  for (int x = 0; x < 32; ++x) { const bool y = x % 4 == 1 || (Q == 0 ? x >= 15 && x < 30 && x % 2 : Q < 3 && x % 8 == 3); if (x == g) return y ? n : -1; n += y; }
  return -1;
}
#define STP(i) do { if (STAMP) X("s_memrealtime s[84:85]" N "s_memtime s[100:101]" N "s_waitcnt lgkmcnt(0)" N "v_writelane_b32 v31, s84, %0" N "v_writelane_b32 v31, s85, %0+1" N \
    "v_writelane_b32 v31, s100, %0+10" N "v_writelane_b32 v31, s101, %0+11", "n"(2 * (i))); } while (0)
#define STQ(i) do { if (STAMP == 3) X("s_memrealtime s[84:85]" N "s_waitcnt lgkmcnt(0)" N "v_writelane_b32 v31, s84, %0", "n"(20 + (i))); } while (0)
#define CNT(lane, n) do { if (STAMP && STAMP != 3) X("v_readlane_b32 s84, v31, %0" N "s_add_u32 s84, s84, %1" N "v_writelane_b32 v31, s84, %0", "n"(2 * (lane)), "n"(n)); } while (0)
#define SAMPLE "buffer_load_dwordx4 v[72:75], v82, " sWS ", " cFB " offen sc0 sc1"
#define ARRIVED "v_min3_u32 v77, v72, v73, v74" N "v_min_u32 v77, v77, v75" N "v_cmp_le_u32 vcc, v76, v77" N "s_or_b64 " cK ", " cK ", vcc"
#define EXPIRED "buffer_load_dword v60, off, " sWS ", 0 sc1" N "s_memrealtime s[84:85]" N "s_sleep 1" N "s_waitcnt vmcnt(0) lgkmcnt(0)" N "v_readfirstlane_b32 s85, v60" N \
    "s_or_b32 s84, s84, 1" N "s_cmp_eq_u32 " sT0 ", 0" N "s_cselect_b32 " sT0 ", s84, " sT0 N "s_sub_u32 s84, s84, " sT0 N \
    "s_cmp_eq_u32 s85, " sCALL N "s_cselect_b32 s84, -1, s84" N "s_cmp_gt_u32 s84, %[dl]" N "s_cbranch_scc0 1f" N \
    "v_mov_b32 v60, " sCALL N "buffer_store_dword v60, off, " sWS ", 0 sc1\n1:" N
#define DL [dl] "n"(DEADLINE)
#define NEXT0 "s_ff1_i32_b32 " pE ", " pM N "s_bitset0_b32 " pM ", " pE
#define NEXT1 "s_lshl_b32 s84, " pE ", 2"
#define NEXT2 "s_lshr_b64 s[84:85], s[20:21], s84"
#define NEXT3 "s_and_b32 s84, s84, 15" N "s_lshl_b32 s84, s84, 7"
#define NEXT4 "s_add_u32 s88, s16, s84" N "s_addc_u32 s89, s17, 0"

constexpr int nth(int g) { return (g & 3) == 1 ? -99 : g - (g + 2) / 4; }
constexpr int sb(int g) { return ((g >> 1) & 3) ^ (g & 8 ? 3 : 0); }
constexpr int sk(int g) { return (g ^ (g >> 1)) & 1; }
template <int ROLE, int P> DI void stile() {
  REP(Q, 0, 4, REP(g, 0, 32,
    constexpr int cur = P ? VY : VX, nxt = P ? VX : VY, keep = Q == 0 || Q == 3, a = g >> 3, m = sb(g), k = g & 1, r = nth(g), y = nth(g & 7);
    mfma<keep, (Q >> 1) * 16 + m * 4 + a, (keep ? FB1 : FB0) + (a * 2 + sk(g)) * 4, (Q >> 1 ? FA1 : FA0) + (m * 2 + sk(g)) * 4>();
    if constexpr (Q == 0 && r >= 0 && r < 4) rdB<FB1, 12 + r, cur, 2 * U>();
    if constexpr (Q == 0 && r >= 4 && r < 11) rdB<FB0, 5 + r, VZ, P * U>();
    if constexpr (Q == 1 && g < 16 && !k) rdA<FA1, g / 2, cur, U>();
    if constexpr (Q == 2 && g < 16 && !k) rdA<FA0, g / 2, nxt, 0>();
    if constexpr (Q == 3 && a > 0 && y >= 0 && y < 4) rdB<FB1, (a - 1) * 4 + y, nxt, 2 * U>();
    if constexpr (Q == 3 && a < 3 && y >= (a ? 4 : 1)) rdB<FB0, a * 2 + y - 1, VZ, (P ^ 1) * U>();
    if constexpr (Q == 0 && g == 15) wait<-1, 7>();
    if constexpr (Q == 3 && g == 0) { if constexpr (ROLE % 4) Y("ds_read_b32 v66, v68 offset:8"); else Y("ds_read_b96 v[64:66], v68"); }
    if constexpr (Q == 3 && g == 8) wait<-1, 6>();
    if constexpr (Q == 3 && g == 10) Y("v_readfirstlane_b32 " sMORE ", v66");
    if constexpr (ROLE % 4 == 0) { constexpr int e = early(P, Q, g), f = early(P, Q, g + 1);
      if constexpr (!ROLE && g % 16 == 0) fm0<P, Q, g / 4>();
      if constexpr (!ROLE && g % 4 == 1) fill<Q, g / 4>();
      if constexpr (ROLE && f >= 0 && f % 4 == 0) fm0<(f / 8 + 5) / 4 % 2, (f / 8 + 1) % 4, f % 8>();
      if constexpr (ROLE && e >= 0) { if constexpr (e < 56) fill<(e / 8 + 1) % 4, e % 8, 93 + e / 8>(); else fill<(e / 8 + 1) % 4, e % 8>(); }
      if constexpr (ROLE && P && Q == 1 && g == 18) Y("v_readfirstlane_b32 s4, v63");
      if constexpr (ROLE && P && Q == 1 && g == 22) Y("s_add_u32 " sCALL ", s4, 1" N "s_lshl_b32 s5, " sCALL ", 24");
      if constexpr (ROLE && P && Q == 1 && g == 23) Y("s_and_b32 s5, s5, 0x1000000" N "s_add_u32 s44, s44, s5");
      if constexpr (ROLE && P && Q == 1 && g == 26) Y("s_xor_b32 s45, s44, 0x4000");
      if constexpr (Q == 0 && g == 22) Y("s_xor_b32 s79, s78, " sAH);
      if constexpr (Q == 3 && g == 11) Y("v_readfirstlane_b32 s78, v64");
      if constexpr (Q == 3 && g == 12) Y("v_readfirstlane_b32 s80, v65");
      if constexpr (Q == 3 && g == 14) Y("s_xor_b32 s80, s80, s86");
      if constexpr (Q == 3 && g == 15) Y("s_xor_b32 s81, s80, 0x100");
    }
    if constexpr (ROLE == 1 && Q == 1 && g == 17) Y("v_readlane_b32 s87, v83, " sT);
    if constexpr (ROLE == 1 && Q == 1 && g == 23) Y("buffer_load_dword v91, v92, s[56:59], s87 offen");
    if constexpr (ROLE == 1 && !(Q & 1) && g == 26) Y("s_lshl_b64 exec, 1, " sT);
    if constexpr (ROLE == 1 && !(Q & 1) && g == 27) X("buffer_atomic_add v[%0], v[%1], " sWS ", " cCM " offen sc0" N "s_mov_b64 exec, -1", "n"(Q ? 69 : 78), "n"(70 + Q / 2));
    if constexpr (ROLE == 1 && Q == 1) {
      if constexpr (g == 24) Y("v_cmp_le_u32 vcc, " cTK ", v78" N "v_cmp_le_u32 s[84:85], " cTK ", v69");
      if constexpr (g == 26) Y("s_bcnt1_i32_b64 s86, vcc" N "s_bcnt1_i32_b64 s84, s[84:85]");
      if constexpr (g == 27) Y("s_lshr_b32 s86, " cCL ", s86" N "s_lshr_b32 s86, s86, s84");
      if constexpr (g == 28) Y("s_and_b32 s93, s86, " cCL N "s_cselect_b32 s92, 0, 1");
    }
    if constexpr (ROLE == 1 && Q == 2) {
      if constexpr (g == 7) { wait<2, -1>(); Y("v_min3_u32 v77, v72, v73, v74"); }
      if constexpr (g == 8) Y("v_min_u32 v77, v77, v75");
      if constexpr (g == 9) Y("v_cmp_le_u32 vcc, v76, v77");
      if constexpr (g == 11) Y("s_or_b64 " cK ", " cK ", vcc" N "s_cmp_eq_u64 " cT ", 0");
      if constexpr (g == 13) Y("s_cselect_b64 " cT ", " cC ", " cT N "s_cselect_b32 " cCL ", 0, " cCL);
      if constexpr (g == 15) Y("s_cmp_eq_u64 " cK ", -2");
      if constexpr (g == 16) Y("s_cselect_b64 " cC ", 63, 62" N "s_or_b64 " cC ", " cC ", " cK);
      if constexpr (g == 17) X("s_and_b64 " cC ", " cC ", " cT N "s_cbranch_scc0 .Lc_slow\n.Lc_back%0:", "n"(P));
      if constexpr (g == 18) Y("s_ff1_i32_b64 " cB ", " cC N "s_lshl_b64 " cC ", 1, " cB);
      if constexpr (g == 19) Y("s_andn2_b64 " cT ", " cT ", " cC);
      if constexpr (g == 20) Y("v_cndmask_b32 v88, v89, v68, " cC N "v_cndmask_b32 v82, v82, v90, " cK);
      if constexpr (g >= 21 && g < 24) X("ds_write_b32 v88, v[%0] offset:%1", "n"(79 + (g - 20) % 3), "n"((g - 21) * 4));
      if constexpr (g == 25) Y(SAMPLE);
    }
    if constexpr (ROLE == 1 && STAMP == 2 && Q == 1 && g == 20) Y("s_memtime s[100:101]" N "s_mov_b32 m0, " sT);
    if constexpr (ROLE == 1 && STAMP == 2 && Q == 2 && g == 1) Y("v_writelane_b32 v59, s100, m0");
    if constexpr ((ROLE & 2) && P * 4 + Q <= (ROLE & 1 ? 4 : FPQ)) {
      constexpr int young = ROLE & 1 ? (P * 4 + Q) * 3 + 3 : P * 4 + Q + 1;
      if constexpr (g == 9) { wait<(P * 4 + Q < FPQ ? 15 : young), -1>(); Y("s_getreg_b32 s84, hwreg(HW_REG_IB_STS, 0, 4)"); }
      if constexpr (g == 13) X("s_or_b32 s84, s84, " pB N "s_cmp_le_u32 s84, %0", "n"(young));
      if constexpr (g == 17) Y("s_cselect_b32 s94, 4, 0" N "s_cselect_b32 " pB ", 64, " pB);
      if constexpr (g == 21) Y("buffer_store_dword v81, off, " pF ", 0 sc0 sc1");
    }
    if constexpr (ROLE == 3) { constexpr int u = P * 4 + Q;
      if constexpr (g == 1 && u % 7 == 0) wait<19, -1>();
      if constexpr (g == 1) X("global_store_dwordx4 v56, v[%0:%0+3], " pD " offset:%1 sc0 sc1", "n"(96 + u * 4), "n"(u * 1024 - 4096));
      if constexpr (g == 5) X("global_load_dwordx4 v[%0:%0+3], v[%1], " pXN " nt", "n"(96 + u * 4), "n"(88 + u));
      if constexpr (u == 6 && g == 17) X("s_mul_i32 s84, " pE ", %0" N "s_add_i32 s84, s84, " pS, "n"(PER));
      if constexpr (u == 6 && g == 18) Y("s_ashr_i32 s84, s84, 5" N "s_and_b32 s84, s84, -2");
      if constexpr (u == 6 && g == 19) Y("s_add_u32 " sEV ", " sEV ", 2" N "s_max_i32 " sEV ", " sEV ", s84");
      if constexpr (u == 6 && g == 20) Y("s_cmp_ge_i32 " pE ", 0" N "s_cselect_b64 vcc, -1, 0");
      if constexpr (u == 6 && g == 21) Y("s_orn2_b32 " sEV ", " sEV ", vcc_lo" N "s_lshl_b32 s84, " pE ", 15");
      if constexpr (u == 6 && g == 22) Y("s_mov_b32 " pB ", 0" N "v_add_u32 v81, 1, v81");
      if constexpr (u == 7 && g == 1) Y("s_add_u32 s90, s22, s84" N "s_addc_u32 s91, s23, 0");
      if constexpr (u == 7 && g == 9) Y(NEXT0);
      if constexpr (u == 7 && g == 11) Y(NEXT1);
      if constexpr (u == 7 && g == 12) Y(NEXT2);
      if constexpr (u == 7 && g == 13) Y(NEXT3);
      if constexpr (u == 7 && g == 17) Y(NEXT4);
    }
    if constexpr (!(Q & 1) && g == 30) wait<ROLE % 4 ? -1 : !ROLE ? 32 : P * 12 + Q * 4 + 12, 0>();
    if constexpr (!(Q & 1) && g == 31) Y("s_barrier");
    if constexpr (Q == 1 && g == 31) wait<-1, 0>();
    if constexpr (Q == 3 && g == 18) Y("s_add_u32 " sT ", " sT ", 1");
    if constexpr (Q == 3 && g == 19) Y("s_cmp_eq_u32 " sT ", " sEV);
    if constexpr (Q == 3 && g == 20) Y("s_cselect_b32 s84, 0, " sMORE);
    if constexpr (Q == 3 && g == 22) Y("s_cmp_lg_u32 s84, 1");
    if constexpr (Q == 3 && g == 30) wait<-1, 6>();
  ));
}
#define SLOOP(role, l, ev) do { Y(l "0:"); stile<role, 0>(); Y("s_cbranch_scc1 " ev "\n" l "1:"); stile<role, 1>(); Y("s_cbranch_scc0 " l "0"); } while (0)
#define BACK(l) "s_bitcmp1_b32 " sT ", 0" N "s_cbranch_scc1 " l "1" N "s_branch " l "0"
#define LEAVE "s_cmp_lg_u32 " sMORE ", 1" N "s_cbranch_scc1 .Lexit" N

enum { NONE, RA, RB, M0, DA, DB, WT, BR, RN, ZB, XS, PB, FL, FC, PR, AD, OU, NP, TS, ID, OA };
struct Op { short k, a, b, c, d; };
constexpr int NQ = 8 + 2 * RL, NG = NQ * 32, L = RL - 1, NX = NG + 48;
struct Plan {
  Op mf[NG] = {}, op[NX][8 * AB + 6] = {};
  char n[NX] = {}, room[NX] = {};
  bool ok = true;
  short mark[160] = {};
  static constexpr bool st4(const Op& o) { return (o.k == XS && o.a % 13 == 12) || (o.k == OU && o.b == 5); }
  static constexpr bool vm(const Op& o) { return o.k == DA || o.k == DB || o.k == FL || o.k == PR || st4(o) || (o.k == PB && o.a); }
  static constexpr bool ds(const Op& o) { return o.k == RA || o.k == RB || (o.k == OU && (o.b == 2 || o.b == 3)); }
  static constexpr bool vs(const Op& o) { return o.k == FC && o.a < 2; }
  static constexpr bool va(const Op& o) { return vs(o) || o.k == ZB || o.k == ID || ((o.k == XS || o.k == PB || o.k == OU) && !vm(o) && !ds(o)); }
  static constexpr bool wr(const Op& o) { return (o.k == PB && o.a) || (o.k == OU && o.b == 2); }
  constexpr bool fits(int g, const Op& o) {
    if (n[g] >= 2 + room[g]) return false;
    if (n[g] && op[g][n[g] - 1].k == AD) return va(o) || (ds(o) && !wr(o));
    if (st4(o)) return !n[g] || (n[g] == 1 && op[g][0].k == WT);
    if (g < NG - 128 && (o.k == RA || o.k == RB) && g % 4 == 1) return false;
    for (int j = 0; j < n[g]; ++j) { const Op& p = op[g][j];
      if (wr(p) || ((vm(o) || ds(o)) && (va(p) || p.k == BR || ds(p) || (vm(p) && !st4(p)))) || (vs(p) && !va(o) && o.k != WT)) return false; }
    return true;
  }
  constexpr int put(int g, int k, int a = 0, int b = 0, int c = 0, int d = 0) { while (!fits(g, Op{(short)k, (short)a, (short)b, (short)c, (short)d})) ++g; return at(g, k, a, b, c, d); }
  constexpr int at(int g, int k, int a = 0, int b = 0, int c = 0, int d = 0) { const Op o = {(short)k, (short)a, (short)b, (short)c, (short)d}; if (st4(o)) room[g] = 2 + n[g]; if (k == AD) room[g] = n[g]; op[g][(int)n[g]++] = o; return g; }
  constexpr void mm(int q, int keep, int half, int fb, int fa) {
    for (int g = q * 32; g < q * 32 + 32; ++g) { const int nn = (g >> 3) & 3, m = sb(g), k = sk(g);
      mf[g] = Op{(short)keep, (short)(half * 16 + m * 4 + nn), (short)(fb + (nn * 2 + k) * 4), (short)(fa + (m * 2 + k) * 4), 0}; }
  }
  static constexpr int set(int r) { return r % 3 == 0 ? VX : r % 3 == 1 ? VY : VZ; }
  constexpr void ra(int g, int st, int f, int s, int u, int tag) { for (int i = 0; i < 8; ++i) put(g + i * st, RA, tag, f * 16 + i, s, u); }
  constexpr void rb(int g, int i0, int i1, int f, int s, int u, int tag) { for (int i = i0; i < i1; ++i) put(g + i - i0, RB, tag, f * 16 + i, s, u); }
  constexpr void jit(int q, int s) { for (int j = 0; j < 4; ++j) rb(q * 32 + 8 + j * 8, j * 4, j * 4 + 4, FB1, s, 2, 90 + (j >> 1)); }
  constexpr void sync(int q, int t0, int t1, int lt) { at(q * 32 + 31, WT, t0, t1, lt); at(q * 32 + 31, BR); }
  int nf = 0;
  constexpr void fill(int g, int skip, int u, int r) {
    if (r > L) return;
    const int so = SA0 + (nf++ & 1), tag = (u == 2 ? 20 : u * 40) + r;
    ok = ok && put(g - 7, RN, so, u, r) <= g - 4 && (u < 2 || put(g - 3, RN, so, 3) < g) && put(g - 2, M0, r % 3 < 2 ? SLX + r % 3 : SLW, u + r % 3 / 2 * 6) < g;
    for (int i = 0; i < 4; g += 4) if (!((skip >> (g / 4 % 4)) & 1)) ok = ok && put(g, u == 2 ? DB : DA, tag, i++, so) == g;
  }
  constexpr int count(bool fix, int tag) {
    int nv = 0, nl = 0;
    for (int i = 0; i < 160; ++i) mark[i] = 30000;
    mark[60] = -24; mark[61] = -8; mark[62] = 0;
    for (int g = 0; g < NX; ++g) for (int j = 0; j < n[g]; ++j) { Op& o = op[g][j];
      if (vm(o)) { ++nv; mark[o.k == XS ? (o.a == 207 ? 64 : 159) : o.k == FL ? 65 : o.k == PR ? 70 + o.a : (o.k == DA || o.k == DB) && o.b == 3 ? o.a : 159] = nv; }
      if (ds(o)) { ++nl; mark[o.k == OU ? 100 + o.a : o.a] = nl; }
      if (o.k == WT) {
        const int v = o.a < 0 ? -1 : nv - (o.b >= 0 && mark[o.b] > mark[o.a] ? mark[o.b] : mark[o.a]), l = o.c < 0 ? -1 : o.c == 0 ? 0 : nl - mark[o.c];
        if (!fix && o.a >= 0 && v <= nv - mark[tag]) return g;
        if (fix) { ok = ok && (o.a < 0 || (v >= 0 && v < 64)) && (o.c < 0 || l >= 0); o.a = v; o.c = l > 14 ? 14 : l; }
      } }
    return -1;
  }
  int st = 0, lw[32] = {}, fin[32] = {}, ad[32] = {}, io = 0, no = 0;
  Op ou[200] = {};
  constexpr void out(int g) {
    for (bool go = true; go && io < no; ) {
      go = false;
      for (int x = io; x < no && x < io + 12 && !go; ++x) { const Op o = ou[x]; const int T = o.a * 4 + o.c;
        if (o.k == NONE || (x > io && !va(o))) continue;
        if (o.k == OU && o.b == 0 && !(g >= fin[T] && st - lw[T] >= 8 && ad[T])) {
          if (x == io && g >= NG && ad[T]) { at(g, NP, 7 - (st - lw[T])); st += 8 - (st - lw[T]); go = true; }
          break; }
        if (g < NG ? fits(g, o) && (o.k != WT || !n[g]) : x == io && n[g] < 2) { at(g, o.k, o.a, o.b, o.c, o.d); ou[x].k = NONE; ++st; go = true; }
        else if (va(o)) break;
      }
      while (io < no && ou[io].k == NONE) ++io;
    }
  }
  constexpr Plan() {
    sync(0, 60, -1, 0); sync(2, 61, -1, 0); sync(4, 62, 0, 0); sync(6, 20, -1, 0); sync(7, 40, -1, 90);
    for (int r = 0; r < L - 1; ++r) { sync(8 + 2 * r, r + 1, 21 + r, 0); if (r < L - 2) sync(9 + 2 * r, 41 + r, -1, 90); else sync(9 + 2 * r, L, 20 + L, 90); }
    sync(NQ - 4, 40 + L, -1, 97);
    if (STAMP == 2) for (int q = 0; q < NQ; ++q) { at(q * 32 + 23, TS, q); at(q * 32 + 23, NONE); }
    rb(0, 12, 16, FB1, VX, 2, 91); rb(4, 9, 16, FB0, VZ, 0, 92);
    for (int t = 0; t < 2; ++t) { const int q = t * 4;
      mm(q, 1, 0, FB1, FA0); mm(q + 1, 0, 0, FB0, FA0); mm(q + 2, 0, 1, FB0, FA1); mm(q + 3, 1, 1, FB1, FA1);
      ra(q * 32 + 32, 2, FA1, set(t), 1, 92); ra(q * 32 + 64, 2, FA0, set(t ^ 1), 0, 92); jit(q + 3, set(t ^ 1));
      if (!t) rb(98, 0, 16, FB0, VZ, 0, 92);
      at(q * 32 + 63, WT, -1, -1, 0);
    }
    for (int r = 0; r < L - 1; ++r) { const int q = 8 + 2 * r;
      mm(q, 1, 0, FB1, FA0); mm(q + 1, 1, 1, FB1, FA1);
      jit(q + 1, set(r + 1)); ra(q * 32 + 6, 2, FA1, set(r), 1, 92); ra(q * 32 + 32, 1, FA0, set(r + 1), 0, 92);
    }
    for (int g = 0; g < 128; ++g) { const int k = g & 1, m = g < 24 ? g % 6 / 2 : g < 48 ? g / 8 - 3 : g / 16, nn = g < 24 ? g / 6 : g < 48 ? g / 2 % 4 : g / 4 % 4, s = g < 48 ? g >= 24 : g / 2 % 2;
      mf[NG - 128 + g] = Op{1, (short)(m * 4 + nn), (short)((s ? FB0 : FB1) + (nn * 2 + k) * 4), (short)((m < 4 ? s ? FA2 : FA0 : s ? FA3 : FA1) + (m % 4 * 2 + k) * 4), 0}; }
    for (int i = 0; i < 8; ++i) { put(NG - 124, RA, 97 + i / 6, FA2 * 16 + i, set(L), 0); if (i == 1) { rb(NG - 124, 0, 8, FB0, set(L), 2, 92); rb(NG - 124, 8, 16, FB0, set(L), 2, 89); } }
    for (int i = 0; i < 16; ++i) put(NG - 84 + (i >> 2) * 16 + (i & 3), RA, 93 + (i >> 2), (i & 2 ? FA3 : FA1) * 16 + (i >> 2) * 2 + (i & 1), set(L - 1 + ((i >> 1) & 1)), 1);
    at(15, WT, -1, -1, 91); at(127, WT, -1, -1, 90); at(143, WT, -1, -1, 91);
    for (int q = 8; q < NQ - 4; q += 2) at(q * 32 + 15, WT, -1, -1, 91);
    at(NG - 117, WT, -1, -1, 91); at(NG - 105, WT, -1, -1, 92); at(NG - 101, WT, -1, -1, 89); at(NG - 81, WT, -1, -1, 98);
    for (int i = 0; i < 4; ++i) at(NG - 65 + i * 16, WT, -1, -1, 93 + i);
    fill(9, 2, 0, 0); fill(33, 10, 2, 0); fill(65, 10, 2, 2); fill(97, 10, 1, 0); fill(129, 10, 0, 1); fill(161, 10, 2, 1); fill(193, 10, 0, 2);
    for (int r = -1; r < L; ++r) { const int g = (9 + 2 * r) * 32; if (r < 0) fill(g + 1, 8, 1, 1); else fill(g + 1, 8, 2, r + 3); fill(g + 21, 8, 1, r + 3); fill(g + 41, 8, 0, r + 4); }
    for (int t = 0; t < 2; ++t) for (int i = 0; i < 4; ++i) put(t * 128 + 50 + 2 * i, ZB, i, t);
    { int g = 6 * 32 + 2;
      for (int i = 0; i < 208; ++i) {
        if (i % 13 == 12) { g += 1; while (g % 4 != 1 || n[g] || g < 6 * 32 + 13 + i / 13 * XQ) ++g; }
        g = put(g, XS, i); ok = ok && (i < 104 || g >= 7 * 32 + 8);
      } }
    { int g = count(false, 64); ok = ok && g > 0; g = put(g - 3, PB, 0) + 4; while (n[g] > 1 || va(op[g][0])) ++g; at(g, PB, 1); ok = ok && g < NG - FG;
      for (int i = 0; i < 21; ++i) g = put(g + 1, OA, i);
      for (int i = 0; i < 4; ++i) g = put(g + 1, ID, i); }
    int gp[16] = {};
    { int g = put(NG - FG, FL, 1); ok = ok && put(g - 16, FL, 0) < g; g += LAGF;
      g = put(g, WT, 65, -1, -1); g = put(g, FC, 0); g = put(g + 1, FC, 1); ++g;
      do ++g; while (n[g] || g % 4 == 1);
      at(g, FC, 2); at(g, NONE);
      for (int i = 0; i < 16; ++i) { do ++g; while (g % 4 != 1 || n[g] > 1 || vm(op[g][0]) || va(op[g][0])); gp[i] = at(g, PR, i); } }
    for (int G = 0; G <= 8; ++G) for (int c = 0; c < 4; ++c) {
      if (G < 8) for (int e = 0; e < 4; ++e) ou[no++] = Op{OU, (short)G, (short)(e >> 1), (short)c, (short)(e & 1)};
      if (G > 0 && c == 2) ou[no++] = Op{WT, -1, -1, (short)(99 + G), 0};
      if (G > 0 && c >= 2) ou[no++] = Op{OU, (short)(G - 1), 5, (short)(c - 2), 0};
      if (G < 8) ou[no++] = Op{OU, (short)G, 2, (short)c, 0};
      if (G < 8 && c == 3) for (int e = 0; e < 2; ++e) ou[no++] = Op{OU, (short)G, 3, (short)e, 0};
    }
    for (int g = 0; g < NG; ++g) if (mf[g].k) fin[mf[g].a] = g;
    for (int g = 0; g < NX; ++g) {
      if (g < NG) { ++st; if (mf[g].k) lw[mf[g].a] = st; }
      st += n[g];
      out(g);
      if (g < NG - 4 && !room[g] && (!n[g] || (op[g][n[g] - 1].k != BR && op[g][0].k != BR))) {
        int c = 0, u[8] = {}, f = 0;
        while (f < 8 && ad[f * 4]) ++f;
        for (int G = f; G < 8 && c < AB; ++G) {
          bool y = !ad[G * 4] && g >= gp[G * 2 + 1] + LAGP;
          for (int i = 0; i < 4; ++i) y = y && st + 1 + c * 4 + i - lw[G * 4 + i] >= 6 && mf[g + 1 + i].a / 4 != G;
          if (y) u[c++] = G; }
        if (c && u[0] == f && (c > 1 || (g >= fin[f * 4 + 3] && io >= f * 25 - 14) || f == 7)) {
          at(g, WT, 71 + u[c - 1] * 2, -1, -1); ++st;
          for (int i = 0; i < c * 4; ++i) { const int T = u[i / 4] * 4 + i % 4; at(g, AD, T); lw[T] = ++st - 3; ad[T] = 1; out(g); } }
      }
    }
    ok = ok && io == no;
    count(true, 0);
  }
};
constexpr Plan PLAN;
static_assert(PLAN.ok, "");
template <int k, int a, int b, int c, int d> DI void emit() {
  if constexpr (k == RA) rdA<b / 16, b % 16, c, d * U>();
  if constexpr (k == RB) rdB<b / 16, b % 16, c, d * U>();
  if constexpr (k == M0) m0<a, b * U>();
  if constexpr (k == DA) dmaA<VRA + (b & 1), SWS, c, b>();
  if constexpr (k == DB) dmaB<VRB + b, c, b>();
  if constexpr (k == WT) wait<a, c>();
  if constexpr (k == BR) Y("s_barrier");
  if constexpr (k == RN && b < 3) X("s_add_u32 s[%0], s[%1], %2", "n"(a), "n"(SRA + b), "n"(b < 2 ? offA(rsrc(c), rcol(c), ~c & 1) : offW(rsrc(c), rcol(c), ~c & 1)));
  if constexpr (k == RN && b == 3) X("s_and_b32 s[%0], s[%0], 0x7ffffff", "n"(a));
  if constexpr (k == ZB) { if constexpr (b) X("v_and_b32 v[%0], 0xffffbfff, v[%0]", "n"(VZ + 2 + a)); else X("v_xor_b32 v[%0], 0x4000, v[%0]", "n"(VZ + 2 + a)); }
  if constexpr (k == XS) { constexpr int i = a / 13, e = a % 13, o = 64 + (i & 1) * 4;
    if constexpr (e == 12) X("global_store_dwordx4 v[%0], v[%1:%1+3], " sXD " offset:%2" XST, "n"(56 + (i >> 3)), "n"(o), "n"((i & 7) * 1024 - 4096));
    else if constexpr (e >= 8) X("v_cvt_pk_bf16_f32 v[%0], v[%1], v[%1+1]", "n"(o + e - 8), "n"((e - 8) * 2));
    else X("v_accvgpr_read_b32 v[%0], a[%1]", "n"(e), "n"(i * 8 + e)); }
  if constexpr (k == PB) { if constexpr (a) Y("buffer_store_dword v62, off, " sWS ", " sXP " sc0 sc1"); else Y("v_mov_b32 v62, " sCALL); }
  if constexpr (k == FL) { if constexpr (a) Y("buffer_load_dword v61, off, " sWS ", " sXF " sc0 sc1"); else Y("buffer_load_dword v60, off, " sWS ", 0 sc1"); }
  if constexpr (k == FC) {
    if constexpr (a == 0) Y("v_readfirstlane_b32 " sMORE ", v60");
    if constexpr (a == 1) Y("v_readfirstlane_b32 s84, v61");
    if constexpr (a == 2) Y("s_cmp_lt_u32 s84, " sCALL N "s_cbranch_scc1 .Lx_slow\n.Lx_back:"); }
  if constexpr (k == PR) X("buffer_load_dwordx4 v[%0:%0+3], v56, " sWS ", s[%2] offen offset:%1 sc0 sc1", "n"(64 + a * 4), "n"((a & 3) * 1024), "n"(24 + a / 4));
  if constexpr (k == AD) X("v_mfma_f32_4x4x4_16b_bf16 v[%0:%0+3], v[8:9], v[%1:%1+1], v[%0:%0+3]", "n"(128 + a * 4), "n"(64 + a * 2));
  if constexpr (k == ID) { if constexpr (a == 0) Y("v_and_b32 v10, 48, v56"); if constexpr (a == 1) Y("v_mov_b32 v8, 0x3f80"); if constexpr (a == 2) Y("v_mov_b32 v9, 0");
    if constexpr (a == 3) Y("v_lshlrev_b64 v[8:9], v10, v[8:9]"); }
  if constexpr (k == OA && a == 20) Y("s_mov_b32 " sT0 ", 0");
  if constexpr (k == OA && a >= 16 && a < 20) X("s_add_u32 s[%0], " sXS ", %1", "n"(8 + a), "n"((a - 16) * 4096));
  if constexpr (k == OA && a < 16) X("s_xor_b32 s[%0], " sOF ", %1", "n"(8 + a), "n"(((a >> 3) * 128 + (a & 7) * 8) * (NCOL * 2)));
  if constexpr (k == OU) { constexpr int t = c * 2 + d, r = 128 + (a * 4 + c) * 4 + d * 2, o = FO + (a * 2 + c) * 4;
    if constexpr (b == 0) X("v_cvt_pk_bf16_f32 v[%0], v[%1], v[%1+1]", "n"(t), "n"(r));
    if constexpr (b == 1) X("v_pk_max_i16 v[%0], v[%0], 0", "n"(t));
    if constexpr (b == 2) X("ds_write_b64 v[%0], v[%1:%1+1] offset:%2", "n"(24 + c), "n"(t), "n"((a & 1) * 2048));
    if constexpr (b == 3) X("ds_read_b128 a[%0:%0+3], v28 offset:%1", "n"(o), "n"((a & 1) * 2048 + c * 1024));
    if constexpr (b == 5) X("buffer_store_dwordx4 a[%0:%0+3], v29, " sOUT ", s[%1] offen" OST, "n"(d ? FO : o), "n"(8 + a * 2 + c)); }
  if constexpr (k == NP) X("s_nop %0", "n"(a));
  if constexpr (k == TS) { if constexpr (a) X("v_writelane_b32 v30, s100, %0", "n"(a - 1)); Y("s_memtime s[100:101]"); }
}

template <int i> DI void zero() {
  if constexpr (i & 1) X("v_mfma_f32_32x32x4_2b_bf16 v[%0:%0+31], v[60:61], v[60:61], 0", "n"(128 + i / 2 * 32));
  else X("v_mfma_f32_32x32x4_2b_bf16 a[%0:%0+31], v[60:61], v[60:61], 0", "n"(i / 2 * 32));
}
#define CALL(n) "s_waitcnt vmcnt(" #n ")" N "v_readfirstlane_b32 s4, v63" N "s_add_u32 " sCALL ", s4, 1" N "s_lshl_b32 s5, " sCALL ", 24" N "s_and_b32 s5, s5, 0x1000000" N \
    "s_add_u32 s44, s44, s5" N "s_xor_b32 s45, s44, 0x4000" N
template <int V> DI void go() {
  STQ(2); wait<V, -1>(); STQ(3); Y("s_barrier");
  REP(i, 0, 8, rdA<FA0, i, VX, 0>()); REP(i, 0, 12, rdB<FB1, i, VX, 2 * U>()); REP(i, 0, 9, rdB<FB0, i, VZ, 0>());
}
#define TAKE(n) "v_mov_b32 v78, " #n N "buffer_atomic_add v78, v93, " sWS ", s22 offen sc0" N

__global__ __launch_bounds__(256, 1) void a2a_gemm(Params) {
  if (STAMP) { Y("v_mov_b32 v31, 0"); STP(0); }
  asm volatile("s_load_dwordx16 s[52:67], s[0:1], 0" N "s_getreg_b32 s3, hwreg(HW_REG_XCC_ID, 0, 3)" N "s_and_b32 s4, s2, 0xf8" N "s_mov_b32 s15, 0"
    :: "{s[0:1]}"(__builtin_amdgcn_kernarg_segment_ptr()), "{s2}"(blockIdx.x), "{v0}"(threadIdx.x) : "v255", "a255", "s101", "vcc", "memory");
  if (FAKEX) Y("s_bitcmp1_b32 s2, 6" N "s_cselect_b32 s5, 5, 7" N "s_and_b32 s3, s3, s5");
  X("s_or_b32 s3, s3, s4\n.Lstart:" N "s_and_b32 s4, s3, 4" N "s_lshr_b32 s5, s3, 6" N "s_add_u32 s4, s4, s5" N "s_lshl_b32 s4, s4, 5" N
    "s_load_dwordx8 s[88:95], s[0:1], s4 offset:0x40" N "s_getpc_b64 s[16:17]\n.Lpc:" N "v_readfirstlane_b32 s14, v0" N "v_mov_b64 v[60:61], 0" N
    "v_lshrrev_b32 v9, 4, v0" N "s_and_b32 s4, s3, 0xf8" N "s_lshl_b32 s4, s4, 8" N "v_lshl_add_u32 v5, v9, 7, s4" N "v_min_u32 v5, .Lend - .Lpc - 4, v5" N
    "global_load_dword v65, v5, s[16:17]" N "s_bfe_u32 s9, s3, 0x10002" N "s_and_b32 s13, s3, 3" N "s_bfe_u32 s10, s3, 0x20003" N
    "s_xor_b32 s12, s13, s9" N "s_lshr_b32 s5, s3, 5" N "s_lshl3_add_u32 s12, s12, s5" N "s_lshl_b32 s12, s12, 9" N "s_and_b32 s11, s5, 1" N
    "s_lshr_b32 s8, s14, 6" N "s_lshl_b32 s4, s2, 5" N "s_add_u32 s4, s4, s8" N "s_lshl2_add_u32 " sCTR ", s4, %0" N
    "v_and_b32 v1, 63, v0" N "v_lshlrev_b32 v56, 4, v1" N "v_lshrrev_b32 v4, 3, v1" N
    "s_brev_b32 s14, s14" N "s_lshr_b32 s77, s14, 12" N "s_lshr_b32 s4, s14, 8" N
    "v_lshrrev_b32 v3, 4, v1" N "v_bitop3_b32 v2, v1, v3, 7 bitop3:0x6c" N "v_lshl_add_u32 v72, v4, 11, s4" N
    "v_lshl_add_u32 v72, v2, 4, v72" N "v_xor_b32 v5, 64, v72" N "v_add_u32 v73, 0x3c00, v5" N "v_add_u32 v74, 0x7800, v72" N "v_add_u32 v75, 0xb400, v5" N
    "s_lshr_b32 s4, s14, 6" N "s_add_u32 s4, s4, s12" N "v_bfe_u32 v6, v1, 1, 3" N
    "v_xor_b32 v6, v6, v3" N "v_lshl_add_u32 v52, v3, 14, s4" N "v_lshl_add_u32 v52, v6, 5, v52" N
    "v_and_or_b32 v52, v56, 16, v52" N "v_xor_b32 v5, 0x80, v52" N "v_add_u32 v53, 0xfc00, v52" N "v_add_u32 v54, 0x1f800, v5" N "v_add_u32 v55, 0x2f400, v5" N
    "s_lshl1_add_u32 s4, s10, s11" N "s_lshl_b32 s4, s4, 8" N "s_or_b32 s4, s4, s9" N "s_lshl_b32 s78, s4, 10" N "s_mov_b32 m0, s77" N
    "s_xor_b32 s4, s10, s8" N "s_and_b32 s4, s4, 1" N
    "s_lshl_b32 s18, s4, 16" N "s_or_b32 s18, s18, s77" N "s_bitset1_b32 s18, 15" N
    "s_xor_b32 s4, s4, s9" N "s_lshl_b32 s4, s4, 8" N "s_lshl_b32 s5, s9, 23" N "s_or_b32 s80, s4, s5", "n"(WS_CTR));
#define CTR "buffer_load_dword v63, off, " sWS ", " sCTR " sc1"
  X("s_cmp_lg_u32 s8, 2" N "s_cbranch_scc1 .Lall" N
    "s_lshr_b32 s4, s3, 1" N "s_xor_b32 s4, s4, s3" N "s_and_b32 s4, s4, 2" N "s_lshl_b32 s4, s4, 9" N "v_or_b32 v5, s4, v56" N "v_xor_b32 v7, 0x400, v5" N
    "v_add_u32 v5, .Ltab - .Lpc, v5" N "v_add_u32 v7, .Ltab - .Lpc, v7" N "global_load_dwordx4 v[80:83], v5, s[16:17]" N "global_load_dwordx4 v[84:87], v7, s[16:17]" N
    "v_min_u32 v79, 1, v1" N "v_lshlrev_b32 v93, 30, v79" N "s_lshl_b32 s22, s3, 7" N "s_or_b32 s22, s22, s15" N "s_add_u32 s22, s22, %0" N
    "v_add_u32 v89, %1 + 16, v56" N "s_waitcnt lgkmcnt(0)" N CTR N TAKE(1) ".Lall:" N "s_waitcnt lgkmcnt(0)", "n"(WS_ROLE), "n"(LDS_CTRL));
  STQ(0);
  REP(i, 0, 4, fill<0, i>());
  Y("s_mov_b32 m0, s18" N "s_andn2_b32 s47, s94, 7" N "s_or_b32 s80, s80, s47");
  REP(i, 0, 4, fill<1, i>());
  Y("s_xor_b32 m0, s18, 0x10000" N "s_xor_b32 s81, s80, 0x100");
  REP(i, 0, 4, fill<2, i>());
  if (CE) Y(CTR);
  REP(i, 0, 4, zero<i>());
  Y("v_and_b32 v7, 0x70, v56");
  Y("s_bfe_u32 s6, s8, 0x10001" N "s_and_b32 s7, s8, 1" N "s_lshl_b32 s7, s7, 7" N "s_lshl_b32 s4, s6, 13" N
    "v_and_b32 v12, 15, v1" N "v_lshl_add_u32 v32, v6, 4, s4" N "v_lshl_add_u32 v32, v12, 7, v32" N "v_xor_b32 v33, 64, v32" N
    "v_bfe_u32 v5, v1, 2, 3" N "v_lshlrev_b32 v34, 5, v5" N "v_xor_b32 v34, s7, v34" N "v_and_b32 v5, 3, v5" N "v_lshl_add_u32 v34, v5, 8, v34");
  zero<4>();
  Y("v_lshl_add_u32 v34, v3, 11, v34" N "v_and_b32 v14, 3, v1" N "v_lshl_add_u32 v34, v14, 3, v34" N "v_xor_b32 v35, 32, v34" N "v_xor_b32 v36, 64, v34" N "v_xor_b32 v37, 0x60, v34");
  REP(i, 0, 12, X("v_add_u32 v[%0], %1, v[%2]", "n"(VY + i), "n"((i / 6 + 1) * 3 * U), "n"(VX + i % 6)); if constexpr (i == 6) zero<5>());
  X("s_lshl_b32 s4, s8, 12" N "s_add_u32 s4, s4, %0" N "v_lshrrev_b32 v5, 5, v1" N "v_bitop3_b32 v5, v1, v5, 7 bitop3:0x6c" N "v_lshl_add_u32 v24, v12, 7, s4" N
    "v_lshl_add_u32 v24, v5, 4, v24" N "v_and_b32 v5, 1, v3" N "v_lshl_add_u32 v24, v5, 3, v24", "n"(9 * U));
  zero<6>();
  X("v_xor_b32 v25, 32, v24" N "v_xor_b32 v26, 64, v24" N "v_xor_b32 v27, 0x60, v24" N
    "v_bitop3_b32 v5, v1, v4, 7 bitop3:0x6c" N "v_lshl_add_u32 v28, v4, 7, s4" N "v_lshl_add_u32 v28, v5, 4, v28" N
    "s_lshl2_add_u32 s4, s10, s6" N "s_lshl_b32 s4, s4, 20" N "s_lshl_b32 s5, s9, 8" N "s_or_b32 s46, s47, s5" N "s_or_b32 s4, s4, s5" N "s_or_b32 s4, s4, s7" N "s_or_b32 s4, s4, s12" N
    "v_lshl_add_u32 v29, v4, 14, s4" N "v_add_u32 v29, v29, v7" N "v_mov_b32 v68, %0", "n"(LDS_CTRL));
  zero<7>();
  X("v_lshl_add_u32 v50, v4, 7, s77" N "v_lshl_add_u32 v50, v2, 4, v50" N "s_bfm_b64 exec, 8, 24" N "v_add_u32 v50, %0, v50" N "v_add_u32 v56, %0, v56" N "s_mov_b64 exec, -1" N
    "v_xor_b32 v51, 64, v50" N "v_add_u32 v57, 0x2000, v56" N
    "s_lshl2_add_u32 s4, s3, s8" N "s_lshl_b32 s4, s4, 14" N "s_add_u32 " sXS ", s4, %1" N "s_xor_b32 s4, " sXS ", 0x51000" N "s_add_u32 s68, s52, s4" N "s_addc_u32 s69, s53, 0" N
    "s_lshl_b32 s4, s3, 5" N "s_add_u32 s4, s4, s8" N "s_lshl2_add_u32 " sXF ", s4, %2" N "s_xor_b32 " sXP ", " sXF ", 0x280" N
    "s_lshl_b32 " sOF ", s11, 21" N "s_mov_b32 " sT ", 0" N "s_mov_b32 s74, s77" N "s_add_u32 s75, s77, 0xc000" N
    "s_lshl_b32 s21, s10, 19" N "s_lshl_b32 s4, s11, 14" N "s_or_b32 s4, s4, s21" N "s_add_u32 s44, s4, %3", "n"(AWAY), "n"(WS_XCH), "n"(WS_XFLAG), "n"(WS_A));
  STQ(1);
  Y("s_cmp_eq_u32 s8, 2" N "s_cbranch_scc1 .Lc_go" N "s_cmp_eq_u32 s8, 3" N "s_cbranch_scc1 .Lp_go" N "s_branch .Ld_go\n"
    ".p2align 6\n.fill ((.Ld_go - .Ld0) & 63) / 4, 4, 0xbf800000\n.Ld_go:");
  {
    REP(i, 0, 4, X("v_add_u32 v[%0+4], 0x10000, v[%0]" N "v_add_u32 v[%1], 0x40000, v[%2]", "n"(VDA + i), "n"(VDB + 4 + i), "n"(VRB + i)));
    Y("s_xor_b32 s79, s78, 0x40000" N "s_add_u32 s92, s78, 128" N "s_mov_b32 s88, s18" N "s_xor_b32 s89, s18, 0x10000" N
      "s_lshr_b32 s4, s18, 1" N "s_and_b32 s4, s4, 0x8000" N "s_or_b32 s4, s4, s77" N "s_or_b32 s90, s4, 0x14000" N "s_xor_b32 s91, s90, 0x8000" N
      "s_add_u32 s93, s80, 0x100000" N "s_add_u32 s94, s81, 0x100000" N "s_add_u32 s95, s79, 128" N "s_add_u32 s96, s78, 0x100" N
      "s_add_u32 s97, s80, 0x200000" N "s_add_u32 s98, s81, 0x200000");
    Y("s_add_u32 m0, s77, 0x4000" N "s_add_u32 s99, s79, 0x100");
    REP(i, 0, 4, fill<0, i, SA1>());
    Y("s_add_u32 m0, s77, 0x5000" N "s_lshl_b32 s86, s8, 8");
    REP(i, 4, 8, fill<0, i, SA1>());
    Y("s_add_u32 m0, s77, 0xc000" N "s_mov_b32 " sEV ", 6");
    REP(i, 0, 4, fill<0, i, 92>());
    Y("s_add_u32 m0, s77, 0xd000" N "s_mov_b32 " sAH ", 0x40000");
    REP(i, 4, 8, fill<0, i, 92>());
    if (!CE) Y(CTR);
    go<17>(); wait<-1, 13>();
    STP(1);
    stile<4, 0>(); Y("s_cbranch_scc1 .Lexit"); stile<4, 1>();
    SLOOP(0, ".Ld", ".Ld_ev");
    Y(".Ld_ev:" N LEAVE "s_mov_b64 s[60:61], s[52:53]" N "s_mov_b32 s62, s54" N "s_mov_b32 " sAH ", 0x4000");
    REP(i, 0, 8, X("v_add_u32 v[%0], %1, v[%2]", "n"(VDA + i), "n"((i >> 2) * 4096), "n"(VRA + (i & 1))));
    Y("s_branch .Ld0");

    X(".p2align 6\n.fill ((.Lc_go - .Lc0) & 63) / 4, 4, 0xbf800000\n.Lc_go:" N
      "s_lshl_b32 s4, s10, 18" N "s_xor_b32 s5, s12, %[px]" N "s_or_b32 s4, s4, s5" N "v_lshrrev_b32 v5, 2, v1" N "v_lshl_add_u32 v92, v5, 14, s4" N "v_lshl_add_u32 v92, v14, 7, v92" N
      "v_cmp_gt_u32 vcc, %[pn], v1" N "v_bfrev_b32 v90, 1" N "v_and_b32 v94, 15, v1" N "v_lshlrev_b32 v94, 2, v94" N "v_cndmask_b32 v92, v93, v92, vcc" N "v_mov_b32 v72, 0" N
      "s_mov_b32 s88, %[t0]" N "s_mov_b32 s89, %[t1]" N "s_mov_b64 " cK ", %[cm]" N "s_mov_b32 " cCL ", 0x7f000000" N "s_mov_b32 " sEV ", %[tv] - 1" N
      "s_lshl_b32 s4, s10, 10" N "s_add_u32 " cFB ", s4, %[af]" N "s_lshl_b32 s4, s10, 7" N "s_or_b32 s4, s4, s12" N "s_add_u32 " cCM ", s4, %[cl]" N
      "s_andn2_b32 s5, s80, 0x800000" N "v_mov_b32 v6, s5" N "s_waitcnt vmcnt(%[ce])" N CALL(13)
      "v_mov_b32 v5, s44" N "s_mov_b64 exec, 0x7f" N "v_mov_b32 v5, s78" N "v_mov_b32 v6, s80" N "s_mov_b64 exec, -1" N
      "v_and_b32 v76, 15, v83" N "v_lshl_add_u32 v76, s4, 4, v76" N "v_and_b32 v67, 15, v87" N "v_lshl_add_u32 v67, s4, 4, v67" N "s_mul_i32 " cTK ", s4, 2 * %[np]" N "s_add_u32 " cTK ", " cTK ", %[np]",
      [ce] "n"(12 + CE), [px] "n"(PFX * 512), [pn] "n"(64 / PFN), [t0] "n"((unsigned)TODO), [t1] "n"((unsigned)(TODO >> 32)), [cm] "n"(-NOCOMM), [tv] "n"(TV), [af] "n"(WS_AFLAG), [cl] "n"(WS_CLAIM), [np] "n"(N_POOL));
    REP(i, 0, 2, X("v_add_u32 v[%0], v[%0], v5" N "v_add_u32 v[%0+1], v[%0+1], v6" N "v_add_u32 v[%0+3], s47, v[%0+3]" N
      "v_and_b32 v[%0+1], 0x7ffffff, v[%0+1]" N "v_and_b32 v[%0+3], 0x87ff0000, v[%0+3]", "n"(80 + i * 4)));
    Y("v_readfirstlane_b32 s5, v78" N "s_cmp_eq_u32 s15, 0" N "s_cselect_b32 s5, s5, s4" N "s_cmp_lg_u32 s5, s4" N "s_cbranch_scc1 .Lc_taken\n.Lc_mine:");
    go<CE>();
    X("v_mov_b32 v78, 1" N "v_mov_b32 v69, 1" N "v_mov_b32 v70, v90" N "v_mov_b32 v71, v90" N "s_lshl_b64 exec, %0, %2" N "v_mov_b32 v70, 0" N "s_lshl_b64 exec, %1, %2" N "v_mov_b32 v71, 0" N "s_mov_b64 exec, -1",
      "n"(DRA), "n"(DRB), "n"(DR0));
    wait<-1, 13>();
    STP(1);
    SLOOP(1, ".Lc", ".Lc_ev");
#define VOTES "buffer_load_dword v95, v94, " sWS ", 0 offen offset:%[vo] sc1" N
    if (EVF) X(".Lc_ev:" N LEAVE "s_lshl_b32 " cB ", " sCALL ", 1" N "s_cmp_eq_u32 " sT ", %[tv]" N "s_cbranch_scc1 3f" N VOTES "s_mov_b32 " sEV ", %[tv]" N "s_mov_b32 " sT0 ", 0" N "s_branch .Lc0\n"
      "3:" N "v_cmp_le_u32 vcc, " cB ", v95" N "v_cmp_lt_u32 s[86:87], " cB ", v95" N "s_cmp_eq_u64 vcc, -1" N "s_cbranch_scc0 2f\n"
      "4:" N "s_bcnt1_i32_b64 " cB ", s[86:87]" N "s_cmp_gt_u32 " cB ", %[th]" N "s_cbranch_scc0 .Lc1" N
      "v_mov_b64 v[82:83], v[86:87]" N "v_mov_b64 v[80:81], v[84:85]" N "v_mov_b32 v76, v67" N "s_mov_b64 " cK ", %[cm]" N SAMPLE N "s_branch .Lc1\n"
      "2:" N VOTES EXPIRED "s_cbranch_scc0 3b" N "s_branch 4b",
      [tv] "n"(TV), [vo] "n"(WS_VOTE), [cm] "n"(-NOCOMM), [th] "n"(VTH), DL);
    else
    X(".Lc_ev:" N LEAVE "s_cmp_eq_u32 " sT ", %[tv]" N "s_cbranch_scc1 2f" N VOTES "s_mov_b32 " sEV ", %[tv]" N "s_branch .Lc0\n"
      "2:" N "s_mov_b32 " sT0 ", 0" N "s_lshl_b32 " cB ", " sCALL ", 1\n"
      "3:" N "v_cmp_le_u32 vcc, " cB ", v95" N "s_cmp_eq_u64 vcc, -1" N "s_cbranch_scc1 4f" N VOTES EXPIRED "s_cbranch_scc0 3b\n"
      "4:" N "v_cmp_lt_u32 vcc, " cB ", v95" N "s_bcnt1_i32_b64 " cB ", vcc" N "s_cmp_gt_u32 " cB ", %[th]" N "s_cbranch_scc0 .Lc1" N
      "v_mov_b64 v[82:83], v[86:87]" N "v_mov_b64 v[80:81], v[84:85]" N "v_mov_b32 v76, v67" N "s_mov_b64 " cK ", %[cm]" N SAMPLE N "s_branch .Lc1",
      [tv] "n"(TV), [vo] "n"(WS_VOTE), [cm] "n"(-NOCOMM), [th] "n"(VTH), DL);
    static_assert(TV == 5, "");
    Y(".Lc_slow:" N "s_mov_b32 " sT0 ", 0\n.Lc_again:");
    CNT(10, 1);
    X(SAMPLE N EXPIRED "s_cbranch_scc0 5f" N "v_mov_b32 v76, 0\n5:" N ARRIVED N "s_cmp_eq_u64 " cK ", -2" N "s_cselect_b64 " cC ", 63, 62" N "s_or_b64 " cC ", " cC ", " cK N
      "s_and_b64 " cC ", " cC ", " cT N "s_cbranch_scc0 .Lc_again" N BACK(".Lc_back"), DL);
    Y(".Lc_taken:" N TAKE(-1) "s_add_u32 s22, s22, 128" N "s_bitset0_b32 s22, 15" N "s_waitcnt vmcnt(0)" N TAKE(1) "s_waitcnt vmcnt(0)" N "v_readfirstlane_b32 s5, v78" N
      "s_cmp_lg_u32 s5, s4" N "s_cbranch_scc1 .Lc_taken" N "v_mov_b32 v79, s22" N "s_branch .Lc_mine");

    Y(".p2align 6\n.fill ((.Lp_go - .Lpi0) & 63) / 4, 4, 0xbf800000\n.Lp_go:");
    if (!CE) Y(CTR);
    X("s_lshl2_add_u32 s20, s10, s13" N "s_lshl_b32 s4, s20, 17" N "v_lshl_add_u32 v88, v4, 11, s4" N "v_add_u32 v88, v88, v7" N
      "s_and_b32 s4, s3, 0xc7" N "s_cmp_lg_u32 s4, %[sw]" N "s_addc_u32 s19, s11, 0" N "s_xor_b32 s4, s19, s94" N "s_bitcmp1_b32 s4, 0" N "s_cselect_b32 s4, 0x100, 0" N
      "s_mov_b64 s[16:17], s[88:89]" N "s_add_u32 s86, s88, s4" N "s_addc_u32 s87, s89, 0" N
      "s_and_b32 s19, s19, 1" N "s_cselect_b32 " pM ", 0, 0xfffc" N "s_xor_b32 " pM ", " pM ", %[em] - 2" N
      "s_lshl_b32 s4, s10, 6" N "s_or_b32 s4, s4, s19" N "s_xor_b32 s4, s4, 1" N "s_lshl2_add_u32 s4, s4, s13" N "s_lshl2_add_u32 s92, s4, s92" N "s_addc_u32 s93, s93, 0" N
      "s_lshl_b32 s4, s13, 13" N "s_or_b32 s7, s21, s4" N
      "s_and_b32 s6, s94, 7" N "s_mul_i32 s4, s20, 7" N "s_add_u32 s4, s4, s6" N "s_mul_i32 s4, s4, %[pr]" N "s_lshr_b32 s4, s4, 16" N "s_sub_i32 " pS ", s4, %[sh]" N
      "s_mul_i32 s4, s19, %[pe]" N "s_add_i32 s4, s4, " pS N "s_max_i32 s4, s4, 0" N "s_lshr_b32 s4, s4, 6" N "s_lshl_b32 " sEV ", s4, 1" N
      "s_lshl_b32 s14, s20, 2" N "s_bitcmp1_b32 s6, 0" N "s_cselect_b32 s4, 0x22222222, 0" N "s_xor_b32 s20, s4, 0x75643120" N "s_xor_b32 s21, s4, 0xfdecb9a8" N
      NEXT0 N NEXT1 N NEXT2 N NEXT3 N NEXT4 N
      "s_mov_b32 s94, 0" N "s_mov_b32 " pB ", 64" N "s_mov_b64 vcc, -1" N "s_lshl_b32 s19, s19, 15" N
      "s_cmp_lt_u32 s6, %[end]" N "s_cbranch_scc1 1f" N
      "v_mov_b32 v86, s14" N "s_lshl_b32 s4, s14, 7" N "s_or_b32 s4, s4, s47" N "s_xor_b32 s4, s4, 0x4800000" N
      "v_lshl_add_u32 v84, v14, 14, s4" N "v_add_u32 v85, 0x2000, v84" N "s_xor_b32 s4, s4, 0x200" N "v_and_b32 v5, 31, v1" N "v_lshl_add_u32 v83, v5, 14, s4" N
      "v_lshrrev_b32 v5, 5, v1" N "v_lshl_add_u32 v83, v5, 13, v83" N
      "s_add_u32 s6, s6, s11" N "s_cmp_eq_u32 s6, 8" N "s_cselect_b32 " sEV ", %[tp], -1" N "s_mov_b64 vcc, 0\n1:",
      [sw] "n"(SWP), [pr] "n"(PHASE), [pe] "n"(PER), [em] "n"(EM), [sh] "n"(SHIFT), [end] "n"(NOCOMM ? 0 : 7), [tp] "n"(TP));
    REP(i, 1, 8, X("v_add_u32 v[%0], %1, v88", "n"(88 + i), "n"(i * 16384)));
    if (XL == 2) go<1>();
    REP(i, 0, 8, X("global_load_dwordx4 v[%0:%0+3], v[%1], s[86:87] nt", "n"(96 + i * 4), "n"(88 + i)));
    if (XL == 1) go<9>();
    X(CALL(8) "s_add_u32 s5, s5, s7" N "s_add_u32 s22, s90, s5" N "s_addc_u32 s23, s91, 0" N "s_add_u32 s90, s22, s19" N "s_addc_u32 s91, s23, 0" N
      "s_lshl_b32 s4, s4, 4" N "v_mov_b32 v81, s4" N
      "s_cmp_eq_u32 " sEV ", %0" N "s_cselect_b64 exec, -1, 0" N "global_load_dword v82, v83, s[56:57]" N "s_mov_b64 exec, -1", "n"(TP));
    wait<-1, 13>();
    STP(1);
    Y("s_cmp_lg_u32 " sEV ", 0" N "s_cbranch_scc1 .Lpi0" N "s_waitcnt vmcnt(0)\n.Lpp0:");
    stile<3, 0>(); Y("s_cbranch_scc1 .Lp_ev"); stile<3, 1>(); Y("s_cbranch_scc1 .Lpp0");
    Y(".Lpi0:"); stile<2, 0>(); Y("s_cbranch_scc1 .Lp_ev\n.Lpi1:"); stile<2, 1>();
    X("s_cbranch_scc0 .Lpi0" N "s_cbranch_vccnz .Lpp0\n.Lp_ev:" N LEAVE
      "s_mov_b32 s98, 0" N "s_mov_b32 s97, 2\n"
      "4:" N "s_mov_b64 exec, vcc" N "global_load_dword v82, v84, s[56:57]" N "global_load_dword v83, v85, s[56:57]" N "s_waitcnt vmcnt(1)" N "s_memtime s[88:89]" N
      "s_waitcnt vmcnt(0)" N "s_memtime s[90:91]" N "v_swap_b32 v84, v85" N "v_add_u32 v84, %0, v84" N "v_add_u32 v85, %0, v85" N "s_mov_b64 exec, -1" N "s_waitcnt lgkmcnt(0)" N
      "s_sub_u32 s88, s90, s88" N "s_min_i32 s88, s88, 400" N "s_and_b32 s88, s88, vcc_lo" N "s_sub_i32 s98, s88, s98" N "s_mov_b64 vcc, -1" N
      "s_sub_u32 s97, s97, 1" N "s_cbranch_scc0 4b" N
      "s_getreg_b32 s88, hwreg(HW_REG_XCC_ID, 2, 1)" N "s_lshr_b32 s98, s98, 31" N "s_mov_b64 vcc, 0" N "s_xor_b32 s88, s88, s98" N "s_xor_b32 s88, s88, 1" N "s_lshl1_add_u32 s88, " sCALL ", s88" N
      "v_mov_b32 v82, s88" N "buffer_store_dword v82, v86, " sWS ", 0 offen offset:%1 sc1" N BACK(".Lpi"), "n"(4 * 16384), "n"(WS_VOTE));

    X(".Lx_slow:" N "buffer_load_dword v61, off, " sWS ", " sXF " sc0 sc1" N EXPIRED "s_cselect_b32 " sMORE ", " sCALL ", " sMORE N "s_cbranch_scc1 .Lx_back" N
      "v_readfirstlane_b32 s84, v61" N "s_cmp_lt_u32 s84, " sCALL N "s_cbranch_scc1 .Lx_slow", DL);
    CNT(14, 1);
    Y("s_branch .Lx_back");

    Y(".Lexit:" N "s_cmp_lg_u32 " sMORE ", 0" N "s_cbranch_scc1 .Lagain" N "s_bitcmp1_b32 " sT ", 0" N "s_cbranch_scc0 .Lt" N
      "s_mov_b32 s84, s74" N "s_mov_b32 s74, s75" N "s_mov_b32 s75, s84");
    REP(i, 0, 6, X("v_swap_b32 v[%0], v[%1]", "n"(VX + i), "n"(VY + i)));
    REP(i, 0, 4, emit<ZB, i, 0, 0, 0>());
    Y(".Lt:");
    STP(2);
    REP(q, 0, NX / 16, REP(j, 0, 16, constexpr int g = q * 16 + j;
      if constexpr (g < NG) { constexpr Op f = PLAN.mf[g < NG ? g : 0]; mfma<f.k, f.a, f.b, f.c>(); }
      if constexpr (g == NG) STP(3);
      REP(i, 0, PLAN.n[g], constexpr Op o = PLAN.op[g][i]; emit<o.k, o.a, o.b, o.c, o.d>())));
    Y("s_cmp_eq_u32 " sMORE ", " sCALL N "s_cbranch_scc0 .Lfin" N "s_waitcnt vmcnt(0)" N "v_mov_b32 v0, 0x7fc07fc0");
    REP(i, 0, 4, X("v_accvgpr_write_b32 a[%0], v0", "n"(FO + i)));
    REP(i, 0, 16, emit<OU, i / 2, 5, i % 2, 1>());
    Y(".Lfin:" N "v_mov_b32 v0, " sCALL N "buffer_store_dword v0, off, " sWS ", " sCTR " sc1");
    if (STAMP) {
      Y("s_waitcnt vmcnt(0)"); STP(4);
      Y("s_getreg_b32 s84, hwreg(HW_REG_XCC_ID, 0, 4)" N "v_writelane_b32 v31, s84, 30" N "v_writelane_b32 v31, " sT ", 24" N "v_mbcnt_lo_u32_b32 v0, -1, 0" N "v_mbcnt_hi_u32_b32 v0, -1, v0" N "v_lshlrev_b32 v0, 2, v0" N
        "s_and_b32 s84, " sCTR ", 15" N "s_lshr_b32 s84, 0x7310, s84" N "s_and_b32 s84, s84, 7" N "s_lshl_b32 s84, s84, 7" N "s_lshr_b32 s85, " sCTR ", 7" N "s_sub_u32 s85, s85, 128" N
        "s_lshl_b32 s86, s85, 10" N "s_add_u32 s84, s84, s86" N "s_add_u32 s84, s84, 0x6500000" N "s_mov_b32 exec_hi, 0" N "buffer_store_dword v31, v0, " sWS ", s84 offen" N "s_mov_b64 exec, -1");
      if (STAMP == 2) Y("s_and_b32 s84, " sCTR ", 15" N "s_cmp_eq_u32 s84, 8" N "s_cbranch_scc0 1f" N "s_lshl_b32 s85, s85, 9" N "s_add_u32 s85, s85, 0x6600000" N "v_writelane_b32 v30, s100, 63" N
        "buffer_store_dword v59, v0, " sWS ", s85 offen" N "buffer_store_dword v30, v0, " sWS ", s85 offen offset:256\n1:");
    }
    Y("s_endpgm\n.Lagain:" N "s_waitcnt vmcnt(0) lgkmcnt(0)" N "s_barrier" N "s_bfe_u32 s3, " sMORE ", 0x80007" N "s_brev_b32 s15, 4" N "s_branch .Lstart\n.p2align 7\n.Ltab:");
    REP(q, 0, 2, REP(l, 0, 64, X(".long %0, %1, %2, %3", "n"(table(q, l, 0)), "n"(table(q, l, 1)), "n"(table(q, l, 2)), "n"(table(q, l, 3)))));
    Y(".Lend:");
  }
}

extern "C" void kernel_launch(int rank, int nranks, ncclComm_t comm,
                              hipStream_t stream,
                              void* const* d_in, const int* in_sizes, int n_in,
                              void* d_out, int out_size,
                              const int* in_dtypes, int out_dtype,
                              void* const* d_out_all,
                              void* const* d_ws_all, size_t ws_bytes) {
  if (nranks != NRANK || (unsigned)rank >= (unsigned)NRANK || n_in < 2 || ws_bytes < WS_NEED ||
      in_sizes[0] != 8192 * 1024 || in_sizes[1] != 8192 * 8192 || out_size != 1024 * 8192) {
    fprintf(stderr, "a2a_gemm: unexpected problem shape\n");
    return;
  }
  HIP_CHECK(hipFuncSetAttribute((const void*)a2a_gemm, hipFuncAttributeMaxDynamicSharedMemorySize, LDS_BYTES));
  const char* x = (const char*)d_in[0];
  char* ws = (char*)d_ws_all[rank];
  Params p = {{ws, (unsigned)WS_NEED, 0x20000}, {d_in[1], 8192 * 16384, 0x20000}, {x + rank * BLK_B, (unsigned)BLK_B, 0x20000}, {d_out, 1024 * 16384, 0x20000}, {}};
  p.duty[3] = {x, ws, ws, (unsigned)rank << 24 | 7, 0x20000};
  for (int j = 0, n[2] = {0, 4}; j < 7; ++j) {
    const int q = (rank + 1 + j) & 7, e = n[((0xb2 >> rank) ^ (0xb2 >> q)) & 1]++;
    char* to = (char*)d_ws_all[q];
    p.duty[e] = {x + q * BLK_B, to + WS_A + (6 - j) * BLK_B + 4096, to + WS_AFLAG + (6 - j) * 128, (unsigned)rank << 24 | j, 0x20000};
  }
  a2a_gemm<<<256, 256, LDS_BYTES, stream>>>(p);
  HIP_CHECK(hipGetLastError());
}
```

```cpp
#include <hip/hip_runtime.h>
#include <rccl/rccl.h>
#include <cstdio>

#define HIP_CHECK(expr) do { hipError_t _e = (expr); if (_e != hipSuccess) { \
    fprintf(stderr, "HIP error %s at %s:%d\n", hipGetErrorString(_e), __FILE__, __LINE__); \
    return; } } while (0)

typedef unsigned long long u64;

#define PER 160
#define SHIFT 28
#define FPQ 5
#define TV 5
#define TP 1
#define RL 11
#define AB 3
#define FG 291
#define LAGF 72
#define LAGP 80
#define PFR 44
#define PFS 1
#define PFN 1
#define DR0 46
#define DRA 0x1b
#define DRB 0x16
#define XQ 16

constexpr int NRANK = 8;
constexpr int NCOL = 8192;
constexpr int U = 16384;
constexpr int LDS_BYTES = 10 * U;
constexpr int LDS_CTRL = 9 * U + 1024;
constexpr size_t BLK_B = (size_t)1024 * 1024 * 2;
constexpr int N_POOL = 7;
constexpr int N_X = (RL - 7) / 2;
constexpr unsigned XSRC = N_X == 1 ? 6u : N_X == 4 ? 0x6614u : 0x14614u;
static_assert(PFR >= TV && PFR + ((7 + N_X) * PFN - 1) * PFS <= 54 - N_X, "");
static_assert(__builtin_popcount(DRA) + __builtin_popcount(DRB) == N_POOL && ((u64)(DRA | DRB) << DR0) >> (56 - 3 - N_X) == 0, "the tickets come in time");
static_assert(RL >= 7 && RL <= 17 && (RL & 1), "");
static_assert(PER <= 200 && PER >= 128 && PER / 2 - SHIFT <= 64 && PER - SHIFT <= 160 && 8 * PER - SHIFT <= 32 * (50 - N_X), "a K-tile must be published before anyone can be waiting for it");
constexpr unsigned RROW = 128, RCOL = 32768;
constexpr int WS_TMO = 0;
constexpr int WS_VOTE = 256;
constexpr size_t WS_CTR = 16384;
constexpr size_t WS_CLAIM = 49152;
constexpr size_t WS_XFLAG = 65536;
constexpr size_t WS_AFLAG = 131072;
constexpr size_t WS_ROLE = 196608;
constexpr size_t WS_A = (size_t)1 << 20;
constexpr size_t WS_XCH = (size_t)34 << 20;
constexpr unsigned AWAY = (50u << 20) + 128;
constexpr size_t WS_NEED = (size_t)103 << 20;
constexpr unsigned DEADLINE = 50000000u;

constexpr unsigned offA(int s, int k, int p) { return s * BLK_B + (2 * k + p) * RCOL; }
constexpr unsigned offW(int s, int k, int p) { return ((1 + s) * 16 + k / 2 * 4 + p * 2 + k % 2) << 20; }
constexpr int rsrc(int r) { return r < 7 ? r : (XSRC >> (r - 7) / 2 * 4) & 7; }
constexpr int rcol(int r) { return r < 7 ? 7 : (0x55666 >> (r - 7) / 2 * 4) & 7; }
constexpr unsigned table(int q, int l, int i) {
  const int k = l / 7, o = q ? 6 - l % 7 : l % 7, w = l < 7 || l == 63, s = w ? (l + 6) % 7 : (o * 2) % 7, p = w ? ~s & 1 : k < 8 ? q : s & 1, own = l < 5 ? l + 3 : 3;
  const int z = (l - PFR) / PFS, y = z / PFN, r = y < 7 ? y : 2 * y - 7;
  return i == 0 ? (w ? own * 128 : offA(s, k - 1, p)) : i == 1 ? (w ? own << 20 : offW(s, k - 1, p)) : i == 2 ? (l ? s * 128 + p * 16 : 1u << 31)
       : (l >= PFR && z * PFS + PFR == l && r < RL ? offW(rsrc(r), rcol(r), q) + z % PFN * (0x40000 / PFN) : 1u << 31) | (l ? w ? 8 : k : 1);
}
constexpr u64 TODO = 0x00ffffffffffffbeull & ~((N_X & 1 ? 1ull << 52 : 0) | (N_X >= 2 ? 5ull << 51 : 0) | (N_X == 4 ? 0x81ull << 45 : 0) | (N_X == 5 ? 5ull << 44 : 0));

struct Srd { const void* base; unsigned bytes, mode; };
struct Duty { const char* x; char* to; char* flag; unsigned who, mode; };
struct Params {
  Srd ws, w, x, out;
  Duty duty[8];
};

constexpr int FA0 = 128, FA1 = 160, FB0 = 192, FB1 = 224, FA2 = 0, FA3 = 32, FO = 64, VX = 32, VY = 38, VZ = 44, VRA = 50, VRB = 52, VDA = 72, VDB = 80;
constexpr int SWS = 52, SW = 56, SA = 60, SRA = 44, SLX = 74, SLW = 77, SA0 = 78, SA1 = 79, SB1 = 80, SB0 = 81;
#define sT "s48"
#define sCALL "s49"
#define sMORE "s50"
#define sEV "s51"
#define sWS "s[52:55]"
#define sOUT "s[64:67]"
#define sXD "s[68:69]"
#define sXS "s70"
#define sXF "s71"
#define sXP "s72"
#define sCTR "s73"
#define sAH "s82"
#define sOF "s83"
#define cT "s[88:89]"
#define cK "s[90:91]"
#define cC "s[92:93]"
#define cB "s94"
#define cCL "s95"
#define cTK "s96"
#define cFB "s97"
#define cCM "s98"
#define sT0 "s99"
#define pXN "s[88:89]"
#define pD "s[90:91]"
#define pF "s[92:95]"
#define pB "s96"
#define pCS "s97"
#define pS "s98"
#define pL "s99"

#define DI __device__ __forceinline__
#define X(s, ...) asm volatile(s :: __VA_ARGS__ : "memory")
#define Y(s) asm volatile(s ::: "memory")
#define N "\n\t"
template <int V> struct IC { static constexpr int value = V; };
template <int I, int E, class F> DI void rep(F f) { if constexpr (I < E) { f(IC<I>{}); rep<I + 1, E>(f); } }
#define REP(i, a, b, ...) rep<a, b>([](auto i##_) __attribute__((always_inline)) { constexpr int i = decltype(i##_)::value; __VA_ARGS__; })

template <int KEEP, int T, int FB, int FA> DI void mfma() {
  if constexpr (KEEP) X("v_mfma_f32_16x16x32_bf16 v[%0:%0+3], a[%1:%1+3], a[%2:%2+3], v[%0:%0+3]", "n"(128 + T * 4), "n"(FB), "n"(FA));
  else X("v_mfma_f32_16x16x32_bf16 a[%0:%0+3], a[%1:%1+3], a[%2:%2+3], a[%0:%0+3]", "n"(T * 4), "n"(FB), "n"(FA));
}
template <int F, int i, int S, int O> DI void rdA() { X("ds_read_b128 a[%0:%0+3], v[%1] offset:%2", "n"(F + i * 4), "n"(S + (i & 1)), "n"(O + (i >> 1) * 2048)); }
template <int F, int i, int S, int O> DI void rdB() { X("ds_read_b64_tr_b16 a[%0:%0+1], v[%1] offset:%2", "n"(F + i * 2), "n"(S + 2 + (i >> 2)), "n"(O + ((i >> 1) & 1) * 8192 + (i & 1) * 1024)); }
template <int S, int O> DI void m0() { X("s_add_u32 m0, s[%0], %1", "n"(S), "n"(O)); }
template <int V, int R, int SO, int i> DI void dmaA() { X("buffer_load_dwordx4 v[%0], s[%1:%1+3], s[%2] offen offset:%3 sc0 sc1 lds", "n"(V), "n"(R), "n"(SO), "n"((i & 3) * 1024)); }
template <int V, int SO, int i> DI void dmaB() { X("buffer_load_dwordx4 v[%0], s[56:59], s[%1] offen offset:%2 lds", "n"(V), "n"(SO), "n"((i & 3) * 1024)); }
template <int V, int L> DI void wait() {
  if constexpr (V < 0) X("s_waitcnt lgkmcnt(%0)", "n"(L)); else if constexpr (L < 0) X("s_waitcnt vmcnt(%0)", "n"(V)); else X("s_waitcnt vmcnt(%0) lgkmcnt(%1)", "n"(V), "n"(L));
}
template <int P, int Q, int i> DI void fm0() { if constexpr (Q == 0 || Q == 3) m0<SLW, (P * 3 + Q / 3) * U + (i >> 2) * 4096>(); else m0<87 + P * 2 + Q, (i >> 2) * 4096>(); }
template <int Q, int i, int SO = Q == 3 ? SA1 : Q ? SA1 + Q : SA0> DI void fill() { if constexpr (Q == 0 || Q == 3) dmaA<VDA + i, SA, SO, i>(); else dmaB<(i < 4 ? VRB : VDB) + i, SO, i>(); }
constexpr int early(int P, int Q, int g) {
  int n = P * 44 + Q * 12;
  for (int x = 0; x < 32; ++x) { const bool y = x % 4 == 1 || (Q == 0 ? x >= 15 && x < 30 && x % 2 : Q < 3 && x % 8 == 3); if (x == g) return y ? n : -1; n += y; }
  return -1;
}
#define SAMPLE "buffer_load_dwordx4 v[72:75], v82, " sWS ", " cFB " offen sc0 sc1"
#define ARRIVED "v_min3_u32 v77, v72, v73, v74" N "v_min_u32 v77, v77, v75" N "v_cmp_le_u32 vcc, v76, v77" N "s_or_b64 " cK ", " cK ", vcc"
#define EXPIRED "buffer_load_dword v60, off, " sWS ", 0 sc1" N "s_memrealtime s[84:85]" N "s_sleep 1" N "s_waitcnt vmcnt(0) lgkmcnt(0)" N "v_readfirstlane_b32 s85, v60" N \
    "s_or_b32 s84, s84, 1" N "s_cmp_eq_u32 " sT0 ", 0" N "s_cselect_b32 " sT0 ", s84, " sT0 N "s_sub_u32 s84, s84, " sT0 N \
    "s_cmp_eq_u32 s85, " sCALL N "s_cselect_b32 s84, -1, s84" N "s_cmp_gt_u32 s84, %[dl]" N "s_cbranch_scc0 1f" N \
    "v_mov_b32 v60, " sCALL N "buffer_store_dword v60, off, " sWS ", 0 sc1\n1:" N
#define DL [dl] "n"(DEADLINE)

constexpr int nth(int g) { return (g & 3) == 1 ? -99 : g - (g + 2) / 4; }
constexpr int sb(int g) { return ((g >> 1) & 3) ^ (g & 8 ? 3 : 0); }
constexpr int sk(int g) { return (g ^ (g >> 1)) & 1; }
template <int ROLE, int P> DI void stile() {
  REP(Q, 0, 4, REP(g, 0, 32,
    constexpr int cur = P ? VY : VX, nxt = P ? VX : VY, keep = Q == 0 || Q == 3, a = g >> 3, m = sb(g), k = g & 1, r = nth(g), y = nth(g & 7);
    mfma<keep, (Q >> 1) * 16 + m * 4 + a, (keep ? FB1 : FB0) + (a * 2 + sk(g)) * 4, (Q >> 1 ? FA1 : FA0) + (m * 2 + sk(g)) * 4>();
    if constexpr (Q == 0 && r >= 0 && r < 4) rdB<FB1, 12 + r, cur, 2 * U>();
    if constexpr (Q == 0 && r >= 4 && r < 11) rdB<FB0, 5 + r, VZ, P * U>();
    if constexpr (Q == 1 && g < 16 && !k) rdA<FA1, g / 2, cur, U>();
    if constexpr (Q == 2 && g < 16 && !k) rdA<FA0, g / 2, nxt, 0>();
    if constexpr (Q == 3 && a > 0 && y >= 0 && y < 4) rdB<FB1, (a - 1) * 4 + y, nxt, 2 * U>();
    if constexpr (Q == 3 && a < 3 && y >= (a ? 4 : 1)) rdB<FB0, a * 2 + y - 1, VZ, (P ^ 1) * U>();
    if constexpr (Q == 0 && g == 15) wait<-1, 7>();
    if constexpr (Q == 3 && g == 0) { if constexpr (ROLE % 4) Y("ds_read_b32 v66, v68 offset:8"); else Y("ds_read_b96 v[64:66], v68"); }
    if constexpr (Q == 3 && g == 8) wait<-1, 6>();
    if constexpr (Q == 3 && g == 10) Y("v_readfirstlane_b32 " sMORE ", v66");
    if constexpr (ROLE % 4 == 0) { constexpr int e = early(P, Q, g), f = early(P, Q, g + 1);
      if constexpr (!ROLE && g % 16 == 0) fm0<P, Q, g / 4>();
      if constexpr (!ROLE && g % 4 == 1) fill<Q, g / 4>();
      if constexpr (ROLE && f >= 0 && f % 4 == 0) fm0<(f / 8 + 5) / 4 % 2, (f / 8 + 1) % 4, f % 8>();
      if constexpr (ROLE && e >= 0) { if constexpr (e < 56) fill<(e / 8 + 1) % 4, e % 8, 93 + e / 8>(); else fill<(e / 8 + 1) % 4, e % 8>(); }
      if constexpr (Q == 0 && g == 22) Y("s_xor_b32 s79, s78, " sAH);
      if constexpr (Q == 3 && g == 11) Y("v_readfirstlane_b32 s78, v64");
      if constexpr (Q == 3 && g == 12) Y("v_readfirstlane_b32 s80, v65");
      if constexpr (Q == 3 && g == 14) Y("s_xor_b32 s80, s80, s86");
      if constexpr (Q == 3 && g == 15) Y("s_xor_b32 s81, s80, 0x100");
    }
    if constexpr (ROLE == 1 && Q == 1 && g == 17) Y("v_readlane_b32 s87, v83, " sT);
    if constexpr (ROLE == 1 && Q == 1 && g == 23) Y("buffer_load_dword v91, v92, s[56:59], s87 offen");
    if constexpr (ROLE == 1 && !(Q & 1) && g == 26) Y("s_lshl_b64 exec, 1, " sT);
    if constexpr (ROLE == 1 && !(Q & 1) && g == 27) X("buffer_atomic_add v[%0], v[%1], " sWS ", " cCM " offen sc0" N "s_mov_b64 exec, -1", "n"(Q ? 69 : 78), "n"(70 + Q / 2));
    if constexpr (ROLE == 1 && Q == 1) {
      if constexpr (g == 24) Y("v_cmp_le_u32 vcc, " cTK ", v78" N "v_cmp_le_u32 s[84:85], " cTK ", v69");
      if constexpr (g == 26) Y("s_bcnt1_i32_b64 s86, vcc" N "s_bcnt1_i32_b64 s84, s[84:85]");
      if constexpr (g == 27) Y("s_lshr_b32 s86, " cCL ", s86" N "s_lshr_b32 s86, s86, s84");
      if constexpr (g == 28) Y("s_and_b32 s93, s86, " cCL N "s_cselect_b32 s92, 0, 1");
    }
    if constexpr (ROLE == 1 && Q == 2) {
      if constexpr (g == 7) { wait<2, -1>(); Y("v_min3_u32 v77, v72, v73, v74"); }
      if constexpr (g == 8) Y("v_min_u32 v77, v77, v75");
      if constexpr (g == 9) Y("v_cmp_le_u32 vcc, v76, v77");
      if constexpr (g == 11) Y("s_or_b64 " cK ", " cK ", vcc" N "s_cmp_eq_u64 " cT ", 0");
      if constexpr (g == 13) Y("s_cselect_b64 " cT ", " cC ", " cT N "s_cselect_b32 " cCL ", 0, " cCL);
      if constexpr (g == 15) Y("s_cmp_eq_u64 " cK ", -2");
      if constexpr (g == 16) Y("s_cselect_b64 " cC ", 63, 62" N "s_or_b64 " cC ", " cC ", " cK);
      if constexpr (g == 17) X("s_and_b64 " cC ", " cC ", " cT N "s_cbranch_scc0 .Lc_slow\n.Lc_back%0:", "n"(P));
      if constexpr (g == 18) Y("s_ff1_i32_b64 " cB ", " cC N "s_lshl_b64 " cC ", 1, " cB);
      if constexpr (g == 19) Y("s_andn2_b64 " cT ", " cT ", " cC);
      if constexpr (g == 20) Y("v_cndmask_b32 v88, v89, v68, " cC N "v_cndmask_b32 v82, v82, v90, " cK);
      if constexpr (g >= 21 && g < 24) X("ds_write_b32 v88, v[%0] offset:%1", "n"(79 + (g - 20) % 3), "n"((g - 21) * 4));
      if constexpr (g == 25) Y(SAMPLE);
    }
    if constexpr (ROLE == 2 && P * 4 + Q <= FPQ) {
      if constexpr (g == 5) { wait<(P * 4 + Q < FPQ ? 15 : FPQ + 8), -1>(); Y("s_getreg_b32 s84, hwreg(HW_REG_IB_STS, 0, 4)"); }
      if constexpr (g == 9) X("s_or_b32 s84, s84, " pB N "s_cmp_le_u32 s84, %0", "n"(P * 4 + Q + 8));
      if constexpr (g == 13) Y("s_cselect_b32 s94, 4, 0" N "s_cselect_b32 " pB ", 64, " pB);
      if constexpr (g == 17) Y("buffer_store_dword v81, off, " pF ", 0 sc0 sc1");
    }
    if constexpr (ROLE == 3) { constexpr int s = g == 1 ? P * 4 + Q : -1, l = P && Q == 3 && g % 4 == 1 ? g / 4 : -1;
      if constexpr (s == 0) wait<FPQ + 1, -1>();
      if constexpr (s >= 0) X("global_store_dwordx4 v56, v[%0:%0+3], " pD " offset:%1 sc0 sc1", "n"(96 + s * 4), "n"(s * 1024 - 4096));
      if constexpr (s == 0) Y("s_cmp_gt_u32 " pL ", 1" N "s_cselect_b32 s85, " pCS ", 0" N "s_cselect_b64 vcc, -1, 0");
      if constexpr (!P && !Q && g == 2) Y("s_add_u32 s88, s88, s85");
      if constexpr (!P && !Q && g == 3) Y("s_addc_u32 s89, s89, 0");
      if constexpr (l >= 0) X("global_load_dwordx4 v[%0:%0+3], v[%1], " pXN " nt", "n"(96 + l * 4), "n"(88 + l));
      if constexpr (s == 7) X("s_add_u32 s90, s90, %0" N "s_addc_u32 s91, s91, 0", "n"(2 * RCOL));
      if constexpr (P && Q == 1 && g == 17) X("s_add_i32 " pS ", " pS ", %0" N "s_ashr_i32 s84, " pS ", 5", "n"(PER));
      if constexpr (P && Q == 1 && g == 18) Y("s_and_b32 s84, s84, -2" N "s_add_u32 " sEV ", " sEV ", 4");
      if constexpr (P && Q == 1 && g == 19) Y("s_max_i32 " sEV ", " sEV ", s84" N "s_xor_b32 " pCS ", " pCS ", 0x100");
      if constexpr (P && Q == 1 && g == 20) Y("s_orn2_b32 " sEV ", " sEV ", vcc_lo" N "s_sub_u32 " pL ", " pL ", 1");
      if constexpr (P && Q == 1 && g == 21) Y("s_mov_b32 " pB ", 0" N "v_add_u32 v81, 1, v81");
    }
    if constexpr (!(Q & 1) && g == 30) wait<ROLE % 4 ? -1 : !ROLE ? 32 : P * 12 + Q * 4 + 12, 0>();
    if constexpr (!(Q & 1) && g == 31) Y("s_barrier");
    if constexpr (Q == 1 && g == 31) wait<-1, 0>();
    if constexpr (Q == 3 && g == 18) Y("s_add_u32 " sT ", " sT ", 1");
    if constexpr (Q == 3 && g == 19) Y("s_cmp_eq_u32 " sT ", " sEV);
    if constexpr (Q == 3 && g == 20) Y("s_cselect_b32 s84, 0, " sMORE);
    if constexpr (Q == 3 && g == 22) Y("s_cmp_lg_u32 s84, 1");
    if constexpr (Q == 3 && g == 30) wait<-1, 6>();
  ));
}
#define SLOOP(role, l, ev) do { Y(l "0:"); stile<role, 0>(); Y("s_cbranch_scc1 " ev "\n" l "1:"); stile<role, 1>(); Y("s_cbranch_scc0 " l "0"); } while (0)
#define BACK(l) "s_bitcmp1_b32 " sT ", 0" N "s_cbranch_scc1 " l "1" N "s_branch " l "0"
#define LEAVE "s_cmp_lg_u32 " sMORE ", 1" N "s_cbranch_scc1 .Lexit" N

enum { NONE, RA, RB, M0, DA, DB, WT, BR, RN, ZB, XS, PB, FL, FC, PR, AD, OU, NP, ID, OA };
struct Op { short k, a, b, c, d; };
constexpr int NQ = 8 + 2 * RL, NG = NQ * 32, L = RL - 1, NX = NG + 48;
struct Plan {
  Op mf[NG] = {}, op[NX][8 * AB + 6] = {};
  char n[NX] = {}, room[NX] = {};
  bool ok = true;
  short mark[160] = {};
  static constexpr bool st4(const Op& o) { return (o.k == XS && o.a % 13 == 12) || (o.k == OU && o.b == 5); }
  static constexpr bool vm(const Op& o) { return o.k == DA || o.k == DB || o.k == FL || o.k == PR || st4(o) || (o.k == PB && o.a); }
  static constexpr bool ds(const Op& o) { return o.k == RA || o.k == RB || (o.k == OU && (o.b == 2 || o.b == 3)); }
  static constexpr bool vs(const Op& o) { return o.k == FC && o.a < 2; }
  static constexpr bool va(const Op& o) { return vs(o) || o.k == ZB || o.k == ID || ((o.k == XS || o.k == PB || o.k == OU) && !vm(o) && !ds(o)); }
  static constexpr bool wr(const Op& o) { return (o.k == PB && o.a) || (o.k == OU && o.b == 2); }
  constexpr bool fits(int g, const Op& o) {
    if (n[g] >= 2 + room[g]) return false;
    if (n[g] && op[g][n[g] - 1].k == AD) return va(o) || (ds(o) && !wr(o));
    if (st4(o)) return !n[g] || (n[g] == 1 && op[g][0].k == WT);
    if (g < NG - 128 && (o.k == RA || o.k == RB) && g % 4 == 1) return false;
    for (int j = 0; j < n[g]; ++j) { const Op& p = op[g][j];
      if (wr(p) || ((vm(o) || ds(o)) && (va(p) || p.k == BR || ds(p) || (vm(p) && !st4(p)))) || (vs(p) && !va(o) && o.k != WT)) return false; }
    return true;
  }
  constexpr int put(int g, int k, int a = 0, int b = 0, int c = 0, int d = 0) { while (!fits(g, Op{(short)k, (short)a, (short)b, (short)c, (short)d})) ++g; return at(g, k, a, b, c, d); }
  constexpr int at(int g, int k, int a = 0, int b = 0, int c = 0, int d = 0) { const Op o = {(short)k, (short)a, (short)b, (short)c, (short)d}; if (st4(o)) room[g] = 2 + n[g]; if (k == AD) room[g] = n[g]; op[g][(int)n[g]++] = o; return g; }
  constexpr void mm(int q, int keep, int half, int fb, int fa) {
    for (int g = q * 32; g < q * 32 + 32; ++g) { const int nn = (g >> 3) & 3, m = sb(g), k = sk(g);
      mf[g] = Op{(short)keep, (short)(half * 16 + m * 4 + nn), (short)(fb + (nn * 2 + k) * 4), (short)(fa + (m * 2 + k) * 4), 0}; }
  }
  static constexpr int set(int r) { return r % 3 == 0 ? VX : r % 3 == 1 ? VY : VZ; }
  constexpr void ra(int g, int st, int f, int s, int u, int tag) { for (int i = 0; i < 8; ++i) put(g + i * st, RA, tag, f * 16 + i, s, u); }
  constexpr void rb(int g, int i0, int i1, int f, int s, int u, int tag) { for (int i = i0; i < i1; ++i) put(g + i - i0, RB, tag, f * 16 + i, s, u); }
  constexpr void jit(int q, int s) { for (int j = 0; j < 4; ++j) rb(q * 32 + 8 + j * 8, j * 4, j * 4 + 4, FB1, s, 2, 90 + (j >> 1)); }
  constexpr void sync(int q, int t0, int t1, int lt) { at(q * 32 + 31, WT, t0, t1, lt); at(q * 32 + 31, BR); }
  int nf = 0;
  constexpr void fill(int g, int skip, int u, int r) {
    if (r > L) return;
    const int so = SA0 + (nf++ & 1), tag = (u == 2 ? 20 : u * 40) + r;
    ok = ok && put(g - 7, RN, so, u, r) <= g - 4 && (u < 2 || put(g - 3, RN, so, 3) < g) && put(g - 2, M0, r % 3 < 2 ? SLX + r % 3 : SLW, u + r % 3 / 2 * 6) < g;
    for (int i = 0; i < 4; g += 4) if (!((skip >> (g / 4 % 4)) & 1)) ok = ok && put(g, u == 2 ? DB : DA, tag, i++, so) == g;
  }
  constexpr int count(bool fix, int tag) {
    int nv = 0, nl = 0;
    for (int i = 0; i < 160; ++i) mark[i] = 30000;
    mark[60] = -24; mark[61] = -8; mark[62] = 0;
    for (int g = 0; g < NX; ++g) for (int j = 0; j < n[g]; ++j) { Op& o = op[g][j];
      if (vm(o)) { ++nv; mark[o.k == XS ? (o.a == 207 ? 64 : 159) : o.k == FL ? 65 : o.k == PR ? 70 + o.a : (o.k == DA || o.k == DB) && o.b == 3 ? o.a : 159] = nv; }
      if (ds(o)) { ++nl; mark[o.k == OU ? 100 + o.a : o.a] = nl; }
      if (o.k == WT) {
        const int v = o.a < 0 ? -1 : nv - (o.b >= 0 && mark[o.b] > mark[o.a] ? mark[o.b] : mark[o.a]), l = o.c < 0 ? -1 : o.c == 0 ? 0 : nl - mark[o.c];
        if (!fix && o.a >= 0 && v <= nv - mark[tag]) return g;
        if (fix) { ok = ok && (o.a < 0 || (v >= 0 && v < 64)) && (o.c < 0 || l >= 0); o.a = v; o.c = l > 14 ? 14 : l; }
      } }
    return -1;
  }
  int st = 0, lw[32] = {}, fin[32] = {}, ad[32] = {}, io = 0, no = 0;
  Op ou[200] = {};
  constexpr void out(int g) {
    for (bool go = true; go && io < no; ) {
      go = false;
      for (int x = io; x < no && x < io + 12 && !go; ++x) { const Op o = ou[x]; const int T = o.a * 4 + o.c;
        if (o.k == NONE || (x > io && !va(o))) continue;
        if (o.k == OU && o.b == 0 && !(g >= fin[T] && st - lw[T] >= 8 && ad[T])) {
          if (x == io && g >= NG && ad[T]) { at(g, NP, 7 - (st - lw[T])); st += 8 - (st - lw[T]); go = true; }
          break; }
        if (g < NG ? fits(g, o) && (o.k != WT || !n[g]) : x == io && n[g] < 2) { at(g, o.k, o.a, o.b, o.c, o.d); ou[x].k = NONE; ++st; go = true; }
        else if (va(o)) break;
      }
      while (io < no && ou[io].k == NONE) ++io;
    }
  }
  constexpr Plan() {
    sync(0, 60, -1, 0); sync(2, 61, -1, 0); sync(4, 62, 0, 0); sync(6, 20, -1, 0); sync(7, 40, -1, 90);
    for (int r = 0; r < L - 1; ++r) { sync(8 + 2 * r, r + 1, 21 + r, 0); if (r < L - 2) sync(9 + 2 * r, 41 + r, -1, 90); else sync(9 + 2 * r, L, 20 + L, 90); }
    sync(NQ - 4, 40 + L, -1, 97);
    rb(0, 12, 16, FB1, VX, 2, 91); rb(4, 9, 16, FB0, VZ, 0, 92);
    for (int t = 0; t < 2; ++t) { const int q = t * 4;
      mm(q, 1, 0, FB1, FA0); mm(q + 1, 0, 0, FB0, FA0); mm(q + 2, 0, 1, FB0, FA1); mm(q + 3, 1, 1, FB1, FA1);
      ra(q * 32 + 32, 2, FA1, set(t), 1, 92); ra(q * 32 + 64, 2, FA0, set(t ^ 1), 0, 92); jit(q + 3, set(t ^ 1));
      if (!t) rb(98, 0, 16, FB0, VZ, 0, 92);
      at(q * 32 + 63, WT, -1, -1, 0);
    }
    for (int r = 0; r < L - 1; ++r) { const int q = 8 + 2 * r;
      mm(q, 1, 0, FB1, FA0); mm(q + 1, 1, 1, FB1, FA1);
      jit(q + 1, set(r + 1)); ra(q * 32 + 6, 2, FA1, set(r), 1, 92); ra(q * 32 + 32, 1, FA0, set(r + 1), 0, 92);
    }
    for (int g = 0; g < 128; ++g) { const int k = g & 1, m = g < 24 ? g % 6 / 2 : g < 48 ? g / 8 - 3 : g / 16, nn = g < 24 ? g / 6 : g < 48 ? g / 2 % 4 : g / 4 % 4, s = g < 48 ? g >= 24 : g / 2 % 2;
      mf[NG - 128 + g] = Op{1, (short)(m * 4 + nn), (short)((s ? FB0 : FB1) + (nn * 2 + k) * 4), (short)((m < 4 ? s ? FA2 : FA0 : s ? FA3 : FA1) + (m % 4 * 2 + k) * 4), 0}; }
    for (int i = 0; i < 8; ++i) { put(NG - 124, RA, 97 + i / 6, FA2 * 16 + i, set(L), 0); if (i == 1) { rb(NG - 124, 0, 8, FB0, set(L), 2, 92); rb(NG - 124, 8, 16, FB0, set(L), 2, 89); } }
    for (int i = 0; i < 16; ++i) put(NG - 84 + (i >> 2) * 16 + (i & 3), RA, 93 + (i >> 2), (i & 2 ? FA3 : FA1) * 16 + (i >> 2) * 2 + (i & 1), set(L - 1 + ((i >> 1) & 1)), 1);
    at(15, WT, -1, -1, 91); at(127, WT, -1, -1, 90); at(143, WT, -1, -1, 91);
    for (int q = 8; q < NQ - 4; q += 2) at(q * 32 + 15, WT, -1, -1, 91);
    at(NG - 117, WT, -1, -1, 91); at(NG - 105, WT, -1, -1, 92); at(NG - 101, WT, -1, -1, 89); at(NG - 81, WT, -1, -1, 98);
    for (int i = 0; i < 4; ++i) at(NG - 65 + i * 16, WT, -1, -1, 93 + i);
    fill(9, 2, 0, 0); fill(33, 10, 2, 0); fill(65, 10, 2, 2); fill(97, 10, 1, 0); fill(129, 10, 0, 1); fill(161, 10, 2, 1); fill(193, 10, 0, 2);
    for (int r = -1; r < L; ++r) { const int g = (9 + 2 * r) * 32; if (r < 0) fill(g + 1, 8, 1, 1); else fill(g + 1, 8, 2, r + 3); fill(g + 21, 8, 1, r + 3); fill(g + 41, 8, 0, r + 4); }
    for (int t = 0; t < 2; ++t) for (int i = 0; i < 4; ++i) put(t * 128 + 50 + 2 * i, ZB, i, t);
    { int g = 6 * 32 + 2;
      for (int i = 0; i < 208; ++i) {
        if (i % 13 == 12) { g += 1; while (g % 4 != 1 || n[g] || g < 6 * 32 + 13 + i / 13 * XQ) ++g; }
        g = put(g, XS, i); ok = ok && (i < 104 || g >= 7 * 32 + 8);
      } }
    { int g = count(false, 64); ok = ok && g > 0; g = put(g - 3, PB, 0) + 4; while (n[g] > 1 || va(op[g][0])) ++g; at(g, PB, 1); ok = ok && g < NG - FG;
      for (int i = 0; i < 21; ++i) g = put(g + 1, OA, i);
      for (int i = 0; i < 4; ++i) g = put(g + 1, ID, i); }
    int gp[16] = {};
    { int g = put(NG - FG, FL, 1); ok = ok && put(g - 16, FL, 0) < g; g += LAGF;
      g = put(g, WT, 65, -1, -1); g = put(g, FC, 0); g = put(g + 1, FC, 1); ++g;
      do ++g; while (n[g] || g % 4 == 1);
      at(g, FC, 2); at(g, NONE);
      for (int i = 0; i < 16; ++i) { do ++g; while (g % 4 != 1 || n[g] > 1 || vm(op[g][0]) || va(op[g][0])); gp[i] = at(g, PR, i); } }
    for (int G = 0; G <= 8; ++G) for (int c = 0; c < 4; ++c) {
      if (G < 8) for (int e = 0; e < 4; ++e) ou[no++] = Op{OU, (short)G, (short)(e >> 1), (short)c, (short)(e & 1)};
      if (G > 0 && c == 2) ou[no++] = Op{WT, -1, -1, (short)(99 + G), 0};
      if (G > 0 && c >= 2) ou[no++] = Op{OU, (short)(G - 1), 5, (short)(c - 2), 0};
      if (G < 8) ou[no++] = Op{OU, (short)G, 2, (short)c, 0};
      if (G < 8 && c == 3) for (int e = 0; e < 2; ++e) ou[no++] = Op{OU, (short)G, 3, (short)e, 0};
    }
    for (int g = 0; g < NG; ++g) if (mf[g].k) fin[mf[g].a] = g;
    for (int g = 0; g < NX; ++g) {
      if (g < NG) { ++st; if (mf[g].k) lw[mf[g].a] = st; }
      st += n[g];
      out(g);
      if (g < NG - 4 && !room[g] && (!n[g] || (op[g][n[g] - 1].k != BR && op[g][0].k != BR))) {
        int c = 0, u[8] = {}, f = 0;
        while (f < 8 && ad[f * 4]) ++f;
        for (int G = f; G < 8 && c < AB; ++G) {
          bool y = !ad[G * 4] && g >= gp[G * 2 + 1] + LAGP;
          for (int i = 0; i < 4; ++i) y = y && st + 1 + c * 4 + i - lw[G * 4 + i] >= 6 && mf[g + 1 + i].a / 4 != G;
          if (y) u[c++] = G; }
        if (c && u[0] == f && (c > 1 || (g >= fin[f * 4 + 3] && io >= f * 25 - 14) || f == 7)) {
          at(g, WT, 71 + u[c - 1] * 2, -1, -1); ++st;
          for (int i = 0; i < c * 4; ++i) { const int T = u[i / 4] * 4 + i % 4; at(g, AD, T); lw[T] = ++st - 3; ad[T] = 1; out(g); } }
      }
    }
    ok = ok && io == no;
    count(true, 0);
  }
};
constexpr Plan PLAN;
static_assert(PLAN.ok, "");
template <int k, int a, int b, int c, int d> DI void emit() {
  if constexpr (k == RA) rdA<b / 16, b % 16, c, d * U>();
  if constexpr (k == RB) rdB<b / 16, b % 16, c, d * U>();
  if constexpr (k == M0) m0<a, b * U>();
  if constexpr (k == DA) dmaA<VRA + (b & 1), SWS, c, b>();
  if constexpr (k == DB) dmaB<VRB + b, c, b>();
  if constexpr (k == WT) wait<a, c>();
  if constexpr (k == BR) Y("s_barrier");
  if constexpr (k == RN && b < 3) X("s_add_u32 s[%0], s[%1], %2", "n"(a), "n"(SRA + b), "n"(b < 2 ? offA(rsrc(c), rcol(c), ~c & 1) : offW(rsrc(c), rcol(c), ~c & 1)));
  if constexpr (k == RN && b == 3) X("s_and_b32 s[%0], s[%0], 0x7ffffff", "n"(a));
  if constexpr (k == ZB) { if constexpr (b) X("v_and_b32 v[%0], 0xffffbfff, v[%0]", "n"(VZ + 2 + a)); else X("v_xor_b32 v[%0], 0x4000, v[%0]", "n"(VZ + 2 + a)); }
  if constexpr (k == XS) { constexpr int i = a / 13, e = a % 13, o = 64 + (i & 1) * 4;
    if constexpr (e == 12) X("global_store_dwordx4 v[%0], v[%1:%1+3], " sXD " offset:%2 sc0 sc1", "n"(56 + (i >> 3)), "n"(o), "n"((i & 7) * 1024 - 4096));
    else if constexpr (e >= 8) X("v_cvt_pk_bf16_f32 v[%0], v[%1], v[%1+1]", "n"(o + e - 8), "n"((e - 8) * 2));
    else X("v_accvgpr_read_b32 v[%0], a[%1]", "n"(e), "n"(i * 8 + e)); }
  if constexpr (k == PB) { if constexpr (a) Y("buffer_store_dword v62, off, " sWS ", " sXP " sc0 sc1"); else Y("v_mov_b32 v62, " sCALL); }
  if constexpr (k == FL) { if constexpr (a) Y("buffer_load_dword v61, off, " sWS ", " sXF " sc0 sc1"); else Y("buffer_load_dword v60, off, " sWS ", 0 sc1"); }
  if constexpr (k == FC) {
    if constexpr (a == 0) Y("v_readfirstlane_b32 " sMORE ", v60");
    if constexpr (a == 1) Y("v_readfirstlane_b32 s84, v61");
    if constexpr (a == 2) Y("s_cmp_lt_u32 s84, " sCALL N "s_cbranch_scc1 .Lx_slow\n.Lx_back:"); }
  if constexpr (k == PR) X("buffer_load_dwordx4 v[%0:%0+3], v56, " sWS ", s[%2] offen offset:%1 sc0 sc1", "n"(64 + a * 4), "n"((a & 3) * 1024), "n"(24 + a / 4));
  if constexpr (k == AD) X("v_mfma_f32_4x4x4_16b_bf16 v[%0:%0+3], v[8:9], v[%1:%1+1], v[%0:%0+3]", "n"(128 + a * 4), "n"(64 + a * 2));
  if constexpr (k == ID) { if constexpr (a == 0) Y("v_and_b32 v10, 48, v56"); if constexpr (a == 1) Y("v_mov_b32 v8, 0x3f80"); if constexpr (a == 2) Y("v_mov_b32 v9, 0");
    if constexpr (a == 3) Y("v_lshlrev_b64 v[8:9], v10, v[8:9]"); }
  if constexpr (k == OA && a == 20) Y("s_mov_b32 " sT0 ", 0");
  if constexpr (k == OA && a >= 16 && a < 20) X("s_add_u32 s[%0], " sXS ", %1", "n"(8 + a), "n"((a - 16) * 4096));
  if constexpr (k == OA && a < 16) X("s_xor_b32 s[%0], " sOF ", %1", "n"(8 + a), "n"(((a >> 3) * 128 + (a & 7) * 8) * (NCOL * 2)));
  if constexpr (k == OU) { constexpr int t = c * 2 + d, r = 128 + (a * 4 + c) * 4 + d * 2, o = FO + (a * 2 + c) * 4;
    if constexpr (b == 0) X("v_cvt_pk_bf16_f32 v[%0], v[%1], v[%1+1]", "n"(t), "n"(r));
    if constexpr (b == 1) X("v_pk_max_i16 v[%0], v[%0], 0", "n"(t));
    if constexpr (b == 2) X("ds_write_b64 v[%0], v[%1:%1+1] offset:%2", "n"(24 + c), "n"(t), "n"((a & 1) * 2048));
    if constexpr (b == 3) X("ds_read_b128 a[%0:%0+3], v28 offset:%1", "n"(o), "n"((a & 1) * 2048 + c * 1024));
    if constexpr (b == 5) X("buffer_store_dwordx4 a[%0:%0+3], v29, " sOUT ", s[%1] offen sc1", "n"(d ? FO : o), "n"(8 + a * 2 + c)); }
  if constexpr (k == NP) X("s_nop %0", "n"(a));
}

template <int i> DI void zero() {
  if constexpr (i & 1) X("v_mfma_f32_32x32x4_2b_bf16 v[%0:%0+31], v[60:61], v[60:61], 0", "n"(128 + i / 2 * 32));
  else X("v_mfma_f32_32x32x4_2b_bf16 a[%0:%0+31], v[60:61], v[60:61], 0", "n"(i / 2 * 32));
}
#define CALL(n) "s_waitcnt vmcnt(" #n ")" N "v_readfirstlane_b32 s4, v64" N "s_add_u32 " sCALL ", s4, 1" N "s_lshl_b32 s5, " sCALL ", 24" N "s_and_b32 s5, s5, 0x1000000" N \
    "s_add_u32 s44, s44, s5" N "s_xor_b32 s45, s44, 0x4000" N
template <int V> DI void go() {
  wait<V, -1>(); Y("s_barrier");
  REP(i, 0, 8, rdA<FA0, i, VX, 0>()); REP(i, 0, 12, rdB<FB1, i, VX, 2 * U>()); REP(i, 0, 9, rdB<FB0, i, VZ, 0>());
}
#define TAKE(n) "v_mov_b32 v78, " #n N "buffer_atomic_add v78, v93, " sWS ", s22 offen sc0" N

__global__ __launch_bounds__(256, 1) void a2a_gemm(Params) {
  asm volatile("s_load_dwordx16 s[52:67], s[0:1], 0" N "s_getreg_b32 s3, hwreg(HW_REG_XCC_ID, 0, 3)" N "s_and_b32 s4, s2, 0xf8" N "s_mov_b32 s15, 0"
    :: "{s[0:1]}"(__builtin_amdgcn_kernarg_segment_ptr()), "{s2}"(blockIdx.x), "{v0}"(threadIdx.x) : "v255", "a255", "s101", "vcc", "memory");
  X("s_or_b32 s3, s3, s4\n.Lstart:" N "s_and_b32 s4, s3, 4" N "s_lshr_b32 s5, s3, 6" N "s_add_u32 s4, s4, s5" N "s_lshl_b32 s4, s4, 5" N
    "s_load_dwordx8 s[88:95], s[0:1], s4 offset:0x40" N "s_getpc_b64 s[16:17]\n.Lpc:" N "v_readfirstlane_b32 s14, v0" N "v_mov_b64 v[60:61], 0" N
    "v_lshrrev_b32 v9, 4, v0" N "s_and_b32 s4, s3, 0xf8" N "s_lshl_b32 s4, s4, 8" N "v_lshl_add_u32 v5, v9, 7, s4" N "v_min_u32 v5, .Lend - .Lpc - 4, v5" N
    "global_load_dword v65, v5, s[16:17]" N "s_bfe_u32 s9, s3, 0x10002" N "s_and_b32 s13, s3, 3" N "s_bfe_u32 s10, s3, 0x20003" N
    "s_xor_b32 s12, s13, s9" N "s_lshr_b32 s5, s3, 5" N "s_lshl3_add_u32 s12, s12, s5" N "s_lshl_b32 s12, s12, 9" N "s_and_b32 s11, s5, 1" N
    "s_lshl_b32 s4, s9, 5" N "s_or_b32 s4, s4, s10" N "s_lshl_b32 s4, s4, 18" N "s_or_b32 s4, s4, s12" N
    "v_lshl_add_u32 v9, v9, 14, s4" N "v_bfe_u32 v6, v0, 2, 2" N "v_lshl_add_u32 v9, v6, 7, v9" N
    "s_lshl3_add_u32 s4, s10, s5" N "s_lshl_b32 s4, s4, 16" N "s_lshl_b32 s7, s9, 10" N "s_or_b32 s4, s4, s7" N
    "v_lshrrev_b32 v10, 3, v0" N "v_lshl_add_u32 v10, v10, 11, s4" N "v_and_b32 v6, 1, v6" N "v_lshl_add_u32 v10, v6, 7, v10" N
    "s_lshr_b32 s8, s14, 6" N "s_lshl_b32 s4, s2, 5" N "s_add_u32 s4, s4, s8" N "s_lshl2_add_u32 " sCTR ", s4, %0" N
    "v_and_b32 v1, 63, v0" N "v_lshlrev_b32 v56, 4, v1" N "v_lshrrev_b32 v4, 3, v1" N "v_and_b32 v7, 0x70, v56", "n"(WS_CTR));
  zero<0>();
  Y("s_waitcnt lgkmcnt(0)");
  Y("s_andn2_b32 s47, s94, 7");
  X("buffer_load_dword v64, off, " sWS ", " sCTR " sc1" N "s_cmp_lt_u32 s8, 2" N "s_cbranch_scc1 .Lall" N "s_cmp_eq_u32 s8, 3" N "s_cbranch_scc1 .Lp_a" N
    "s_lshr_b32 s4, s3, 1" N "s_xor_b32 s4, s4, s3" N "s_and_b32 s4, s4, 2" N "s_lshl_b32 s4, s4, 9" N "v_or_b32 v5, s4, v56" N "v_xor_b32 v6, 0x400, v5" N
    "v_add_u32 v5, .Ltab - .Lpc, v5" N "v_add_u32 v6, .Ltab - .Lpc, v6" N "global_load_dwordx4 v[80:83], v5, s[16:17]" N "global_load_dwordx4 v[84:87], v6, s[16:17]" N
    "v_min_u32 v79, 1, v1" N "v_lshlrev_b32 v93, 30, v79" N "s_lshl_b32 s22, s3, 7" N "s_or_b32 s22, s22, s15" N "s_add_u32 s22, s22, %0" N TAKE(1)
    "v_add_u32 v89, %1 + 16, v56" N "s_branch .Lall\n"
    ".Lp_a:" N "s_lshl2_add_u32 s20, s10, s13" N "s_lshl_b32 s4, s20, 9" N "s_or_b32 s4, s4, s11" N "s_lshl_b32 s4, s4, 8" N
    "v_lshl_add_u32 v88, v4, 11, s4" N "v_add_u32 v88, v88, v7", "n"(WS_ROLE), "n"(LDS_CTRL));
  REP(i, 0, 8, if constexpr (i < 7) X("v_add_u32 v[%0], %1, v88", "n"(89 + i), "n"((i + 1) * 16384)); X("global_load_dwordx4 v[%0:%0+3], v[%1], " pXN " nt", "n"(96 + i * 4), "n"(88 + i)));
  Y(".Lall:");
  zero<1>();
  Y("s_brev_b32 s14, s14" N "s_lshr_b32 s77, s14, 12" N "s_lshr_b32 s4, s14, 8" N
    "v_lshrrev_b32 v3, 4, v1" N "v_bitop3_b32 v2, v1, v3, 7 bitop3:0x6c" N "v_lshl_add_u32 v72, v4, 11, s4" N
    "v_lshl_add_u32 v72, v2, 4, v72" N "v_xor_b32 v5, 64, v72" N "v_add_u32 v73, 0x3c00, v5" N "v_add_u32 v74, 0x7800, v72" N "v_add_u32 v75, 0xb400, v5" N
    "s_lshr_b32 s4, s14, 6" N "s_add_u32 s4, s4, s12" N "v_bfe_u32 v6, v1, 1, 3");
  zero<2>();
  Y("v_xor_b32 v6, v6, v3" N "v_lshl_add_u32 v52, v3, 14, s4" N "v_lshl_add_u32 v52, v6, 5, v52" N
    "v_and_or_b32 v52, v56, 16, v52" N "v_xor_b32 v5, 0x80, v52" N "v_add_u32 v53, 0xfc00, v52" N "v_add_u32 v54, 0x1f800, v5" N "v_add_u32 v55, 0x2f400, v5" N
    "s_lshl1_add_u32 s4, s10, s11" N "s_lshl_b32 s4, s4, 18" N "s_or_b32 s78, s4, s7" N "s_mov_b32 m0, s77" N
    "s_xor_b32 s4, s10, s8" N "s_and_b32 s4, s4, 1");
  zero<3>();
  Y("s_lshl_b32 s18, s4, 16" N "s_or_b32 s18, s18, s77" N "s_bitset1_b32 s18, 15" N
    "s_xor_b32 s4, s4, s9" N "s_lshl_b32 s4, s4, 8" N "s_lshl_b32 s5, s9, 23" N "s_or_b32 s4, s4, s5" N "s_or_b32 s80, s47, s4" N "s_xor_b32 s81, s80, 0x100");
  REP(i, 0, 4, fill<0, i>());
  { Y("s_mov_b32 m0, s18" N "s_nop 0"); REP(i, 0, 4, fill<1, i>()); }
  { Y("s_xor_b32 m0, s18, 0x10000" N "s_nop 0"); REP(i, 0, 4, fill<2, i>()); }
  Y("s_bfe_u32 s6, s8, 0x10001" N "s_and_b32 s7, s8, 1" N "s_lshl_b32 s7, s7, 7" N "s_lshl_b32 s4, s6, 13" N
    "v_and_b32 v12, 15, v1" N "v_lshl_add_u32 v32, v6, 4, s4" N "v_lshl_add_u32 v32, v12, 7, v32" N "v_xor_b32 v33, 64, v32" N
    "v_bfe_u32 v5, v1, 2, 3" N "v_lshlrev_b32 v34, 5, v5" N "v_xor_b32 v34, s7, v34" N "v_and_b32 v5, 3, v5" N "v_lshl_add_u32 v34, v5, 8, v34");
  zero<4>();
  Y("v_lshl_add_u32 v34, v3, 11, v34" N "v_and_b32 v14, 3, v1" N "v_lshl_add_u32 v34, v14, 3, v34" N "v_xor_b32 v35, 32, v34" N "v_xor_b32 v36, 64, v34" N "v_xor_b32 v37, 0x60, v34");
  REP(i, 0, 12, X("v_add_u32 v[%0], %1, v[%2]", "n"(VY + i), "n"((i / 6 + 1) * 3 * U), "n"(VX + i % 6)); if constexpr (i == 6) zero<5>());
  X("s_lshl_b32 s4, s8, 12" N "s_add_u32 s4, s4, %0" N "v_lshrrev_b32 v5, 5, v1" N "v_bitop3_b32 v5, v1, v5, 7 bitop3:0x6c" N "v_lshl_add_u32 v24, v12, 7, s4" N
    "v_lshl_add_u32 v24, v5, 4, v24" N "v_and_b32 v5, 1, v3" N "v_lshl_add_u32 v24, v5, 3, v24", "n"(9 * U));
  zero<6>();
  X("v_xor_b32 v25, 32, v24" N "v_xor_b32 v26, 64, v24" N "v_xor_b32 v27, 0x60, v24" N
    "v_bitop3_b32 v5, v1, v4, 7 bitop3:0x6c" N "v_lshl_add_u32 v28, v4, 7, s4" N "v_lshl_add_u32 v28, v5, 4, v28" N
    "s_lshl2_add_u32 s4, s10, s6" N "s_lshl_b32 s4, s4, 20" N "s_lshl_b32 s5, s9, 8" N "s_or_b32 s46, s47, s5" N "s_or_b32 s4, s4, s5" N "s_or_b32 s4, s4, s7" N "s_or_b32 s4, s4, s12" N
    "v_lshl_add_u32 v29, v4, 14, s4" N "v_add_u32 v29, v29, v7" N "v_mov_b32 v68, %0", "n"(LDS_CTRL));
  zero<7>();
  X("v_lshl_add_u32 v50, v4, 7, s77" N "v_lshl_add_u32 v50, v2, 4, v50" N "s_bfm_b64 exec, 8, 24" N "v_add_u32 v50, %0, v50" N "v_add_u32 v56, %0, v56" N "s_mov_b64 exec, -1" N
    "v_xor_b32 v51, 64, v50" N "v_add_u32 v57, 0x2000, v56" N
    "s_lshl2_add_u32 s4, s3, s8" N "s_lshl_b32 s4, s4, 14" N "s_add_u32 " sXS ", s4, %1" N "s_xor_b32 s4, " sXS ", 0x51000" N "s_add_u32 s68, s52, s4" N "s_addc_u32 s69, s53, 0" N
    "s_lshl_b32 s4, s3, 5" N "s_add_u32 s4, s4, s8" N "s_lshl2_add_u32 " sXF ", s4, %2" N "s_xor_b32 " sXP ", " sXF ", 0x280" N
    "s_lshl_b32 " sOF ", s11, 21" N "s_mov_b32 " sT ", 0" N "s_mov_b32 s74, s77" N "s_add_u32 s75, s77, 0xc000" N
    "s_lshl_b32 s21, s10, 19" N "s_lshl_b32 s4, s11, 14" N "s_or_b32 s4, s4, s21" N "s_add_u32 s44, s4, %3", "n"(AWAY), "n"(WS_XCH), "n"(WS_XFLAG), "n"(WS_A));
  Y("s_cmp_eq_u32 s8, 2" N "s_cbranch_scc1 .Lc_go" N "s_cmp_eq_u32 s8, 3" N "s_cbranch_scc1 .Lp_go" N "s_branch .Ld_go\n"
    ".p2align 6\n.fill ((.Ld_go - .Ld0) & 63) / 4, 4, 0xbf800000\n.Ld_go:");
  {
    REP(i, 0, 4, X("v_add_u32 v[%0+4], 0x10000, v[%0]" N "v_add_u32 v[%1], 0x40000, v[%2]", "n"(VDA + i), "n"(VDB + 4 + i), "n"(VRB + i)));
    Y("s_xor_b32 s79, s78, 0x40000" N "s_add_u32 s92, s78, 128" N "s_mov_b32 s88, s18" N "s_xor_b32 s89, s18, 0x10000" N
      "s_lshr_b32 s4, s18, 1" N "s_and_b32 s4, s4, 0x8000" N "s_or_b32 s4, s4, s77" N "s_or_b32 s90, s4, 0x14000" N "s_xor_b32 s91, s90, 0x8000" N
      "s_add_u32 s93, s80, 0x100000" N "s_add_u32 s94, s81, 0x100000" N "s_add_u32 s95, s79, 128" N "s_add_u32 s96, s78, 0x100" N
      "s_add_u32 s97, s80, 0x200000" N "s_add_u32 s98, s81, 0x200000" N CALL(12) "s_add_u32 m0, s77, 0x4000" N "s_add_u32 s99, s79, 0x100");
    REP(i, 0, 4, fill<0, i, SA1>());
    Y("s_add_u32 m0, s77, 0x5000" N "s_lshl_b32 s86, s8, 8");
    REP(i, 4, 8, fill<0, i, SA1>());
    Y("s_add_u32 m0, s77, 0xc000" N "s_mov_b32 " sEV ", 6");
    REP(i, 0, 4, fill<0, i, 92>());
    Y("s_add_u32 m0, s77, 0xd000" N "s_mov_b32 " sAH ", 0x40000");
    REP(i, 4, 8, fill<0, i, 92>());
    go<16>(); wait<-1, 13>();
    stile<4, 0>(); Y("s_cbranch_scc1 .Lexit"); stile<4, 1>();
    SLOOP(0, ".Ld", ".Ld_ev");
    Y(".Ld_ev:" N LEAVE "s_mov_b64 s[60:61], s[52:53]" N "s_mov_b32 s62, s54" N "s_mov_b32 " sAH ", 0x4000");
    REP(i, 0, 8, X("v_add_u32 v[%0], %1, v[%2]", "n"(VDA + i), "n"((i >> 2) * 4096), "n"(VRA + (i & 1))));
    Y("s_branch .Ld0");

    X(".p2align 6\n.fill ((.Lc_go - .Lc0) & 63) / 4, 4, 0xbf800000\n.Lc_go:" N
      "s_lshl_b32 s4, s10, 18" N "s_xor_b32 s5, s12, 0x200" N "s_or_b32 s4, s4, s5" N "v_lshrrev_b32 v5, 2, v1" N "v_lshl_add_u32 v92, v5, 14, s4" N "v_lshl_add_u32 v92, v14, 7, v92" N
      "v_bfrev_b32 v90, 1" N "v_cmp_gt_u32 vcc, %[pn], v1" N "v_cndmask_b32 v92, v93, v92, vcc" N "v_and_b32 v94, 31, v1" N "v_lshlrev_b32 v94, 2, v94" N "v_mov_b32 v72, 0" N
      "s_mov_b32 s88, %[t0]" N "s_mov_b32 s89, %[t1]" N "s_mov_b64 " cK ", 0" N "s_mov_b32 " cCL ", 0x7f000000" N "s_mov_b32 " sEV ", %[tv] - 1" N
      "s_lshl_b32 s4, s10, 10" N "s_add_u32 " cFB ", s4, %[af]" N "s_lshl_b32 s4, s10, 7" N "s_or_b32 s4, s4, s12" N "s_add_u32 " cCM ", s4, %[cl]" N
      "s_andn2_b32 s5, s80, 0x800000" N "v_mov_b32 v6, s5" N CALL(12)
      "v_mov_b32 v5, s44" N "s_mov_b64 exec, 0x7f" N "v_mov_b32 v5, s78" N "v_mov_b32 v6, s80" N "s_mov_b64 exec, -1" N
      "v_and_b32 v76, 15, v83" N "v_lshl_add_u32 v76, s4, 3, v76" N "s_mul_i32 " cTK ", s4, 2 * %[np]" N "s_add_u32 " cTK ", " cTK ", %[np]",
      [pn] "n"(64 / PFN), [t0] "n"((unsigned)TODO), [t1] "n"((unsigned)(TODO >> 32)), [tv] "n"(TV), [af] "n"(WS_AFLAG), [cl] "n"(WS_CLAIM), [np] "n"(N_POOL));
    REP(i, 0, 2, X("v_add_u32 v[%0], v[%0], v5" N "v_add_u32 v[%0+1], v[%0+1], v6" N "v_add_u32 v[%0+3], s47, v[%0+3]" N
      "v_and_b32 v[%0+1], 0x7ffffff, v[%0+1]" N "v_and_b32 v[%0+3], 0x87ff0000, v[%0+3]", "n"(80 + i * 4)));
    Y("v_readfirstlane_b32 s5, v78" N "s_cmp_eq_u32 s15, 0" N "s_cselect_b32 s5, s5, s4" N "s_cmp_lg_u32 s5, s4" N "s_cbranch_scc1 .Lc_taken\n.Lc_mine:");
    go<0>();
    X("v_mov_b32 v78, 1" N "v_mov_b32 v69, 1" N "v_mov_b32 v70, v90" N "v_mov_b32 v71, v90" N "s_lshl_b64 exec, %0, %2" N "v_mov_b32 v70, 0" N "s_lshl_b64 exec, %1, %2" N "v_mov_b32 v71, 0" N "s_mov_b64 exec, -1",
      "n"(DRA), "n"(DRB), "n"(DR0));
    wait<-1, 13>();
    SLOOP(1, ".Lc", ".Lc_ev");
#define VOTES "buffer_load_dword v95, v94, " sWS ", 0 offen offset:%[vo] sc1" N
    X(".Lc_ev:" N LEAVE "s_cmp_eq_u32 " sT ", %[tv]" N "s_cbranch_scc1 2f" N VOTES "s_mov_b32 " sEV ", %[tv]" N "s_branch .Lc0\n"
      "2:" N "s_mov_b32 " sT0 ", 0" N "s_lshl_b32 " cB ", " sCALL ", 1\n"
      "3:" N "v_cmp_le_u32 vcc, " cB ", v95" N "s_cmp_eq_u64 vcc, -1" N "s_cbranch_scc1 4f" N VOTES EXPIRED "s_cbranch_scc0 3b\n"
      "4:" N "v_cmp_lt_u32 vcc, " cB ", v95" N "s_bcnt1_i32_b64 " cB ", vcc" N "s_cmp_gt_u32 " cB ", 32" N "s_cbranch_scc0 .Lc1" N
      "v_mov_b64 v[82:83], v[86:87]" N "v_mov_b64 v[80:81], v[84:85]" N "s_mov_b64 " cK ", 0" N SAMPLE N "s_branch .Lc1",
      [tv] "n"(TV), [vo] "n"(WS_VOTE), DL);
    static_assert(TV == 5, "");
    Y(".Lc_slow:" N "s_mov_b32 " sT0 ", 0\n.Lc_again:");
    X(SAMPLE N EXPIRED "s_cbranch_scc0 5f" N "v_mov_b32 v76, 0\n5:" N ARRIVED N "s_cmp_eq_u64 " cK ", -2" N "s_cselect_b64 " cC ", 63, 62" N "s_or_b64 " cC ", " cC ", " cK N
      "s_and_b64 " cC ", " cC ", " cT N "s_cbranch_scc0 .Lc_again" N BACK(".Lc_back"), DL);
    Y(".Lc_taken:" N TAKE(-1) "s_add_u32 s22, s22, 128" N "s_bitset0_b32 s22, 15" N "s_waitcnt vmcnt(0)" N TAKE(1) "s_waitcnt vmcnt(0)" N "v_readfirstlane_b32 s5, v78" N
      "s_cmp_lg_u32 s5, s4" N "s_cbranch_scc1 .Lc_taken" N "v_mov_b32 v79, s22" N "s_branch .Lc_mine");

    X(".p2align 6\n.fill ((.Lp_go - .Lpi0) & 63) / 4, 4, 0xbf800000\n.Lp_go:" N
      "s_lshl_b32 s4, s10, 6" N "s_or_b32 s4, s4, s11" N "s_lshl2_add_u32 s4, s4, s13" N "s_lshl2_add_u32 s92, s4, s92" N "s_addc_u32 s93, s93, 0" N
      "s_lshl2_add_u32 s4, s11, s13" N "s_lshl_b32 s4, s4, 13" N "s_or_b32 s21, s21, s4" N
      "s_and_b32 s6, s94, 7" N "s_xor_b32 s4, s6, s11" N "s_and_b32 s4, s4, 1" N "s_lshl4_add_u32 s4, s4, s20" N "s_mul_i32 s4, s4, 7" N "s_add_u32 s4, s4, s6" N
      "s_mul_i32 s4, s4, %[pr]" N "s_lshr_b32 s4, s4, 16" N "s_sub_i32 " pS ", s4, %[sh]" N "s_max_i32 s4, " pS ", 0" N "s_lshr_b32 s4, s4, 6" N "s_lshl_b32 " sEV ", s4, 1" N
      "s_mov_b32 s94, 0" N "s_mov_b32 " pB ", 64" N "s_movk_i32 " pCS ", 0x80" N "s_mov_b32 " pL ", 8" N "s_mov_b64 vcc, -1" N CALL(20)
      "s_add_u32 s5, s5, s21" N "s_add_u32 s90, s90, s5" N "s_addc_u32 s91, s91, 0" N "s_lshl_b32 s4, s4, 3" N "v_mov_b32 v81, s4" N
      "s_cmp_lt_u32 s6, 7" N "s_cbranch_scc1 1f" N
      "s_lshl4_add_u32 s4, s11, s20" N "s_lshl_b32 s5, s4, 2" N "v_mov_b32 v86, s5" N "s_lshl_b32 s4, s4, 8" N "s_or_b32 s4, s4, s47" N "s_xor_b32 s4, s4, 0x4800000" N
      "v_lshl_add_u32 v84, v14, 14, s4" N "v_add_u32 v85, 0x2000, v84" N "s_xor_b32 s4, s4, 0x100" N "v_and_b32 v5, 31, v1" N "v_lshl_add_u32 v83, v5, 14, s4" N
      "v_lshrrev_b32 v5, 5, v1" N "v_lshl_add_u32 v83, v5, 13, v83" N "global_load_dword v82, v83, s[56:57]" N
      "s_cmp_eq_u32 s6, 7" N "s_cselect_b32 " sEV ", %[tp], -1" N "s_mov_b64 vcc, 0\n1:",
      [pr] "n"(PER * 65536 / 224 + 1), [sh] "n"(SHIFT), [tp] "n"(TP));
    go<0>(); wait<-1, 13>();
    Y("s_cmp_lg_u32 " sEV ", 0" N "s_cbranch_scc1 .Lpi0\n.Lpp0:");
    stile<3, 0>(); Y("s_cbranch_scc1 .Lp_ev"); stile<3, 1>();
    Y(".Lpi0:"); stile<2, 0>(); Y("s_cbranch_scc1 .Lp_ev\n.Lpi1:"); stile<2, 1>();
    X("s_cbranch_scc0 .Lpi0" N "s_cbranch_vccnz .Lpp0\n.Lp_ev:" N LEAVE
      "s_mov_b32 s98, 0" N "s_mov_b32 s97, 2\n"
      "4:" N "s_mov_b64 exec, vcc" N "global_load_dword v82, v84, s[56:57]" N "global_load_dword v83, v85, s[56:57]" N "s_waitcnt vmcnt(1)" N "s_memtime s[88:89]" N
      "s_waitcnt vmcnt(0)" N "s_memtime s[90:91]" N "v_swap_b32 v84, v85" N "v_add_u32 v84, %0, v84" N "v_add_u32 v85, %0, v85" N "s_mov_b64 exec, -1" N "s_waitcnt lgkmcnt(0)" N
      "s_sub_u32 s88, s90, s88" N "s_min_i32 s88, s88, 400" N "s_and_b32 s88, s88, vcc_lo" N "s_sub_i32 s98, s88, s98" N "s_mov_b64 vcc, -1" N
      "s_sub_u32 s97, s97, 1" N "s_cbranch_scc0 4b" N
      "s_getreg_b32 s88, hwreg(HW_REG_XCC_ID, 2, 1)" N "s_lshr_b32 s98, s98, 31" N "s_mov_b64 vcc, 0" N "s_xor_b32 s88, s88, s98" N "s_xor_b32 s88, s88, 1" N "s_lshl1_add_u32 s88, " sCALL ", s88" N
      "v_mov_b32 v82, s88" N "buffer_store_dword v82, v86, " sWS ", 0 offen offset:%1 sc1" N BACK(".Lpi"), "n"(4 * 16384), "n"(WS_VOTE));

    X(".Lx_slow:" N "buffer_load_dword v61, off, " sWS ", " sXF " sc0 sc1" N EXPIRED "s_cselect_b32 " sMORE ", " sCALL ", " sMORE N "s_cbranch_scc1 .Lx_back" N
      "v_readfirstlane_b32 s84, v61" N "s_cmp_lt_u32 s84, " sCALL N "s_cbranch_scc1 .Lx_slow", DL);
    Y("s_branch .Lx_back");

    Y(".Lexit:" N "s_cmp_lg_u32 " sMORE ", 0" N "s_cbranch_scc1 .Lagain" N "s_bitcmp1_b32 " sT ", 0" N "s_cbranch_scc0 .Lt" N
      "s_mov_b32 s84, s74" N "s_mov_b32 s74, s75" N "s_mov_b32 s75, s84");
    REP(i, 0, 6, X("v_swap_b32 v[%0], v[%1]", "n"(VX + i), "n"(VY + i)));
    REP(i, 0, 4, emit<ZB, i, 0, 0, 0>());
    Y(".Lt:");
    REP(q, 0, NX / 16, REP(j, 0, 16, constexpr int g = q * 16 + j;
      if constexpr (g < NG) { constexpr Op f = PLAN.mf[g < NG ? g : 0]; mfma<f.k, f.a, f.b, f.c>(); }
      REP(i, 0, PLAN.n[g], constexpr Op o = PLAN.op[g][i]; emit<o.k, o.a, o.b, o.c, o.d>())));
    Y("s_cmp_eq_u32 " sMORE ", " sCALL N "s_cbranch_scc0 .Lfin" N "s_waitcnt vmcnt(0)" N "v_mov_b32 v0, 0x7fc07fc0");
    REP(i, 0, 4, X("v_accvgpr_write_b32 a[%0], v0", "n"(FO + i)));
    REP(i, 0, 16, emit<OU, i / 2, 5, i % 2, 1>());
    Y(".Lfin:" N "v_mov_b32 v0, " sCALL N "buffer_store_dword v0, off, " sWS ", " sCTR " sc1");
    Y("s_endpgm\n.Lagain:" N "s_waitcnt vmcnt(0) lgkmcnt(0)" N "s_barrier" N "s_bfe_u32 s3, " sMORE ", 0x80007" N "s_brev_b32 s15, 4" N "s_branch .Lstart\n.p2align 7\n.Ltab:");
    REP(q, 0, 2, REP(l, 0, 64, X(".long %0, %1, %2, %3", "n"(table(q, l, 0)), "n"(table(q, l, 1)), "n"(table(q, l, 2)), "n"(table(q, l, 3)))));
    Y(".Lend:");
  }
}

extern "C" void kernel_launch(int rank, int nranks, ncclComm_t comm,
                              hipStream_t stream,
                              void* const* d_in, const int* in_sizes, int n_in,
                              void* d_out, int out_size,
                              const int* in_dtypes, int out_dtype,
                              void* const* d_out_all,
                              void* const* d_ws_all, size_t ws_bytes) {
  if (nranks != NRANK || (unsigned)rank >= (unsigned)NRANK || n_in < 2 || ws_bytes < WS_NEED ||
      in_sizes[0] != 8192 * 1024 || in_sizes[1] != 8192 * 8192 || out_size != 1024 * 8192) {
    fprintf(stderr, "a2a_gemm: unexpected problem shape\n");
    return;
  }
  HIP_CHECK(hipFuncSetAttribute((const void*)a2a_gemm, hipFuncAttributeMaxDynamicSharedMemorySize, LDS_BYTES));
  const char* x = (const char*)d_in[0];
  char* ws = (char*)d_ws_all[rank];
  Params p = {{ws, (unsigned)WS_NEED, 0x20000}, {d_in[1], 8192 * 16384, 0x20000}, {x + rank * BLK_B, (unsigned)BLK_B, 0x20000}, {d_out, 1024 * 16384, 0x20000}, {}};
  p.duty[3] = {x, ws, ws, (unsigned)rank << 24 | 7, 0x20000};
  for (int j = 0, n[2] = {0, 4}; j < 7; ++j) {
    const int q = (rank + 1 + j) & 7, e = n[((0xb2 >> rank) ^ (0xb2 >> q)) & 1]++;
    char* to = (char*)d_ws_all[q];
    p.duty[e] = {x + q * BLK_B, to + WS_A + (6 - j) * BLK_B + 4096, to + WS_AFLAG + (6 - j) * 128, (unsigned)rank << 24 | j, 0x20000};
  }
  a2a_gemm<<<256, 256, LDS_BYTES, stream>>>(p);
  HIP_CHECK(hipGetLastError());
}
```

```cpp
#include <hip/hip_runtime.h>
#include <rccl/rccl.h>
#include <cstdio>

#define HIP_CHECK(expr) do { hipError_t _e = (expr); if (_e != hipSuccess) { \
    fprintf(stderr, "HIP error %s at %s:%d\n", hipGetErrorString(_e), __FILE__, __LINE__); \
    return; } } while (0)

typedef unsigned long long u64;

#define PER 80
#define SHIFT 28
#define FPQ 5
#define TV 5
#define TP 1
#define VTH 40
#define RL 11
#define AB 3
#define FG 291
#define LAGF 72
#define LAGP 80
#define PFR 44
#define PFS 1
#define PFN 1
#define DR0 46
#define DRA 0x1b
#define DRB 0x16
#define EM 0x54aa
#define SWP 0xc4
#define XQ 16

constexpr int NRANK = 8;
constexpr int NCOL = 8192;
constexpr int U = 16384;
constexpr int LDS_BYTES = 10 * U;
constexpr int LDS_CTRL = 9 * U + 1024;
constexpr size_t BLK_B = (size_t)1024 * 1024 * 2;
constexpr int N_POOL = 7;
constexpr int N_X = (RL - 7) / 2;
constexpr unsigned XSRC = N_X == 1 ? 6u : N_X == 4 ? 0x6614u : 0x14614u;
static_assert(PFR >= TV && PFR + ((7 + N_X) * PFN - 1) * PFS <= 54 - N_X, "");
static_assert(__builtin_popcount(DRA) + __builtin_popcount(DRB) == N_POOL && ((u64)(DRA | DRB) << DR0) >> (56 - 3 - N_X) == 0, "the tickets come in time");
static_assert(RL >= 7 && RL <= 17 && (RL & 1), "");
constexpr unsigned RROW = 128, RCOL = 32768;
constexpr int WS_TMO = 0;
constexpr int WS_VOTE = 256;
constexpr size_t WS_CTR = 16384;
constexpr size_t WS_CLAIM = 49152;
constexpr size_t WS_XFLAG = 65536;
constexpr size_t WS_AFLAG = 131072;
constexpr size_t WS_ROLE = 196608;
constexpr size_t WS_A = (size_t)1 << 20;
constexpr size_t WS_XCH = (size_t)34 << 20;
constexpr unsigned AWAY = (50u << 20) + 128;
constexpr size_t WS_NEED = (size_t)103 << 20;
constexpr unsigned DEADLINE = 50000000u;

constexpr int event(int s, int k, int p) { return 2 * k + (p ^ (s & 1)); }
constexpr unsigned sender(int e) { return (EM >> e & 1 ? EM : ~EM) & 0xffff; }
constexpr unsigned offA(int s, int k, int p) { return s * BLK_B + event(s, k, p) * RCOL; }
constexpr unsigned offW(int s, int k, int p) { return ((1 + s) * 16 + k / 2 * 4 + p * 2 + k % 2) << 20; }
constexpr int rsrc(int r) { return r < 7 ? r : (XSRC >> (r - 7) / 2 * 4) & 7; }
constexpr int rcol(int r) { return r < 7 ? 7 : (0x55666 >> (r - 7) / 2 * 4) & 7; }
constexpr unsigned table(int q, int l, int i) {
  const int k = l / 7, o = q ? 6 - l % 7 : l % 7, w = l < 7 || l == 63, s = w ? (l + 6) % 7 : (o * 2) % 7, p = w ? ~s & 1 : k < 8 ? q : s & 1, own = l < 5 ? l + 3 : 3;
  const int z = (l - PFR) / PFS, y = z / PFN, r = y < 7 ? y : 2 * y - 7;
  const int e = event(s, w ? 7 : k - 1, p);
  return i == 0 ? (w ? own * 128 : offA(s, k - 1, p)) : i == 1 ? (w ? own << 20 : offW(s, k - 1, p)) : i == 2 ? (l ? s * 128 + (sender(e) & 1) * 16 : 1u << 31)
       : (l >= PFR && z * PFS + PFR == l && r < RL ? offW(rsrc(r), rcol(r), q) + z % PFN * (0x40000 / PFN) : 1u << 31) | __builtin_popcount(sender(e) & ((2 << e) - 1));
}
constexpr u64 TODO = 0x00ffffffffffffbeull & ~((N_X & 1 ? 1ull << 52 : 0) | (N_X >= 2 ? 5ull << 51 : 0) | (N_X == 4 ? 0x81ull << 45 : 0) | (N_X == 5 ? 5ull << 44 : 0));
constexpr int PHASE = PER * 65536 / 112 + 1;
constexpr bool live() {
  for (int q = 0; q < 2; ++q) for (int m = 0; m < 4; ++m) {
    int n[99] = {};
    for (int l = 1; l < 64; ++l) { const int s = table(q, l, 2) / 128, c = table(q, l, 3) & 15, r = table(q, l, 2) / 16 & 1; int t = -4;
      for (int e = 0, i = 0; i < c; ++e) if ((~EM >> e & 1) == r) { const int u = ((((m * 4 + 3) * 7 + 6 - s) * PHASE >> 16) - SHIFT + e * PER) >> 5 & -2; t = u > t + 4 ? u : t + 4; ++i; }
      if (t + 3 > 54 - N_X) return false;
      n[t + 3] += l > 6 && (TODO >> l & 1); }
    for (int t = 1; t < 54 - N_X; ++t) if ((n[t] += n[t - 1]) < t - 4) return false; }
  return (EM & 3) == 2 && (EM >> 14 == 1 || EM >> 14 == 2);
}
static_assert(live(), "");

struct Srd { const void* base; unsigned bytes, mode; };
struct Duty { const char* x; char* to; char* flag; unsigned who, mode; };
struct Params {
  Srd ws, w, x, out;
  Duty duty[8];
};

constexpr int FA0 = 128, FA1 = 160, FB0 = 192, FB1 = 224, FA2 = 0, FA3 = 32, FO = 64, VX = 32, VY = 38, VZ = 44, VRA = 50, VRB = 52, VDA = 72, VDB = 80;
constexpr int SWS = 52, SW = 56, SA = 60, SRA = 44, SLX = 74, SLW = 77, SA0 = 78, SA1 = 79, SB1 = 80, SB0 = 81;
#define sT "s48"
#define sCALL "s49"
#define sMORE "s50"
#define sEV "s51"
#define sWS "s[52:55]"
#define sOUT "s[64:67]"
#define sXD "s[68:69]"
#define sXS "s70"
#define sXF "s71"
#define sXP "s72"
#define sCTR "s73"
#define sAH "s82"
#define sOF "s83"
#define cT "s[88:89]"
#define cK "s[90:91]"
#define cC "s[92:93]"
#define cB "s94"
#define cCL "s95"
#define cTK "s96"
#define cFB "s97"
#define cCM "s98"
#define sT0 "s99"
#define pXN "s[88:89]"
#define pD "s[90:91]"
#define pF "s[92:95]"
#define pB "s96"
#define pE "s97"
#define pS "s98"
#define pM "s99"

#define DI __device__ __forceinline__
#define X(s, ...) asm volatile(s :: __VA_ARGS__ : "memory")
#define Y(s) asm volatile(s ::: "memory")
#define N "\n\t"
template <int V> struct IC { static constexpr int value = V; };
template <int I, int E, class F> DI void rep(F f) { if constexpr (I < E) { f(IC<I>{}); rep<I + 1, E>(f); } }
#define REP(i, a, b, ...) rep<a, b>([](auto i##_) __attribute__((always_inline)) { constexpr int i = decltype(i##_)::value; __VA_ARGS__; })

template <int KEEP, int T, int FB, int FA> DI void mfma() {
  if constexpr (KEEP) X("v_mfma_f32_16x16x32_bf16 v[%0:%0+3], a[%1:%1+3], a[%2:%2+3], v[%0:%0+3]", "n"(128 + T * 4), "n"(FB), "n"(FA));
  else X("v_mfma_f32_16x16x32_bf16 a[%0:%0+3], a[%1:%1+3], a[%2:%2+3], a[%0:%0+3]", "n"(T * 4), "n"(FB), "n"(FA));
}
template <int F, int i, int S, int O> DI void rdA() { X("ds_read_b128 a[%0:%0+3], v[%1] offset:%2", "n"(F + i * 4), "n"(S + (i & 1)), "n"(O + (i >> 1) * 2048)); }
template <int F, int i, int S, int O> DI void rdB() { X("ds_read_b64_tr_b16 a[%0:%0+1], v[%1] offset:%2", "n"(F + i * 2), "n"(S + 2 + (i >> 2)), "n"(O + ((i >> 1) & 1) * 8192 + (i & 1) * 1024)); }
template <int S, int O> DI void m0() { X("s_add_u32 m0, s[%0], %1", "n"(S), "n"(O)); }
template <int V, int R, int SO, int i> DI void dmaA() { X("buffer_load_dwordx4 v[%0], s[%1:%1+3], s[%2] offen offset:%3 sc0 sc1 lds", "n"(V), "n"(R), "n"(SO), "n"((i & 3) * 1024)); }
template <int V, int SO, int i> DI void dmaB() { X("buffer_load_dwordx4 v[%0], s[56:59], s[%1] offen offset:%2 lds", "n"(V), "n"(SO), "n"((i & 3) * 1024)); }
template <int V, int L> DI void wait() {
  if constexpr (V < 0) X("s_waitcnt lgkmcnt(%0)", "n"(L)); else if constexpr (L < 0) X("s_waitcnt vmcnt(%0)", "n"(V)); else X("s_waitcnt vmcnt(%0) lgkmcnt(%1)", "n"(V), "n"(L));
}
template <int P, int Q, int i> DI void fm0() { if constexpr (Q == 0 || Q == 3) m0<SLW, (P * 3 + Q / 3) * U + (i >> 2) * 4096>(); else m0<87 + P * 2 + Q, (i >> 2) * 4096>(); }
template <int Q, int i, int SO = Q == 3 ? SA1 : Q ? SA1 + Q : SA0> DI void fill() { if constexpr (Q == 0 || Q == 3) dmaA<VDA + i, SA, SO, i>(); else dmaB<(i < 4 ? VRB : VDB) + i, SO, i>(); }
constexpr int early(int P, int Q, int g) {
  int n = P * 44 + Q * 12;
  for (int x = 0; x < 32; ++x) { const bool y = x % 4 == 1 || (Q == 0 ? x >= 15 && x < 30 && x % 2 : Q < 3 && x % 8 == 3); if (x == g) return y ? n : -1; n += y; }
  return -1;
}
#define SAMPLE "buffer_load_dwordx4 v[72:75], v82, " sWS ", " cFB " offen sc0 sc1"
#define ARRIVED "v_min3_u32 v77, v72, v73, v74" N "v_min_u32 v77, v77, v75" N "v_cmp_le_u32 vcc, v76, v77" N "s_or_b64 " cK ", " cK ", vcc"
#define EXPIRED "buffer_load_dword v60, off, " sWS ", 0 sc1" N "s_memrealtime s[84:85]" N "s_sleep 1" N "s_waitcnt vmcnt(0) lgkmcnt(0)" N "v_readfirstlane_b32 s85, v60" N \
    "s_or_b32 s84, s84, 1" N "s_cmp_eq_u32 " sT0 ", 0" N "s_cselect_b32 " sT0 ", s84, " sT0 N "s_sub_u32 s84, s84, " sT0 N \
    "s_cmp_eq_u32 s85, " sCALL N "s_cselect_b32 s84, -1, s84" N "s_cmp_gt_u32 s84, %[dl]" N "s_cbranch_scc0 1f" N \
    "v_mov_b32 v60, " sCALL N "buffer_store_dword v60, off, " sWS ", 0 sc1\n1:" N
#define DL [dl] "n"(DEADLINE)
#define NEXT0 "s_ff1_i32_b32 " pE ", " pM N "s_bitset0_b32 " pM ", " pE
#define NEXT1 "s_lshl_b32 s84, " pE ", 2"
#define NEXT2 "s_lshr_b64 s[84:85], s[20:21], s84"
#define NEXT3 "s_and_b32 s84, s84, 15" N "s_lshl_b32 s84, s84, 7"
#define NEXT4 "s_add_u32 s88, s16, s84" N "s_addc_u32 s89, s17, 0"

constexpr int nth(int g) { return (g & 3) == 1 ? -99 : g - (g + 2) / 4; }
constexpr int sb(int g) { return ((g >> 1) & 3) ^ (g & 8 ? 3 : 0); }
constexpr int sk(int g) { return (g ^ (g >> 1)) & 1; }
template <int ROLE, int P> DI void stile() {
  REP(Q, 0, 4, REP(g, 0, 32,
    constexpr int cur = P ? VY : VX, nxt = P ? VX : VY, keep = Q == 0 || Q == 3, a = g >> 3, m = sb(g), k = g & 1, r = nth(g), y = nth(g & 7);
    mfma<keep, (Q >> 1) * 16 + m * 4 + a, (keep ? FB1 : FB0) + (a * 2 + sk(g)) * 4, (Q >> 1 ? FA1 : FA0) + (m * 2 + sk(g)) * 4>();
    if constexpr (Q == 0 && r >= 0 && r < 4) rdB<FB1, 12 + r, cur, 2 * U>();
    if constexpr (Q == 0 && r >= 4 && r < 11) rdB<FB0, 5 + r, VZ, P * U>();
    if constexpr (Q == 1 && g < 16 && !k) rdA<FA1, g / 2, cur, U>();
    if constexpr (Q == 2 && g < 16 && !k) rdA<FA0, g / 2, nxt, 0>();
    if constexpr (Q == 3 && a > 0 && y >= 0 && y < 4) rdB<FB1, (a - 1) * 4 + y, nxt, 2 * U>();
    if constexpr (Q == 3 && a < 3 && y >= (a ? 4 : 1)) rdB<FB0, a * 2 + y - 1, VZ, (P ^ 1) * U>();
    if constexpr (Q == 0 && g == 15) wait<-1, 7>();
    if constexpr (Q == 3 && g == 0) { if constexpr (ROLE % 4) Y("ds_read_b32 v66, v68 offset:8"); else Y("ds_read_b96 v[64:66], v68"); }
    if constexpr (Q == 3 && g == 8) wait<-1, 6>();
    if constexpr (Q == 3 && g == 10) Y("v_readfirstlane_b32 " sMORE ", v66");
    if constexpr (ROLE % 4 == 0) { constexpr int e = early(P, Q, g), f = early(P, Q, g + 1);
      if constexpr (!ROLE && g % 16 == 0) fm0<P, Q, g / 4>();
      if constexpr (!ROLE && g % 4 == 1) fill<Q, g / 4>();
      if constexpr (ROLE && f >= 0 && f % 4 == 0) fm0<(f / 8 + 5) / 4 % 2, (f / 8 + 1) % 4, f % 8>();
      if constexpr (ROLE && e >= 0) { if constexpr (e < 56) fill<(e / 8 + 1) % 4, e % 8, 93 + e / 8>(); else fill<(e / 8 + 1) % 4, e % 8>(); }
      if constexpr (ROLE && P && Q == 1 && g == 18) Y("v_readfirstlane_b32 s4, v63");
      if constexpr (ROLE && P && Q == 1 && g == 22) Y("s_add_u32 " sCALL ", s4, 1" N "s_lshl_b32 s5, " sCALL ", 24");
      if constexpr (ROLE && P && Q == 1 && g == 23) Y("s_and_b32 s5, s5, 0x1000000" N "s_add_u32 s44, s44, s5");
      if constexpr (ROLE && P && Q == 1 && g == 26) Y("s_xor_b32 s45, s44, 0x4000");
      if constexpr (Q == 0 && g == 22) Y("s_xor_b32 s79, s78, " sAH);
      if constexpr (Q == 3 && g == 11) Y("v_readfirstlane_b32 s78, v64");
      if constexpr (Q == 3 && g == 12) Y("v_readfirstlane_b32 s80, v65");
      if constexpr (Q == 3 && g == 14) Y("s_xor_b32 s80, s80, s86");
      if constexpr (Q == 3 && g == 15) Y("s_xor_b32 s81, s80, 0x100");
    }
    if constexpr (ROLE == 1 && Q == 1 && g == 17) Y("v_readlane_b32 s87, v83, " sT);
    if constexpr (ROLE == 1 && Q == 1 && g == 23) Y("buffer_load_dword v91, v92, s[56:59], s87 offen");
    if constexpr (ROLE == 1 && !(Q & 1) && g == 26) Y("s_lshl_b64 exec, 1, " sT);
    if constexpr (ROLE == 1 && !(Q & 1) && g == 27) X("buffer_atomic_add v[%0], v[%1], " sWS ", " cCM " offen sc0" N "s_mov_b64 exec, -1", "n"(Q ? 69 : 78), "n"(70 + Q / 2));
    if constexpr (ROLE == 1 && Q == 1) {
      if constexpr (g == 24) Y("v_cmp_le_u32 vcc, " cTK ", v78" N "v_cmp_le_u32 s[84:85], " cTK ", v69");
      if constexpr (g == 26) Y("s_bcnt1_i32_b64 s86, vcc" N "s_bcnt1_i32_b64 s84, s[84:85]");
      if constexpr (g == 27) Y("s_lshr_b32 s86, " cCL ", s86" N "s_lshr_b32 s86, s86, s84");
      if constexpr (g == 28) Y("s_and_b32 s93, s86, " cCL N "s_cselect_b32 s92, 0, 1");
    }
    if constexpr (ROLE == 1 && Q == 2) {
      if constexpr (g == 7) { wait<2, -1>(); Y("v_min3_u32 v77, v72, v73, v74"); }
      if constexpr (g == 8) Y("v_min_u32 v77, v77, v75");
      if constexpr (g == 9) Y("v_cmp_le_u32 vcc, v76, v77");
      if constexpr (g == 11) Y("s_or_b64 " cK ", " cK ", vcc" N "s_cmp_eq_u64 " cT ", 0");
      if constexpr (g == 13) Y("s_cselect_b64 " cT ", " cC ", " cT N "s_cselect_b32 " cCL ", 0, " cCL);
      if constexpr (g == 15) Y("s_cmp_eq_u64 " cK ", -2");
      if constexpr (g == 16) Y("s_cselect_b64 " cC ", 63, 62" N "s_or_b64 " cC ", " cC ", " cK);
      if constexpr (g == 17) X("s_and_b64 " cC ", " cC ", " cT N "s_cbranch_scc0 .Lc_slow\n.Lc_back%0:", "n"(P));
      if constexpr (g == 18) Y("s_ff1_i32_b64 " cB ", " cC N "s_lshl_b64 " cC ", 1, " cB);
      if constexpr (g == 19) Y("s_andn2_b64 " cT ", " cT ", " cC);
      if constexpr (g == 20) Y("v_cndmask_b32 v88, v89, v68, " cC N "v_cndmask_b32 v82, v82, v90, " cK);
      if constexpr (g >= 21 && g < 24) X("ds_write_b32 v88, v[%0] offset:%1", "n"(79 + (g - 20) % 3), "n"((g - 21) * 4));
      if constexpr (g == 25) Y(SAMPLE);
    }
    if constexpr (ROLE == 2 && P * 4 + Q <= FPQ) {
      constexpr int young = P * 4 + Q + 8;
      if constexpr (g == 9) { wait<(P * 4 + Q < FPQ ? 15 : young), -1>(); Y("s_getreg_b32 s84, hwreg(HW_REG_IB_STS, 0, 4)"); }
      if constexpr (g == 13) X("s_or_b32 s84, s84, " pB N "s_cmp_le_u32 s84, %0", "n"(young));
      if constexpr (g == 17) Y("s_cselect_b32 s94, 4, 0" N "s_cselect_b32 " pB ", 64, " pB);
      if constexpr (g == 21) Y("buffer_store_dword v81, off, " pF ", 0 sc0 sc1");
    }
    if constexpr (ROLE == 3) { constexpr int u = P * 4 + Q;
      if constexpr (g == 1 && u == 0) wait<FPQ + 1, -1>();
      if constexpr (g == 1) X("global_store_dwordx4 v56, v[%0:%0+3], " pD " offset:%1 sc0 sc1", "n"(96 + u * 4), "n"(u * 1024 - 4096));
      if constexpr (u == 7 && g >= 2 && g < 10) X("global_load_dwordx4 v[%0:%0+3], v[%1], " pXN " nt", "n"(88 + g * 4), "n"(86 + g));
      if constexpr (u == 6 && g == 17) X("s_mul_i32 s84, " pE ", %0" N "s_add_i32 s84, s84, " pS, "n"(PER));
      if constexpr (u == 6 && g == 18) Y("s_ashr_i32 s84, s84, 5" N "s_and_b32 s84, s84, -2");
      if constexpr (u == 6 && g == 19) Y("s_add_u32 " sEV ", " sEV ", 4" N "s_max_i32 " sEV ", " sEV ", s84");
      if constexpr (u == 6 && g == 20) Y("s_cmp_ge_i32 " pE ", 0" N "s_cselect_b64 vcc, -1, 0");
      if constexpr (u == 6 && g == 21) Y("s_orn2_b32 " sEV ", " sEV ", vcc_lo" N "s_lshl_b32 s84, " pE ", 15");
      if constexpr (u == 6 && g == 22) Y("s_mov_b32 " pB ", 0" N "v_add_u32 v81, 1, v81");
      if constexpr (u == 7 && g == 1) Y("s_add_u32 s90, s22, s84" N "s_addc_u32 s91, s23, 0");
      if constexpr (u == 6 && g == 23) Y(NEXT0);
      if constexpr (u == 7 && g == 11) Y(NEXT1);
      if constexpr (u == 7 && g == 12) Y(NEXT2);
      if constexpr (u == 7 && g == 13) Y(NEXT3);
      if constexpr (u == 7 && g == 17) Y(NEXT4);
    }
    if constexpr (!(Q & 1) && g == 30) wait<ROLE % 4 ? -1 : !ROLE ? 32 : P * 12 + Q * 4 + 12, 0>();
    if constexpr (!(Q & 1) && g == 31) Y("s_barrier");
    if constexpr (Q == 1 && g == 31) wait<-1, 0>();
    if constexpr (Q == 3 && g == 18) Y("s_add_u32 " sT ", " sT ", 1");
    if constexpr (Q == 3 && g == 19) Y("s_cmp_eq_u32 " sT ", " sEV);
    if constexpr (Q == 3 && g == 20) Y("s_cselect_b32 s84, 0, " sMORE);
    if constexpr (Q == 3 && g == 22) Y("s_cmp_lg_u32 s84, 1");
    if constexpr (Q == 3 && g == 30) wait<-1, 6>();
  ));
}
#define SLOOP(role, l, ev) do { Y(l "0:"); stile<role, 0>(); Y("s_cbranch_scc1 " ev "\n" l "1:"); stile<role, 1>(); Y("s_cbranch_scc0 " l "0"); } while (0)
#define BACK(l) "s_bitcmp1_b32 " sT ", 0" N "s_cbranch_scc1 " l "1" N "s_branch " l "0"
#define LEAVE "s_cmp_lg_u32 " sMORE ", 1" N "s_cbranch_scc1 .Lexit" N

enum { NONE, RA, RB, M0, DA, DB, WT, BR, RN, ZB, XS, PB, FL, FC, PR, AD, OU, NP, ID, OA };
struct Op { short k, a, b, c, d; };
constexpr int NQ = 8 + 2 * RL, NG = NQ * 32, L = RL - 1, NX = NG + 48;
struct Plan {
  Op mf[NG] = {}, op[NX][8 * AB + 6] = {};
  char n[NX] = {}, room[NX] = {};
  bool ok = true;
  short mark[160] = {};
  static constexpr bool st4(const Op& o) { return (o.k == XS && o.a % 13 == 12) || (o.k == OU && o.b == 5); }
  static constexpr bool vm(const Op& o) { return o.k == DA || o.k == DB || o.k == FL || o.k == PR || st4(o) || (o.k == PB && o.a); }
  static constexpr bool ds(const Op& o) { return o.k == RA || o.k == RB || (o.k == OU && (o.b == 2 || o.b == 3)); }
  static constexpr bool vs(const Op& o) { return o.k == FC && o.a < 2; }
  static constexpr bool va(const Op& o) { return vs(o) || o.k == ZB || o.k == ID || ((o.k == XS || o.k == PB || o.k == OU) && !vm(o) && !ds(o)); }
  static constexpr bool wr(const Op& o) { return (o.k == PB && o.a) || (o.k == OU && o.b == 2); }
  constexpr bool fits(int g, const Op& o) {
    if (n[g] >= 2 + room[g]) return false;
    if (n[g] && op[g][n[g] - 1].k == AD) return va(o) || (ds(o) && !wr(o));
    if (st4(o)) return !n[g] || (n[g] == 1 && op[g][0].k == WT);
    if (g < NG - 128 && (o.k == RA || o.k == RB) && g % 4 == 1) return false;
    for (int j = 0; j < n[g]; ++j) { const Op& p = op[g][j];
      if (wr(p) || ((vm(o) || ds(o)) && (va(p) || p.k == BR || ds(p) || (vm(p) && !st4(p)))) || (vs(p) && !va(o) && o.k != WT)) return false; }
    return true;
  }
  constexpr int put(int g, int k, int a = 0, int b = 0, int c = 0, int d = 0) { while (!fits(g, Op{(short)k, (short)a, (short)b, (short)c, (short)d})) ++g; return at(g, k, a, b, c, d); }
  constexpr int at(int g, int k, int a = 0, int b = 0, int c = 0, int d = 0) { const Op o = {(short)k, (short)a, (short)b, (short)c, (short)d}; if (st4(o)) room[g] = 2 + n[g]; if (k == AD) room[g] = n[g]; op[g][(int)n[g]++] = o; return g; }
  constexpr void mm(int q, int keep, int half, int fb, int fa) {
    for (int g = q * 32; g < q * 32 + 32; ++g) { const int nn = (g >> 3) & 3, m = sb(g), k = sk(g);
      mf[g] = Op{(short)keep, (short)(half * 16 + m * 4 + nn), (short)(fb + (nn * 2 + k) * 4), (short)(fa + (m * 2 + k) * 4), 0}; }
  }
  static constexpr int set(int r) { return r % 3 == 0 ? VX : r % 3 == 1 ? VY : VZ; }
  constexpr void ra(int g, int st, int f, int s, int u, int tag) { for (int i = 0; i < 8; ++i) put(g + i * st, RA, tag, f * 16 + i, s, u); }
  constexpr void rb(int g, int i0, int i1, int f, int s, int u, int tag) { for (int i = i0; i < i1; ++i) put(g + i - i0, RB, tag, f * 16 + i, s, u); }
  constexpr void jit(int q, int s) { for (int j = 0; j < 4; ++j) rb(q * 32 + 8 + j * 8, j * 4, j * 4 + 4, FB1, s, 2, 90 + (j >> 1)); }
  constexpr void sync(int q, int t0, int t1, int lt) { at(q * 32 + 31, WT, t0, t1, lt); at(q * 32 + 31, BR); }
  int nf = 0;
  constexpr void fill(int g, int skip, int u, int r) {
    if (r > L) return;
    const int so = SA0 + (nf++ & 1), tag = (u == 2 ? 20 : u * 40) + r;
    ok = ok && put(g - 7, RN, so, u, r) <= g - 4 && (u < 2 || put(g - 3, RN, so, 3) < g) && put(g - 2, M0, r % 3 < 2 ? SLX + r % 3 : SLW, u + r % 3 / 2 * 6) < g;
    for (int i = 0; i < 4; g += 4) if (!((skip >> (g / 4 % 4)) & 1)) ok = ok && put(g, u == 2 ? DB : DA, tag, i++, so) == g;
  }
  constexpr int count(bool fix, int tag) {
    int nv = 0, nl = 0;
    for (int i = 0; i < 160; ++i) mark[i] = 30000;
    mark[60] = -24; mark[61] = -8; mark[62] = 0;
    for (int g = 0; g < NX; ++g) for (int j = 0; j < n[g]; ++j) { Op& o = op[g][j];
      if (vm(o)) { ++nv; mark[o.k == XS ? (o.a == 207 ? 64 : 159) : o.k == FL ? 65 : o.k == PR ? 70 + o.a : (o.k == DA || o.k == DB) && o.b == 3 ? o.a : 159] = nv; }
      if (ds(o)) { ++nl; mark[o.k == OU ? 100 + o.a : o.a] = nl; }
      if (o.k == WT) {
        const int v = o.a < 0 ? -1 : nv - (o.b >= 0 && mark[o.b] > mark[o.a] ? mark[o.b] : mark[o.a]), l = o.c < 0 ? -1 : o.c == 0 ? 0 : nl - mark[o.c];
        if (!fix && o.a >= 0 && v <= nv - mark[tag]) return g;
        if (fix) { ok = ok && (o.a < 0 || (v >= 0 && v < 64)) && (o.c < 0 || l >= 0); o.a = v; o.c = l > 14 ? 14 : l; }
      } }
    return -1;
  }
  int st = 0, lw[32] = {}, fin[32] = {}, ad[32] = {}, io = 0, no = 0;
  Op ou[200] = {};
  constexpr void out(int g) {
    for (bool go = true; go && io < no; ) {
      go = false;
      for (int x = io; x < no && x < io + 12 && !go; ++x) { const Op o = ou[x]; const int T = o.a * 4 + o.c;
        if (o.k == NONE || (x > io && !va(o))) continue;
        if (o.k == OU && o.b == 0 && !(g >= fin[T] && st - lw[T] >= 8 && ad[T])) {
          if (x == io && g >= NG && ad[T]) { at(g, NP, 7 - (st - lw[T])); st += 8 - (st - lw[T]); go = true; }
          break; }
        if (g < NG ? fits(g, o) && (o.k != WT || !n[g]) : x == io && n[g] < 2) { at(g, o.k, o.a, o.b, o.c, o.d); ou[x].k = NONE; ++st; go = true; }
        else if (va(o)) break;
      }
      while (io < no && ou[io].k == NONE) ++io;
    }
  }
  constexpr Plan() {
    sync(0, 60, -1, 0); sync(2, 61, -1, 0); sync(4, 62, 0, 0); sync(6, 20, -1, 0); sync(7, 40, -1, 90);
    for (int r = 0; r < L - 1; ++r) { sync(8 + 2 * r, r + 1, 21 + r, 0); if (r < L - 2) sync(9 + 2 * r, 41 + r, -1, 90); else sync(9 + 2 * r, L, 20 + L, 90); }
    sync(NQ - 4, 40 + L, -1, 97);
    rb(0, 12, 16, FB1, VX, 2, 91); rb(4, 9, 16, FB0, VZ, 0, 92);
    for (int t = 0; t < 2; ++t) { const int q = t * 4;
      mm(q, 1, 0, FB1, FA0); mm(q + 1, 0, 0, FB0, FA0); mm(q + 2, 0, 1, FB0, FA1); mm(q + 3, 1, 1, FB1, FA1);
      ra(q * 32 + 32, 2, FA1, set(t), 1, 92); ra(q * 32 + 64, 2, FA0, set(t ^ 1), 0, 92); jit(q + 3, set(t ^ 1));
      if (!t) rb(98, 0, 16, FB0, VZ, 0, 92);
      at(q * 32 + 63, WT, -1, -1, 0);
    }
    for (int r = 0; r < L - 1; ++r) { const int q = 8 + 2 * r;
      mm(q, 1, 0, FB1, FA0); mm(q + 1, 1, 1, FB1, FA1);
      jit(q + 1, set(r + 1)); ra(q * 32 + 6, 2, FA1, set(r), 1, 92); ra(q * 32 + 32, 1, FA0, set(r + 1), 0, 92);
    }
    for (int g = 0; g < 128; ++g) { const int k = g & 1, m = g < 24 ? g % 6 / 2 : g < 48 ? g / 8 - 3 : g / 16, nn = g < 24 ? g / 6 : g < 48 ? g / 2 % 4 : g / 4 % 4, s = g < 48 ? g >= 24 : g / 2 % 2;
      mf[NG - 128 + g] = Op{1, (short)(m * 4 + nn), (short)((s ? FB0 : FB1) + (nn * 2 + k) * 4), (short)((m < 4 ? s ? FA2 : FA0 : s ? FA3 : FA1) + (m % 4 * 2 + k) * 4), 0}; }
    for (int i = 0; i < 8; ++i) { put(NG - 124, RA, 97 + i / 6, FA2 * 16 + i, set(L), 0); if (i == 1) { rb(NG - 124, 0, 8, FB0, set(L), 2, 92); rb(NG - 124, 8, 16, FB0, set(L), 2, 89); } }
    for (int i = 0; i < 16; ++i) put(NG - 84 + (i >> 2) * 16 + (i & 3), RA, 93 + (i >> 2), (i & 2 ? FA3 : FA1) * 16 + (i >> 2) * 2 + (i & 1), set(L - 1 + ((i >> 1) & 1)), 1);
    at(15, WT, -1, -1, 91); at(127, WT, -1, -1, 90); at(143, WT, -1, -1, 91);
    for (int q = 8; q < NQ - 4; q += 2) at(q * 32 + 15, WT, -1, -1, 91);
    at(NG - 117, WT, -1, -1, 91); at(NG - 105, WT, -1, -1, 92); at(NG - 101, WT, -1, -1, 89); at(NG - 81, WT, -1, -1, 98);
    for (int i = 0; i < 4; ++i) at(NG - 65 + i * 16, WT, -1, -1, 93 + i);
    fill(9, 2, 0, 0); fill(33, 10, 2, 0); fill(65, 10, 2, 2); fill(97, 10, 1, 0); fill(129, 10, 0, 1); fill(161, 10, 2, 1); fill(193, 10, 0, 2);
    for (int r = -1; r < L; ++r) { const int g = (9 + 2 * r) * 32; if (r < 0) fill(g + 1, 8, 1, 1); else fill(g + 1, 8, 2, r + 3); fill(g + 21, 8, 1, r + 3); fill(g + 41, 8, 0, r + 4); }
    for (int t = 0; t < 2; ++t) for (int i = 0; i < 4; ++i) put(t * 128 + 50 + 2 * i, ZB, i, t);
    { int g = 6 * 32 + 2;
      for (int i = 0; i < 208; ++i) {
        if (i % 13 == 12) { g += 1; while (g % 4 != 1 || n[g] || g < 6 * 32 + 13 + i / 13 * XQ) ++g; }
        g = put(g, XS, i); ok = ok && (i < 104 || g >= 7 * 32 + 8);
      } }
    { int g = count(false, 64); ok = ok && g > 0; g = put(g - 3, PB, 0) + 4; while (n[g] > 1 || va(op[g][0])) ++g; at(g, PB, 1); ok = ok && g < NG - FG;
      for (int i = 0; i < 21; ++i) g = put(g + 1, OA, i);
      for (int i = 0; i < 4; ++i) g = put(g + 1, ID, i); }
    int gp[16] = {};
    { int g = put(NG - FG, FL, 1); ok = ok && put(g - 16, FL, 0) < g; g += LAGF;
      g = put(g, WT, 65, -1, -1); g = put(g, FC, 0); g = put(g + 1, FC, 1); ++g;
      do ++g; while (n[g] || g % 4 == 1);
      at(g, FC, 2); at(g, NONE);
      for (int i = 0; i < 16; ++i) { do ++g; while (g % 4 != 1 || n[g] > 1 || vm(op[g][0]) || va(op[g][0])); gp[i] = at(g, PR, i); } }
    for (int G = 0; G <= 8; ++G) for (int c = 0; c < 4; ++c) {
      if (G < 8) for (int e = 0; e < 4; ++e) ou[no++] = Op{OU, (short)G, (short)(e >> 1), (short)c, (short)(e & 1)};
      if (G > 0 && c == 2) ou[no++] = Op{WT, -1, -1, (short)(99 + G), 0};
      if (G > 0 && c >= 2) ou[no++] = Op{OU, (short)(G - 1), 5, (short)(c - 2), 0};
      if (G < 8) ou[no++] = Op{OU, (short)G, 2, (short)c, 0};
      if (G < 8 && c == 3) for (int e = 0; e < 2; ++e) ou[no++] = Op{OU, (short)G, 3, (short)e, 0};
    }
    for (int g = 0; g < NG; ++g) if (mf[g].k) fin[mf[g].a] = g;
    for (int g = 0; g < NX; ++g) {
      if (g < NG) { ++st; if (mf[g].k) lw[mf[g].a] = st; }
      st += n[g];
      out(g);
      if (g < NG - 4 && !room[g] && (!n[g] || (op[g][n[g] - 1].k != BR && op[g][0].k != BR))) {
        int c = 0, u[8] = {}, f = 0;
        while (f < 8 && ad[f * 4]) ++f;
        for (int G = f; G < 8 && c < AB; ++G) {
          bool y = !ad[G * 4] && g >= gp[G * 2 + 1] + LAGP;
          for (int i = 0; i < 4; ++i) y = y && st + 1 + c * 4 + i - lw[G * 4 + i] >= 6 && mf[g + 1 + i].a / 4 != G;
          if (y) u[c++] = G; }
        if (c && u[0] == f && (c > 1 || (g >= fin[f * 4 + 3] && io >= f * 25 - 14) || f == 7)) {
          at(g, WT, 71 + u[c - 1] * 2, -1, -1); ++st;
          for (int i = 0; i < c * 4; ++i) { const int T = u[i / 4] * 4 + i % 4; at(g, AD, T); lw[T] = ++st - 3; ad[T] = 1; out(g); } }
      }
    }
    ok = ok && io == no;
    count(true, 0);
  }
};
constexpr Plan PLAN;
static_assert(PLAN.ok, "");
template <int k, int a, int b, int c, int d> DI void emit() {
  if constexpr (k == RA) rdA<b / 16, b % 16, c, d * U>();
  if constexpr (k == RB) rdB<b / 16, b % 16, c, d * U>();
  if constexpr (k == M0) m0<a, b * U>();
  if constexpr (k == DA) dmaA<VRA + (b & 1), SWS, c, b>();
  if constexpr (k == DB) dmaB<VRB + b, c, b>();
  if constexpr (k == WT) wait<a, c>();
  if constexpr (k == BR) Y("s_barrier");
  if constexpr (k == RN && b < 3) X("s_add_u32 s[%0], s[%1], %2", "n"(a), "n"(SRA + b), "n"(b < 2 ? offA(rsrc(c), rcol(c), ~c & 1) : offW(rsrc(c), rcol(c), ~c & 1)));
  if constexpr (k == RN && b == 3) X("s_and_b32 s[%0], s[%0], 0x7ffffff", "n"(a));
  if constexpr (k == ZB) { if constexpr (b) X("v_and_b32 v[%0], 0xffffbfff, v[%0]", "n"(VZ + 2 + a)); else X("v_xor_b32 v[%0], 0x4000, v[%0]", "n"(VZ + 2 + a)); }
  if constexpr (k == XS) { constexpr int i = a / 13, e = a % 13, o = 64 + (i & 1) * 4;
    if constexpr (e == 12) X("global_store_dwordx4 v[%0], v[%1:%1+3], " sXD " offset:%2 sc0 sc1", "n"(56 + (i >> 3)), "n"(o), "n"((i & 7) * 1024 - 4096));
    else if constexpr (e >= 8) X("v_cvt_pk_bf16_f32 v[%0], v[%1], v[%1+1]", "n"(o + e - 8), "n"((e - 8) * 2));
    else X("v_accvgpr_read_b32 v[%0], a[%1]", "n"(e), "n"(i * 8 + e)); }
  if constexpr (k == PB) { if constexpr (a) Y("buffer_store_dword v62, off, " sWS ", " sXP " sc0 sc1"); else Y("v_mov_b32 v62, " sCALL); }
  if constexpr (k == FL) { if constexpr (a) Y("buffer_load_dword v61, off, " sWS ", " sXF " sc0 sc1"); else Y("buffer_load_dword v60, off, " sWS ", 0 sc1"); }
  if constexpr (k == FC) {
    if constexpr (a == 0) Y("v_readfirstlane_b32 " sMORE ", v60");
    if constexpr (a == 1) Y("v_readfirstlane_b32 s84, v61");
    if constexpr (a == 2) Y("s_cmp_lt_u32 s84, " sCALL N "s_cbranch_scc1 .Lx_slow\n.Lx_back:"); }
  if constexpr (k == PR) X("buffer_load_dwordx4 v[%0:%0+3], v56, " sWS ", s[%2] offen offset:%1 sc0 sc1", "n"(64 + a * 4), "n"((a & 3) * 1024), "n"(24 + a / 4));
  if constexpr (k == AD) X("v_mfma_f32_4x4x4_16b_bf16 v[%0:%0+3], v[8:9], v[%1:%1+1], v[%0:%0+3]", "n"(128 + a * 4), "n"(64 + a * 2));
  if constexpr (k == ID) { if constexpr (a == 0) Y("v_and_b32 v10, 48, v56"); if constexpr (a == 1) Y("v_mov_b32 v8, 0x3f80"); if constexpr (a == 2) Y("v_mov_b32 v9, 0");
    if constexpr (a == 3) Y("v_lshlrev_b64 v[8:9], v10, v[8:9]"); }
  if constexpr (k == OA && a == 20) Y("s_mov_b32 " sT0 ", 0");
  if constexpr (k == OA && a >= 16 && a < 20) X("s_add_u32 s[%0], " sXS ", %1", "n"(8 + a), "n"((a - 16) * 4096));
  if constexpr (k == OA && a < 16) X("s_xor_b32 s[%0], " sOF ", %1", "n"(8 + a), "n"(((a >> 3) * 128 + (a & 7) * 8) * (NCOL * 2)));
  if constexpr (k == OU) { constexpr int t = c * 2 + d, r = 128 + (a * 4 + c) * 4 + d * 2, o = FO + (a * 2 + c) * 4;
    if constexpr (b == 0) X("v_cvt_pk_bf16_f32 v[%0], v[%1], v[%1+1]", "n"(t), "n"(r));
    if constexpr (b == 1) X("v_pk_max_i16 v[%0], v[%0], 0", "n"(t));
    if constexpr (b == 2) X("ds_write_b64 v[%0], v[%1:%1+1] offset:%2", "n"(24 + c), "n"(t), "n"((a & 1) * 2048));
    if constexpr (b == 3) X("ds_read_b128 a[%0:%0+3], v28 offset:%1", "n"(o), "n"((a & 1) * 2048 + c * 1024));
    if constexpr (b == 5) X("buffer_store_dwordx4 a[%0:%0+3], v29, " sOUT ", s[%1] offen sc1", "n"(d ? FO : o), "n"(8 + a * 2 + c)); }
  if constexpr (k == NP) X("s_nop %0", "n"(a));
}

template <int i> DI void zero() {
  if constexpr (i & 1) X("v_mfma_f32_32x32x4_2b_bf16 v[%0:%0+31], v[60:61], v[60:61], 0", "n"(128 + i / 2 * 32));
  else X("v_mfma_f32_32x32x4_2b_bf16 a[%0:%0+31], v[60:61], v[60:61], 0", "n"(i / 2 * 32));
}
#define CALL(n) "s_waitcnt vmcnt(" #n ")" N "v_readfirstlane_b32 s4, v63" N "s_add_u32 " sCALL ", s4, 1" N "s_lshl_b32 s5, " sCALL ", 24" N "s_and_b32 s5, s5, 0x1000000" N \
    "s_add_u32 s44, s44, s5" N "s_xor_b32 s45, s44, 0x4000" N
template <int V> DI void go() {
  wait<V, -1>(); Y("s_barrier");
  REP(i, 0, 8, rdA<FA0, i, VX, 0>()); REP(i, 0, 12, rdB<FB1, i, VX, 2 * U>()); REP(i, 0, 9, rdB<FB0, i, VZ, 0>());
}
#define TAKE(n) "v_mov_b32 v78, " #n N "buffer_atomic_add v78, v93, " sWS ", s22 offen sc0" N

__global__ __launch_bounds__(256, 1) void a2a_gemm(Params) {
  asm volatile("s_load_dwordx16 s[52:67], s[0:1], 0" N "s_getreg_b32 s3, hwreg(HW_REG_XCC_ID, 0, 3)" N "s_and_b32 s4, s2, 0xf8" N "s_mov_b32 s15, 0"
    :: "{s[0:1]}"(__builtin_amdgcn_kernarg_segment_ptr()), "{s2}"(blockIdx.x), "{v0}"(threadIdx.x) : "v255", "a255", "s101", "vcc", "memory");
  X("s_or_b32 s3, s3, s4\n.Lstart:" N "s_and_b32 s4, s3, 4" N "s_lshr_b32 s5, s3, 6" N "s_add_u32 s4, s4, s5" N "s_lshl_b32 s4, s4, 5" N
    "s_load_dwordx8 s[88:95], s[0:1], s4 offset:0x40" N "s_getpc_b64 s[16:17]\n.Lpc:" N "v_readfirstlane_b32 s14, v0" N "v_mov_b64 v[60:61], 0" N
    "v_lshrrev_b32 v9, 4, v0" N "s_and_b32 s4, s3, 0xf8" N "s_lshl_b32 s4, s4, 8" N "v_lshl_add_u32 v5, v9, 7, s4" N "v_min_u32 v5, .Lend - .Lpc - 4, v5" N
    "global_load_dword v65, v5, s[16:17]" N "s_bfe_u32 s9, s3, 0x10002" N "s_and_b32 s13, s3, 3" N "s_bfe_u32 s10, s3, 0x20003" N
    "s_xor_b32 s12, s13, s9" N "s_lshr_b32 s5, s3, 5" N "s_lshl3_add_u32 s12, s12, s5" N "s_lshl_b32 s12, s12, 9" N "s_and_b32 s11, s5, 1" N
    "s_lshr_b32 s8, s14, 6" N "s_lshl_b32 s4, s2, 5" N "s_add_u32 s4, s4, s8" N "s_lshl2_add_u32 " sCTR ", s4, %0" N
    "v_and_b32 v1, 63, v0" N "v_lshlrev_b32 v56, 4, v1" N "v_lshrrev_b32 v4, 3, v1" N
    "s_brev_b32 s14, s14" N "s_lshr_b32 s77, s14, 12" N "s_lshr_b32 s4, s14, 8" N
    "v_lshrrev_b32 v3, 4, v1" N "v_bitop3_b32 v2, v1, v3, 7 bitop3:0x6c" N "v_lshl_add_u32 v72, v4, 11, s4" N
    "v_lshl_add_u32 v72, v2, 4, v72" N "v_xor_b32 v5, 64, v72" N "v_add_u32 v73, 0x3c00, v5" N "v_add_u32 v74, 0x7800, v72" N "v_add_u32 v75, 0xb400, v5" N
    "s_lshr_b32 s4, s14, 6" N "s_add_u32 s4, s4, s12" N "v_bfe_u32 v6, v1, 1, 3" N
    "v_xor_b32 v6, v6, v3" N "v_lshl_add_u32 v52, v3, 14, s4" N "v_lshl_add_u32 v52, v6, 5, v52" N
    "v_and_or_b32 v52, v56, 16, v52" N "v_xor_b32 v5, 0x80, v52" N "v_add_u32 v53, 0xfc00, v52" N "v_add_u32 v54, 0x1f800, v5" N "v_add_u32 v55, 0x2f400, v5" N
    "s_lshl1_add_u32 s4, s10, s11" N "s_lshl_b32 s4, s4, 8" N "s_or_b32 s4, s4, s9" N "s_lshl_b32 s78, s4, 10" N "s_mov_b32 m0, s77" N
    "s_xor_b32 s4, s10, s8" N "s_and_b32 s4, s4, 1" N
    "s_lshl_b32 s18, s4, 16" N "s_or_b32 s18, s18, s77" N "s_bitset1_b32 s18, 15" N
    "s_xor_b32 s4, s4, s9" N "s_lshl_b32 s4, s4, 8" N "s_lshl_b32 s5, s9, 23" N "s_or_b32 s80, s4, s5", "n"(WS_CTR));
#define CTR "buffer_load_dword v63, off, " sWS ", " sCTR " sc1"
  X("s_cmp_lg_u32 s8, 2" N "s_cbranch_scc1 .Lall" N
    "s_lshr_b32 s4, s3, 1" N "s_xor_b32 s4, s4, s3" N "s_and_b32 s4, s4, 2" N "s_lshl_b32 s4, s4, 9" N "v_or_b32 v5, s4, v56" N "v_xor_b32 v7, 0x400, v5" N
    "v_add_u32 v5, .Ltab - .Lpc, v5" N "v_add_u32 v7, .Ltab - .Lpc, v7" N "global_load_dwordx4 v[80:83], v5, s[16:17]" N "global_load_dwordx4 v[84:87], v7, s[16:17]" N
    "v_min_u32 v79, 1, v1" N "v_lshlrev_b32 v93, 30, v79" N "s_lshl_b32 s22, s3, 7" N "s_or_b32 s22, s22, s15" N "s_add_u32 s22, s22, %0" N
    "v_add_u32 v89, %1 + 16, v56" N "s_waitcnt lgkmcnt(0)" N CTR N TAKE(1) ".Lall:" N "s_waitcnt lgkmcnt(0)", "n"(WS_ROLE), "n"(LDS_CTRL));
  REP(i, 0, 4, fill<0, i>());
  Y("s_mov_b32 m0, s18" N "s_andn2_b32 s47, s94, 7" N "s_or_b32 s80, s80, s47");
  REP(i, 0, 4, fill<1, i>());
  Y("s_xor_b32 m0, s18, 0x10000" N "s_xor_b32 s81, s80, 0x100");
  REP(i, 0, 4, fill<2, i>());
  Y(CTR);
  REP(i, 0, 4, zero<i>());
  Y("v_and_b32 v7, 0x70, v56");
  Y("s_bfe_u32 s6, s8, 0x10001" N "s_and_b32 s7, s8, 1" N "s_lshl_b32 s7, s7, 7" N "s_lshl_b32 s4, s6, 13" N
    "v_and_b32 v12, 15, v1" N "v_lshl_add_u32 v32, v6, 4, s4" N "v_lshl_add_u32 v32, v12, 7, v32" N "v_xor_b32 v33, 64, v32" N
    "v_bfe_u32 v5, v1, 2, 3" N "v_lshlrev_b32 v34, 5, v5" N "v_xor_b32 v34, s7, v34" N "v_and_b32 v5, 3, v5" N "v_lshl_add_u32 v34, v5, 8, v34");
  zero<4>();
  Y("v_lshl_add_u32 v34, v3, 11, v34" N "v_and_b32 v14, 3, v1" N "v_lshl_add_u32 v34, v14, 3, v34" N "v_xor_b32 v35, 32, v34" N "v_xor_b32 v36, 64, v34" N "v_xor_b32 v37, 0x60, v34");
  REP(i, 0, 12, X("v_add_u32 v[%0], %1, v[%2]", "n"(VY + i), "n"((i / 6 + 1) * 3 * U), "n"(VX + i % 6)); if constexpr (i == 6) zero<5>());
  X("s_lshl_b32 s4, s8, 12" N "s_add_u32 s4, s4, %0" N "v_lshrrev_b32 v5, 5, v1" N "v_bitop3_b32 v5, v1, v5, 7 bitop3:0x6c" N "v_lshl_add_u32 v24, v12, 7, s4" N
    "v_lshl_add_u32 v24, v5, 4, v24" N "v_and_b32 v5, 1, v3" N "v_lshl_add_u32 v24, v5, 3, v24", "n"(9 * U));
  zero<6>();
  X("v_xor_b32 v25, 32, v24" N "v_xor_b32 v26, 64, v24" N "v_xor_b32 v27, 0x60, v24" N
    "v_bitop3_b32 v5, v1, v4, 7 bitop3:0x6c" N "v_lshl_add_u32 v28, v4, 7, s4" N "v_lshl_add_u32 v28, v5, 4, v28" N
    "s_lshl2_add_u32 s4, s10, s6" N "s_lshl_b32 s4, s4, 20" N "s_lshl_b32 s5, s9, 8" N "s_or_b32 s46, s47, s5" N "s_or_b32 s4, s4, s5" N "s_or_b32 s4, s4, s7" N "s_or_b32 s4, s4, s12" N
    "v_lshl_add_u32 v29, v4, 14, s4" N "v_add_u32 v29, v29, v7" N "v_mov_b32 v68, %0", "n"(LDS_CTRL));
  zero<7>();
  X("v_lshl_add_u32 v50, v4, 7, s77" N "v_lshl_add_u32 v50, v2, 4, v50" N "s_bfm_b64 exec, 8, 24" N "v_add_u32 v50, %0, v50" N "v_add_u32 v56, %0, v56" N "s_mov_b64 exec, -1" N
    "v_xor_b32 v51, 64, v50" N "v_add_u32 v57, 0x2000, v56" N
    "s_lshl2_add_u32 s4, s3, s8" N "s_lshl_b32 s4, s4, 14" N "s_add_u32 " sXS ", s4, %1" N "s_xor_b32 s4, " sXS ", 0x51000" N "s_add_u32 s68, s52, s4" N "s_addc_u32 s69, s53, 0" N
    "s_lshl_b32 s4, s3, 5" N "s_add_u32 s4, s4, s8" N "s_lshl2_add_u32 " sXF ", s4, %2" N "s_xor_b32 " sXP ", " sXF ", 0x280" N
    "s_lshl_b32 " sOF ", s11, 21" N "s_mov_b32 " sT ", 0" N "s_mov_b32 s74, s77" N "s_add_u32 s75, s77, 0xc000" N
    "s_lshl_b32 s21, s10, 19" N "s_lshl_b32 s4, s11, 14" N "s_or_b32 s4, s4, s21" N "s_add_u32 s44, s4, %3", "n"(AWAY), "n"(WS_XCH), "n"(WS_XFLAG), "n"(WS_A));
  Y("s_cmp_eq_u32 s8, 2" N "s_cbranch_scc1 .Lc_go" N "s_cmp_eq_u32 s8, 3" N "s_cbranch_scc1 .Lp_go" N "s_branch .Ld_go\n"
    ".p2align 6\n.fill ((.Ld_go - .Ld0) & 63) / 4, 4, 0xbf800000\n.Ld_go:");
  {
    REP(i, 0, 4, X("v_add_u32 v[%0+4], 0x10000, v[%0]" N "v_add_u32 v[%1], 0x40000, v[%2]", "n"(VDA + i), "n"(VDB + 4 + i), "n"(VRB + i)));
    Y("s_xor_b32 s79, s78, 0x40000" N "s_add_u32 s92, s78, 128" N "s_mov_b32 s88, s18" N "s_xor_b32 s89, s18, 0x10000" N
      "s_lshr_b32 s4, s18, 1" N "s_and_b32 s4, s4, 0x8000" N "s_or_b32 s4, s4, s77" N "s_or_b32 s90, s4, 0x14000" N "s_xor_b32 s91, s90, 0x8000" N
      "s_add_u32 s93, s80, 0x100000" N "s_add_u32 s94, s81, 0x100000" N "s_add_u32 s95, s79, 128" N "s_add_u32 s96, s78, 0x100" N
      "s_add_u32 s97, s80, 0x200000" N "s_add_u32 s98, s81, 0x200000");
    Y("s_add_u32 m0, s77, 0x4000" N "s_add_u32 s99, s79, 0x100");
    REP(i, 0, 4, fill<0, i, SA1>());
    Y("s_add_u32 m0, s77, 0x5000" N "s_lshl_b32 s86, s8, 8");
    REP(i, 4, 8, fill<0, i, SA1>());
    Y("s_add_u32 m0, s77, 0xc000" N "s_mov_b32 " sEV ", 6");
    REP(i, 0, 4, fill<0, i, 92>());
    Y("s_add_u32 m0, s77, 0xd000" N "s_mov_b32 " sAH ", 0x40000");
    REP(i, 4, 8, fill<0, i, 92>());
    go<17>(); wait<-1, 13>();
    stile<4, 0>(); Y("s_cbranch_scc1 .Lexit"); stile<4, 1>();
    SLOOP(0, ".Ld", ".Ld_ev");
    Y(".Ld_ev:" N LEAVE "s_mov_b64 s[60:61], s[52:53]" N "s_mov_b32 s62, s54" N "s_mov_b32 " sAH ", 0x4000");
    REP(i, 0, 8, X("v_add_u32 v[%0], %1, v[%2]", "n"(VDA + i), "n"((i >> 2) * 4096), "n"(VRA + (i & 1))));
    Y("s_branch .Ld0");

    X(".p2align 6\n.fill ((.Lc_go - .Lc0) & 63) / 4, 4, 0xbf800000\n.Lc_go:" N
      "s_lshl_b32 s4, s10, 18" N "s_xor_b32 s5, s12, 0x200" N "s_or_b32 s4, s4, s5" N "v_lshrrev_b32 v5, 2, v1" N "v_lshl_add_u32 v92, v5, 14, s4" N "v_lshl_add_u32 v92, v14, 7, v92" N
      "v_cmp_gt_u32 vcc, %[pn], v1" N "v_bfrev_b32 v90, 1" N "v_and_b32 v94, 15, v1" N "v_lshlrev_b32 v94, 2, v94" N "v_cndmask_b32 v92, v93, v92, vcc" N "v_mov_b32 v72, 0" N
      "s_mov_b32 s88, %[t0]" N "s_mov_b32 s89, %[t1]" N "s_mov_b64 " cK ", 0" N "s_mov_b32 " cCL ", 0x7f000000" N "s_mov_b32 " sEV ", %[tv] - 1" N
      "s_lshl_b32 s4, s10, 10" N "s_add_u32 " cFB ", s4, %[af]" N "s_lshl_b32 s4, s10, 7" N "s_or_b32 s4, s4, s12" N "s_add_u32 " cCM ", s4, %[cl]" N
      "s_andn2_b32 s5, s80, 0x800000" N "v_mov_b32 v6, s5" N "s_waitcnt vmcnt(13)" N CALL(13)
      "v_mov_b32 v5, s44" N "s_mov_b64 exec, 0x7f" N "v_mov_b32 v5, s78" N "v_mov_b32 v6, s80" N "s_mov_b64 exec, -1" N
      "v_and_b32 v76, 15, v83" N "v_lshl_add_u32 v76, s4, 4, v76" N "v_and_b32 v67, 15, v87" N "v_lshl_add_u32 v67, s4, 4, v67" N "s_mul_i32 " cTK ", s4, 2 * %[np]" N "s_add_u32 " cTK ", " cTK ", %[np]",
      [pn] "n"(64 / PFN), [t0] "n"((unsigned)TODO), [t1] "n"((unsigned)(TODO >> 32)), [tv] "n"(TV), [af] "n"(WS_AFLAG), [cl] "n"(WS_CLAIM), [np] "n"(N_POOL));
    REP(i, 0, 2, X("v_add_u32 v[%0], v[%0], v5" N "v_add_u32 v[%0+1], v[%0+1], v6" N "v_add_u32 v[%0+3], s47, v[%0+3]" N
      "v_and_b32 v[%0+1], 0x7ffffff, v[%0+1]" N "v_and_b32 v[%0+3], 0x87ff0000, v[%0+3]", "n"(80 + i * 4)));
    Y("v_readfirstlane_b32 s5, v78" N "s_cmp_eq_u32 s15, 0" N "s_cselect_b32 s5, s5, s4" N "s_cmp_lg_u32 s5, s4" N "s_cbranch_scc1 .Lc_taken\n.Lc_mine:");
    go<1>();
    X("v_mov_b32 v78, 1" N "v_mov_b32 v69, 1" N "v_mov_b32 v70, v90" N "v_mov_b32 v71, v90" N "s_lshl_b64 exec, %0, %2" N "v_mov_b32 v70, 0" N "s_lshl_b64 exec, %1, %2" N "v_mov_b32 v71, 0" N "s_mov_b64 exec, -1",
      "n"(DRA), "n"(DRB), "n"(DR0));
    wait<-1, 13>();
    SLOOP(1, ".Lc", ".Lc_ev");
#define VOTES "buffer_load_dword v95, v94, " sWS ", 0 offen offset:%[vo] sc1" N
    X(".Lc_ev:" N LEAVE "s_lshl_b32 " cB ", " sCALL ", 1" N "s_cmp_eq_u32 " sT ", %[tv]" N "s_cbranch_scc1 3f" N VOTES "s_mov_b32 " sEV ", %[tv]" N "s_mov_b32 " sT0 ", 0" N "s_branch .Lc0\n"
      "3:" N "v_cmp_le_u32 vcc, " cB ", v95" N "v_cmp_lt_u32 s[86:87], " cB ", v95" N "s_cmp_eq_u64 vcc, -1" N "s_cbranch_scc0 2f\n"
      "4:" N "s_bcnt1_i32_b64 " cB ", s[86:87]" N "s_cmp_gt_u32 " cB ", %[th]" N "s_cbranch_scc0 .Lc1" N
      "v_mov_b64 v[82:83], v[86:87]" N "v_mov_b64 v[80:81], v[84:85]" N "v_mov_b32 v76, v67" N "s_mov_b64 " cK ", 0" N SAMPLE N "s_branch .Lc1\n"
      "2:" N VOTES EXPIRED "s_cbranch_scc0 3b" N "s_branch 4b",
      [tv] "n"(TV), [vo] "n"(WS_VOTE), [th] "n"(VTH), DL);
    static_assert(TV == 5, "");
    Y(".Lc_slow:" N "s_mov_b32 " sT0 ", 0\n.Lc_again:");
    X(SAMPLE N EXPIRED "s_cbranch_scc0 5f" N "v_mov_b32 v76, 0\n5:" N ARRIVED N "s_cmp_eq_u64 " cK ", -2" N "s_cselect_b64 " cC ", 63, 62" N "s_or_b64 " cC ", " cC ", " cK N
      "s_and_b64 " cC ", " cC ", " cT N "s_cbranch_scc0 .Lc_again" N BACK(".Lc_back"), DL);
    Y(".Lc_taken:" N TAKE(-1) "s_add_u32 s22, s22, 128" N "s_bitset0_b32 s22, 15" N "s_waitcnt vmcnt(0)" N TAKE(1) "s_waitcnt vmcnt(0)" N "v_readfirstlane_b32 s5, v78" N
      "s_cmp_lg_u32 s5, s4" N "s_cbranch_scc1 .Lc_taken" N "v_mov_b32 v79, s22" N "s_branch .Lc_mine");

    Y(".p2align 6\n.fill ((.Lp_go - .Lpi0) & 63) / 4, 4, 0xbf800000\n.Lp_go:");
    X("s_lshl2_add_u32 s20, s10, s13" N "s_lshl_b32 s4, s20, 17" N "v_lshl_add_u32 v88, v4, 11, s4" N "v_add_u32 v88, v88, v7" N
      "s_and_b32 s4, s3, 0xc7" N "s_cmp_lg_u32 s4, %[sw]" N "s_addc_u32 s19, s11, 0" N "s_xor_b32 s4, s19, s94" N "s_bitcmp1_b32 s4, 0" N "s_cselect_b32 s4, 0x100, 0" N
      "s_mov_b64 s[16:17], s[88:89]" N "s_add_u32 s86, s88, s4" N "s_addc_u32 s87, s89, 0" N
      "s_and_b32 s19, s19, 1" N "s_cselect_b32 " pM ", 0, 0xfffc" N "s_xor_b32 " pM ", " pM ", %[em] - 2" N
      "s_lshl_b32 s4, s10, 6" N "s_or_b32 s4, s4, s19" N "s_xor_b32 s4, s4, 1" N "s_lshl2_add_u32 s4, s4, s13" N "s_lshl2_add_u32 s92, s4, s92" N "s_addc_u32 s93, s93, 0" N
      "s_lshl_b32 s4, s13, 13" N "s_or_b32 s7, s21, s4" N
      "s_and_b32 s6, s94, 7" N "s_mul_i32 s4, s20, 7" N "s_add_u32 s4, s4, s6" N "s_mul_i32 s4, s4, %[pr]" N "s_lshr_b32 s4, s4, 16" N "s_sub_i32 " pS ", s4, %[sh]" N
      "s_mul_i32 s4, s19, %[pe]" N "s_add_i32 s4, s4, " pS N "s_max_i32 s4, s4, 0" N "s_lshr_b32 s4, s4, 6" N "s_lshl_b32 " sEV ", s4, 1" N
      "s_lshl_b32 s14, s20, 2" N "s_bitcmp1_b32 s6, 0" N "s_cselect_b32 s4, 0x22222222, 0" N "s_xor_b32 s20, s4, 0x75643120" N "s_xor_b32 s21, s4, 0xfdecb9a8" N
      NEXT0 N NEXT1 N NEXT2 N NEXT3 N NEXT4 N
      "s_mov_b32 s94, 0" N "s_mov_b32 " pB ", 64" N "s_mov_b64 vcc, -1" N "s_lshl_b32 s19, s19, 15" N
      "s_cmp_lt_u32 s6, 7" N "s_cbranch_scc1 1f" N
      "v_mov_b32 v86, s14" N "s_lshl_b32 s4, s14, 7" N "s_or_b32 s4, s4, s47" N "s_xor_b32 s4, s4, 0x4800000" N
      "v_lshl_add_u32 v84, v14, 14, s4" N "v_add_u32 v85, 0x2000, v84" N "s_xor_b32 s4, s4, 0x200" N "v_and_b32 v5, 31, v1" N "v_lshl_add_u32 v83, v5, 14, s4" N
      "v_lshrrev_b32 v5, 5, v1" N "v_lshl_add_u32 v83, v5, 13, v83" N
      "s_add_u32 s6, s6, s11" N "s_cmp_eq_u32 s6, 8" N "s_cselect_b32 " sEV ", %[tp], -1" N "s_mov_b64 vcc, 0\n1:",
      [sw] "n"(SWP), [pr] "n"(PHASE), [pe] "n"(PER), [em] "n"(EM), [sh] "n"(SHIFT), [tp] "n"(TP));
    REP(i, 1, 8, X("v_add_u32 v[%0], %1, v88", "n"(88 + i), "n"(i * 16384)));
    REP(i, 0, 8, X("global_load_dwordx4 v[%0:%0+3], v[%1], s[86:87] nt", "n"(96 + i * 4), "n"(88 + i)));
    go<9>();
    X(CALL(8) "s_add_u32 s5, s5, s7" N "s_add_u32 s22, s90, s5" N "s_addc_u32 s23, s91, 0" N "s_add_u32 s90, s22, s19" N "s_addc_u32 s91, s23, 0" N
      "s_lshl_b32 s4, s4, 4" N "v_mov_b32 v81, s4" N
      "s_cmp_eq_u32 " sEV ", %0" N "s_cselect_b64 exec, -1, 0" N "global_load_dword v82, v83, s[56:57]" N "s_mov_b64 exec, -1", "n"(TP));
    wait<-1, 13>();
    Y("s_cmp_lg_u32 " sEV ", 0" N "s_cbranch_scc1 .Lpi0" N "s_waitcnt vmcnt(0)\n.Lpp0:");
    stile<3, 0>(); Y("s_cbranch_scc1 .Lp_ev"); stile<3, 1>(); Y("s_cbranch_scc1 .Lpp0");
    Y(".Lpi0:"); stile<2, 0>(); Y("s_cbranch_scc1 .Lp_ev\n.Lpi1:"); stile<2, 1>();
    X("s_cbranch_scc0 .Lpi0" N "s_cbranch_vccnz .Lpp0\n.Lp_ev:" N LEAVE
      "s_mov_b32 s98, 0" N "s_mov_b32 s97, 2\n"
      "4:" N "s_mov_b64 exec, vcc" N "global_load_dword v82, v84, s[56:57]" N "global_load_dword v83, v85, s[56:57]" N "s_waitcnt vmcnt(1)" N "s_memtime s[88:89]" N
      "s_waitcnt vmcnt(0)" N "s_memtime s[90:91]" N "v_swap_b32 v84, v85" N "v_add_u32 v84, %0, v84" N "v_add_u32 v85, %0, v85" N "s_mov_b64 exec, -1" N "s_waitcnt lgkmcnt(0)" N
      "s_sub_u32 s88, s90, s88" N "s_min_i32 s88, s88, 400" N "s_and_b32 s88, s88, vcc_lo" N "s_sub_i32 s98, s88, s98" N "s_mov_b64 vcc, -1" N
      "s_sub_u32 s97, s97, 1" N "s_cbranch_scc0 4b" N
      "s_getreg_b32 s88, hwreg(HW_REG_XCC_ID, 2, 1)" N "s_lshr_b32 s98, s98, 31" N "s_mov_b64 vcc, 0" N "s_xor_b32 s88, s88, s98" N "s_xor_b32 s88, s88, 1" N "s_lshl1_add_u32 s88, " sCALL ", s88" N
      "v_mov_b32 v82, s88" N "buffer_store_dword v82, v86, " sWS ", 0 offen offset:%1 sc1" N BACK(".Lpi"), "n"(4 * 16384), "n"(WS_VOTE));

    X(".Lx_slow:" N "buffer_load_dword v61, off, " sWS ", " sXF " sc0 sc1" N EXPIRED "s_cselect_b32 " sMORE ", " sCALL ", " sMORE N "s_cbranch_scc1 .Lx_back" N
      "v_readfirstlane_b32 s84, v61" N "s_cmp_lt_u32 s84, " sCALL N "s_cbranch_scc1 .Lx_slow", DL);
    Y("s_branch .Lx_back");

    Y(".Lexit:" N "s_cmp_lg_u32 " sMORE ", 0" N "s_cbranch_scc1 .Lagain" N "s_bitcmp1_b32 " sT ", 0" N "s_cbranch_scc0 .Lt" N
      "s_mov_b32 s84, s74" N "s_mov_b32 s74, s75" N "s_mov_b32 s75, s84");
    REP(i, 0, 6, X("v_swap_b32 v[%0], v[%1]", "n"(VX + i), "n"(VY + i)));
    REP(i, 0, 4, emit<ZB, i, 0, 0, 0>());
    Y(".Lt:");
    REP(q, 0, NX / 16, REP(j, 0, 16, constexpr int g = q * 16 + j;
      if constexpr (g < NG) { constexpr Op f = PLAN.mf[g < NG ? g : 0]; mfma<f.k, f.a, f.b, f.c>(); }
      REP(i, 0, PLAN.n[g], constexpr Op o = PLAN.op[g][i]; emit<o.k, o.a, o.b, o.c, o.d>())));
    Y("s_cmp_eq_u32 " sMORE ", " sCALL N "s_cbranch_scc0 .Lfin" N "s_waitcnt vmcnt(0)" N "v_mov_b32 v0, 0x7fc07fc0");
    REP(i, 0, 4, X("v_accvgpr_write_b32 a[%0], v0", "n"(FO + i)));
    REP(i, 0, 16, emit<OU, i / 2, 5, i % 2, 1>());
    Y(".Lfin:" N "v_mov_b32 v0, " sCALL N "buffer_store_dword v0, off, " sWS ", " sCTR " sc1");
    Y("s_endpgm\n.Lagain:" N "s_waitcnt vmcnt(0) lgkmcnt(0)" N "s_barrier" N "s_bfe_u32 s3, " sMORE ", 0x80007" N "s_brev_b32 s15, 4" N "s_branch .Lstart\n.p2align 7\n.Ltab:");
    REP(q, 0, 2, REP(l, 0, 64, X(".long %0, %1, %2, %3", "n"(table(q, l, 0)), "n"(table(q, l, 1)), "n"(table(q, l, 2)), "n"(table(q, l, 3)))));
    Y(".Lend:");
  }
}

extern "C" void kernel_launch(int rank, int nranks, ncclComm_t comm,
                              hipStream_t stream,
                              void* const* d_in, const int* in_sizes, int n_in,
                              void* d_out, int out_size,
                              const int* in_dtypes, int out_dtype,
                              void* const* d_out_all,
                              void* const* d_ws_all, size_t ws_bytes) {
  if (nranks != NRANK || (unsigned)rank >= (unsigned)NRANK || n_in < 2 || ws_bytes < WS_NEED ||
      in_sizes[0] != 8192 * 1024 || in_sizes[1] != 8192 * 8192 || out_size != 1024 * 8192) {
    fprintf(stderr, "a2a_gemm: unexpected problem shape\n");
    return;
  }
  HIP_CHECK(hipFuncSetAttribute((const void*)a2a_gemm, hipFuncAttributeMaxDynamicSharedMemorySize, LDS_BYTES));
  const char* x = (const char*)d_in[0];
  char* ws = (char*)d_ws_all[rank];
  Params p = {{ws, (unsigned)WS_NEED, 0x20000}, {d_in[1], 8192 * 16384, 0x20000}, {x + rank * BLK_B, (unsigned)BLK_B, 0x20000}, {d_out, 1024 * 16384, 0x20000}, {}};
  p.duty[3] = {x, ws, ws, (unsigned)rank << 24 | 7, 0x20000};
  for (int j = 0, n[2] = {0, 4}; j < 7; ++j) {
    const int q = (rank + 1 + j) & 7, e = n[((0xb2 >> rank) ^ (0xb2 >> q)) & 1]++;
    char* to = (char*)d_ws_all[q];
    p.duty[e] = {x + q * BLK_B, to + WS_A + (6 - j) * BLK_B + 4096, to + WS_AFLAG + (6 - j) * 128, (unsigned)rank << 24 | j, 0x20000};
  }
  a2a_gemm<<<256, 256, LDS_BYTES, stream>>>(p);
  HIP_CHECK(hipGetLastError());
}
```
